```python
import numpy as np

kernel_name = "gpu_a2a_gemm_m4096_k4096_n32768_f32_none"


ABSMAX_RTOL = 0.001

COMPUTE_DTYPE = "float32"
EPILOGUE = "none"
IN_QUANT = None
OUT_QUANT = None
IN_DTYPES = {"x": "float32", "w_mat": "float32"}
WORLD_SIZE = 8

def setup_inputs():

    x = np.random.randn(4096, 4096).astype(np.float32)
    w_mat = (np.random.randn(4096, 32768) / np.sqrt(4096)).astype(np.float32)
    return {"x": x, "w_mat": w_mat}

def _epilogue(y):
    return y

def reference(x, w_mat):
    y = (x @ w_mat).astype(np.float32)
    return _epilogue(y)
```

```cpp
#include <hip/hip_runtime.h>
#include <rccl/rccl.h>
#include <cstdio>

typedef unsigned u32x4 __attribute__((ext_vector_type(4)));
constexpr unsigned NR = 8;
constexpr unsigned XPER = 0x20000, XGRP = 0x40000, XBUF = 32 * XGRP;
constexpr unsigned WS_SEQ = 0;
constexpr unsigned WS_FLAG = 32768;
constexpr unsigned WS_LOC = 32768 + 2048;
constexpr unsigned WS_TMO = 32768 + 4096;
constexpr unsigned WS_X = 1u << 20;
struct Ptrs { char* ws[NR]; };
__device__ __forceinline__ u32x4 mk_srd(const void* p, unsigned bytes) {
  unsigned long long a = (unsigned long long)p;
  u32x4 r; r.x = (unsigned)a; r.y = (unsigned)(a >> 32) & 0xffff; r.z = bytes; r.w = 0x20000; return r;
}

__global__ __launch_bounds__(256, 1) void a2a_gemm(const float* __restrict__ x, const float* __restrict__ W, float* __restrict__ out,
                                                   char* myws, unsigned rank, Ptrs P) {
  __shared__ __attribute__((aligned(1024))) char smem[131072 + 1024 + 24576];
  const unsigned lane = threadIdx.x & 63;
  const unsigned wave = __builtin_amdgcn_readfirstlane(threadIdx.x >> 6);
  const unsigned bid = blockIdx.x;
  const unsigned panel = (bid & 7) * 32 + (bid >> 3);
  const unsigned dst = (bid + (bid >> 3)) & 7, mt = bid >> 3;
  const unsigned q = lane & 31, hh = lane >> 5;
  const unsigned ldsb = (unsigned)(size_t)(__attribute__((address_space(3))) char*)smem;
  if (ldsb != 0) return;
  char* dws = P.ws[dst];
  u32x4 srdw = mk_srd(W, 4096u * 32768u * 4u), srdo = mk_srd(out, 512u * 32768u * 4u);
  u32x4 srdxs = mk_srd(x, 4096u * 512u * 4u), srdws = mk_srd(myws, WS_X);
  const unsigned long long xb = (unsigned long long)(myws + WS_X);
  const unsigned long long db = (unsigned long long)(dws + WS_X + rank * 4 * XGRP + (wave >> 1) * XPER + ((mt & 7) * 2 + (wave & 1)) * 1024);
  const unsigned long long sflag = (unsigned long long)(dws + WS_FLAG + rank * 128);
  const unsigned snself = dst ^ rank;
  const unsigned nop = (snself - 1) >> 24;
  const unsigned sfpc = (2 + mt % 3) | nop, sfph = (1 + mt % 3) | nop;
  const unsigned sfs0 = (nop & 29u) | ((0u - wave) >> 24);
  const unsigned lp = bid & 7;
  const unsigned sstag = -10 + (((64 + (mt >> 1) * 6 + ((lp * 3) >> 2) + ((0xaaaa2a00u >> mt) & 1) * (64 + (mt >> 1) * 2) - ((0xaau >> mt) & 1) * (64 + (mt >> 1) * 2)) * 180) >> 8);
  unsigned vw0 = (wave * 16 + hh * 8) * 131072u + panel * 512 + q * 16;
  unsigned vw1 = vw0 + 131072u, vw2 = vw0 + 2 * 131072u, vw3 = vw0 + 3 * 131072u;
  unsigned vxs = ((mt * 16 + (lane & 15)) * 512 + wave * 32 + (lane >> 4) * 8) * 4;
  unsigned vx0, vx1, vlw, vlr, vpad, vpd, vtab;
  unsigned bad;
  asm volatile(
      "s_nop 4\n\t"
      "s_memrealtime s[78:79]\n\t"
      "buffer_load_dwordx2 v[254:255], off, %[srdws], %[sseq] sc1\n\t"
      "s_cmp_gt_u32 %[swave], 1\n\t"
      "s_cbranch_scc1 Lp1_%=\n\t"
      "buffer_load_dwordx4 v[118:121], %[vxs], %[srdxs], %[sxl] offen nt\n\t"
      "buffer_load_dwordx4 v[122:125], %[vxs], %[srdxs], %[sxl] offen offset:16 nt\n\t"
      "Lp1_%=:\n\t"
      "buffer_load_dwordx4 v[78:81], %[vxs], %[srdxs], %[sxr] offen offset:0 nt\n\t"
      "buffer_load_dwordx4 v[82:85], %[vxs], %[srdxs], %[sxr] offen offset:16 nt\n\t"
      "s_mov_b32 s61, %[sg0]\n\t"
      "s_mov_b32 s75, 0\n\t"
      "s_mov_b32 s62, 0\n\t"
      "s_and_b32 s63, %[sseq], 0x380\n\t"
      "s_add_u32 s63, s63, 34816\n\t"
      "v_mov_b32 v130, s63\n\t"
      "v_mov_b32 v131, 1\n\t"
      "v_mbcnt_lo_u32_b32 v138, -1, 0\n\t"
      "v_mbcnt_hi_u32_b32 v138, -1, v138\n\t"
      "v_lshlrev_b32 %[vlr], 4, v138\n\t"
      "s_lshr_b32 s63, %[sseq], 13\n\t"
      "s_lshl_b32 s84, s63, 14\n\t"
      "v_add_u32 %[vpd], s84, %[vlr]\n\t"
      "s_lshl_b32 s84, %[swave], 14\n\t"
      "v_add_u32 %[vx0], s84, %[vlr]\n\t"
      "s_bfm_b64 exec, 8, 24\n\t"
      "s_lshl_b32 s84, s63, 8\n\t"
      "s_add_u32 s84, s84, 65152\n\t"
      "v_add_u32 %[vpd], s84, %[vlr]\n\t"
      "s_lshl_b32 s84, %[swave], 8\n\t"
      "s_add_u32 s84, s84, 65152\n\t"
      "v_add_u32 %[vx0], s84, %[vlr]\n\t"
      "s_mov_b64 exec, -1\n\t"
      "v_add_u32 %[vx1], 0x1000, %[vx0]\n\t"
      "v_bfe_u32 v139, v138, 4, 1\n\t"
      "v_lshrrev_b32 v140, 5, v138\n\t"
      "v_and_b32 %[vlw], 0xff, %[vlr]\n\t"
      "v_lshl_add_u32 %[vlw], v139, 13, %[vlw]\n\t"
      "v_lshl_add_u32 %[vlw], v140, 8, %[vlw]\n\t"
      "s_lshl_b32 s84, %[swave], 9\n\t"
      "v_add_u32 %[vlw], s84, %[vlw]\n\t"
      "s_lshr_b32 s63, %[sg0], 26\n\t"
      "v_mul_u32_u24 v139, 37, v138\n\t"
      "v_subrev_u32 v140, 37, v139\n\t"
      "v_lshrrev_b32 v140, 8, v140\n\t"
      "v_add3_u32 %[vtab], v138, v140, s63\n\t"
      "v_and_b32 %[vtab], 7, %[vtab]\n\t"
      "v_lshl_add_u32 %[vtab], %[vtab], 2, v140\n\t"
      "v_lshlrev_b32 %[vtab], 24, %[vtab]\n\t"
      "v_subrev_u32 v140, 74, v139\n\t"
      "v_lshrrev_b32 v140, 8, v140\n\t"
      "v_add3_u32 v139, v138, v140, s63\n\t"
      "v_add_u32 v139, 7, v139\n\t"
      "v_and_b32 v139, 7, v139\n\t"
      "v_lshl_add_u32 v138, v139, 2, v140\n\t"
      "v_and_b32 v138, 0xff, v138\n\t"
      "v_lshl_add_u32 %[vtab], v138, 16, %[vtab]\n\t"
      "v_min_u32 v140, 1, v140\n\t"
      "v_lshl_add_u32 v139, v140, 3, v139\n\t"
      "v_lshl_add_u32 %[vtab], v139, 7, %[vtab]\n\t"
      "v_add_u32 %[vtab], 32768, %[vtab]\n\t"
      "s_lshr_b32 s60, %[sg0], 8\n\t"
      "s_or_b32 s84, %[sg0], 50497536\n\t"
      "s_or_b32 s84, s84, s60\n\t"
      "v_writelane_b32 %[vtab], s84, 0\n\t"
      "s_add_u32 s84, %[sg0], 67108864\n\t"
      "s_and_b32 s84, s84, 469762048\n\t"
      "s_or_b32 s84, s84, 231424\n\t"
      "s_or_b32 s84, s84, s60\n\t"
      "v_writelane_b32 %[vtab], s84, 1\n\t"
      "s_add_u32 s84, s63, 7\n\t"
      "s_and_b32 s84, s84, 7\n\t"
      "s_mul_i32 s84, s84, 262272\n\t"
      "s_add_u32 s84, s84, 537101312\n\t"
      "v_writelane_b32 %[vtab], s84, 29\n\t"
      "s_mov_b32 s84, 536905728\n\t"
      "v_writelane_b32 %[vtab], s84, 30\n\t"
      "s_mov_b32 s84, 536907776\n\t"
      "v_writelane_b32 %[vtab], s84, 31\n\t"
      "v_mbcnt_lo_u32_b32 v134, -1, 0\n\t"
      "v_and_b32 v134, 7, v134\n\t"
      "v_lshlrev_b32 v134, 7, v134\n\t"
      "v_mov_b32 v238, 0\n\t"
      "v_mov_b32 v239, 0\n\t"
      "v_mov_b32 v240, 0\n\t"
      "v_mov_b32 v241, 0\n\t"
      "s_nop 1\n\t"
      "v_mfma_f32_32x32x16_f16 a[0:15], v[238:241], v[238:241], 0\n\t"
      "v_mfma_f32_32x32x16_f16 a[16:31], v[238:241], v[238:241], 0\n\t"
      "v_mfma_f32_32x32x16_f16 a[32:47], v[238:241], v[238:241], 0\n\t"
      "v_mfma_f32_32x32x16_f16 a[48:63], v[238:241], v[238:241], 0\n\t"
      "v_mfma_f32_32x32x16_f16 a[64:79], v[238:241], v[238:241], 0\n\t"
      "v_mfma_f32_32x32x16_f16 a[80:95], v[238:241], v[238:241], 0\n\t"
      "v_mfma_f32_32x32x16_f16 a[96:111], v[238:241], v[238:241], 0\n\t"
      "v_mfma_f32_32x32x16_f16 a[112:127], v[238:241], v[238:241], 0\n\t"
      "v_mfma_f32_32x32x16_f16 a[128:143], v[238:241], v[238:241], 0\n\t"
      "v_mfma_f32_32x32x16_f16 a[144:159], v[238:241], v[238:241], 0\n\t"
      "v_mfma_f32_32x32x16_f16 a[160:175], v[238:241], v[238:241], 0\n\t"
      "v_mfma_f32_32x32x16_f16 a[176:191], v[238:241], v[238:241], 0\n\t"
      "v_mfma_f32_32x32x16_f16 a[192:207], v[238:241], v[238:241], 0\n\t"
      "v_mfma_f32_32x32x16_f16 a[208:223], v[238:241], v[238:241], 0\n\t"
      "v_mfma_f32_32x32x16_f16 a[224:239], v[238:241], v[238:241], 0\n\t"
      "v_mfma_f32_32x32x16_f16 a[240:255], v[238:241], v[238:241], 0\n\t"
      "s_waitcnt vmcnt(2) lgkmcnt(0)\n\t"
      "v_readfirstlane_b32 s74, v254\n\t"
      "v_readfirstlane_b32 s72, v255\n\t"
      "v_cvt_pk_f16_f32 v126, v118, v119\n\t"
      "v_cvt_pk_f16_f32 v127, v120, v121\n\t"
      "v_cvt_pk_f16_f32 v128, v122, v123\n\t"
      "v_cvt_pk_f16_f32 v129, v124, v125\n\t"
      "s_add_u32 s74, s74, 1\n\t"
      "s_and_b32 s63, s74, 1\n\t"
      "s_lshl_b32 s63, s63, 23\n\t"
      "s_add_u32 s68, %[sxlo], s63\n\t"
      "s_addc_u32 s69, %[sxhi], 0\n\t"
      "s_cmp_gt_u32 %[swave], 1\n\t"
      "s_cbranch_scc1 Lp2_%=\n\t"
      "s_add_u32 s76, s68, %[sdl]\n\t"
      "s_addc_u32 s77, s69, 0\n\t"
      "global_store_dwordx4 %[vpd], v[126:129], s[76:77] sc0 sc1\n\t"
      "Lp2_%=:\n\t"
      "buffer_load_dwordx4 v[174:177], %[vw0], %[srdw], s61 offen nt\n\t"
      "buffer_load_dwordx4 v[178:181], %[vw1], %[srdw], s61 offen nt\n\t"
      "buffer_load_dwordx4 v[182:185], %[vw2], %[srdw], s61 offen nt\n\t"
      "buffer_load_dwordx4 v[186:189], %[vw3], %[srdw], s61 offen nt\n\t"
      "s_add_u32 s61, s61, 0x80000\n\t"
      "buffer_load_dwordx4 v[190:193], %[vw0], %[srdw], s61 offen nt\n\t"
      "s_add_u32 s76, %[sdlo], s63\n\t"
      "s_addc_u32 s77, %[sdhi], 0\n\t"
      "s_mov_b32 s70, 8388608\n\t"
      "s_mov_b32 s71, 0x20000\n\t"
      "s_lshl_b32 s73, s74, 5\n\t"
      "v_mov_b32 v132, s74\n\t"
      "s_mov_b32 s99, s78\n\t"
      "s_sub_u32 s72, s78, s72\n\t"
      "s_cmp_lt_u32 s72, 20000\n\t"
      "s_cselect_b32 s72, 1, 0\n\t"
      "v_mov_b32 %[vpad], 67108864\n\t"
      "s_bfm_b64 exec, 63, 0\n\t"
      "v_mov_b32 %[vpad], 69599272\n\t"
      "s_bfm_b64 exec, 38, 0\n\t"
      "v_mov_b32 %[vpad], 69730345\n\t"
      "s_bfm_b64 exec, 32, 0\n\t"
      "v_mov_b32 %[vpad], 52953131\n\t"
      "s_bfm_b64 exec, 23, 0\n\t"
      "v_mov_b32 %[vpad], 52690985\n\t"
      "s_bfm_b64 exec, 22, 0\n\t"
      "v_mov_b32 %[vpad], 69468197\n\t"
      "s_bfm_b64 exec, 17, 0\n\t"
      "v_mov_b32 %[vpad], 69471269\n\t"
      "s_bfm_b64 exec, 16, 0\n\t"
      "v_mov_b32 %[vpad], 69733416\n\t"
      "s_bfm_b64 exec, 15, 0\n\t"
      "v_mov_b32 %[vpad], 69930028\n\t"
      "s_bfm_b64 exec, 14, 0\n\t"
      "v_mov_b32 %[vpad], 19729457\n\t"
      "s_bfm_b64 exec, 13, 0\n\t"
      "v_mov_b32 %[vpad], 19860532\n\t"
      "s_bfm_b64 exec, 12, 0\n\t"
      "v_mov_b32 %[vpad], 19926071\n\t"
      "s_bfm_b64 exec, 11, 0\n\t"
      "v_mov_b32 %[vpad], 19991608\n\t"
      "s_bfm_b64 exec, 8, 0\n\t"
      "v_mov_b32 %[vpad], 19926072\n\t"
      "s_bfm_b64 exec, 7, 0\n\t"
      "v_mov_b32 %[vpad], 19926071\n\t"
      "s_bfm_b64 exec, 6, 0\n\t"
      "v_mov_b32 %[vpad], 19792180\n\t"
      "s_bfm_b64 exec, 5, 0\n\t"
      "v_mov_b32 %[vpad], 70058289\n\t"
      "s_bfm_b64 exec, 4, 0\n\t"
      "v_mov_b32 %[vpad], 69861677\n\t"
      "s_bfm_b64 exec, 3, 0\n\t"
      "v_mov_b32 %[vpad], 69730602\n\t"
      "s_bfm_b64 exec, 2, 0\n\t"
      "v_mov_b32 %[vpad], 69730601\n\t"
      "s_bfm_b64 exec, 1, 0\n\t"
      "v_mov_b32 %[vpad], 41\n\t"
      "s_mov_b64 exec, -1\n\t"
      "v_mov_b32 v139, 0x20000\n\t"
      "s_cmp_lg_u32 %[swave], 0\n\t"
      "s_cbranch_scc1 Lpm_%=\n\t"
      "v_mov_b32 v133, s72\n\t"
      "s_mov_b64 exec, 1\n\t"
      "ds_write_b32 v139, v133\n\t"
      "s_mov_b64 exec, -1\n\t"
      "Lpm_%=:\n\t"
      "s_waitcnt vmcnt(5) lgkmcnt(0)\n\t"
      "s_barrier\n\t"
      "s_mov_b64 exec, 1\n\t"
      "ds_read_b32 v133, v139\n\t"
      "s_mov_b64 exec, -1\n\t"
      "s_cmp_lg_u32 %[swave], 0\n\t"
      "s_cbranch_scc1 Lp3_%=\n\t"
      "s_mov_b64 exec, 1\n\t"
      "global_atomic_add v130, v131, %[sws] sc1\n\t"
      "buffer_store_dword v132, off, %[srdws], %[sseq] sc1\n\t"
      "s_cmp_lg_u32 %[sseq], 0\n\t"
      "s_cbranch_scc1 Lpz_%=\n\t"
      "s_andn2_b32 s84, 1, s74\n\t"
      "s_lshl_b32 s84, s84, 7\n\t"
      "s_add_u32 s84, s84, 37120\n\t"
      "buffer_store_dword v238, off, %[srdws], s84 sc1\n\t"
      "buffer_store_dword v238, off, %[srdws], s84 offset:256 sc1\n\t"
      "Lpz_%=:\n\t"
      "s_mov_b64 exec, -1\n\t"
      "Lp3_%=:\n\t"
      "v_cvt_pk_f16_f32 v110, v78, v79\n\t"
      "v_cvt_pk_f16_f32 v111, v80, v81\n\t"
      "v_cvt_pk_f16_f32 v112, v82, v83\n\t"
      "v_cvt_pk_f16_f32 v113, v84, v85\n\t"
      "s_bitcmp1_b32 %[sseq], 10\n\t"
      "s_cbranch_scc1 Lph_%=\n\t"
      "s_cmp_eq_u32 %[snself], 0\n\t"
      "s_cbranch_scc1 Lp4a_%=\n\t"
      "global_store_dwordx4 %[vpd], v[110:113], s[76:77] sc0 sc1\n\t"
      "s_add_u32 s76, s76, 262144\n\t"
      "s_addc_u32 s77, s77, 0\n\t"
      "Lp4a_%=:\n\t"
      "Lph_%=:\n\t"
      "buffer_load_dwordx4 v[194:197], %[vw1], %[srdw], s61 offen nt\n\t"
      "buffer_load_dwordx4 v[198:201], %[vw2], %[srdw], s61 offen nt\n\t"
      "buffer_load_dwordx4 v[202:205], %[vw3], %[srdw], s61 offen nt\n\t"
      "s_add_u32 s61, s61, 7864320\n\t"
      "buffer_load_dwordx4 v[142:145], %[vw0], %[srdw], s61 offen nt\n\t"
      "buffer_load_dwordx4 v[146:149], %[vw1], %[srdw], s61 offen nt\n\t"
      "buffer_load_dwordx4 v[150:153], %[vw2], %[srdw], s61 offen nt\n\t"
      "buffer_load_dwordx4 v[154:157], %[vw3], %[srdw], s61 offen nt\n\t"
      "s_add_u32 s61, s61, 0x80000\n\t"
      "buffer_load_dwordx4 v[158:161], %[vw0], %[srdw], s61 offen nt\n\t"
      "buffer_load_dwordx4 v[162:165], %[vw1], %[srdw], s61 offen nt\n\t"
      "buffer_load_dwordx4 v[166:169], %[vw2], %[srdw], s61 offen nt\n\t"
      "buffer_load_dwordx4 v[170:173], %[vw3], %[srdw], s61 offen nt\n\t"
      "s_add_u32 s61, s61, 7864320\n\t"
      "s_waitcnt vmcnt(8)\n\t"
      "v_cvt_pk_f16_f32 v238, v174, v178\n\t"
      "v_cvt_pk_f16_f32 v239, v182, v186\n\t"
      "v_cvt_pk_f16_f32 v240, v190, v194\n\t"
      "v_cvt_pk_f16_f32 v241, v198, v202\n\t"
      "ds_write_b128 %[vlw], v[238:241] offset:0\n\t"
      "s_nop 1\n\t"
      "v_cvt_pk_f16_f32 v238, v175, v179\n\t"
      "v_cvt_pk_f16_f32 v239, v183, v187\n\t"
      "v_cvt_pk_f16_f32 v240, v191, v195\n\t"
      "v_cvt_pk_f16_f32 v241, v199, v203\n\t"
      "ds_write_b128 %[vlw], v[238:241] offset:2048\n\t"
      "s_nop 1\n\t"
      "v_cvt_pk_f16_f32 v238, v176, v180\n\t"
      "v_cvt_pk_f16_f32 v239, v184, v188\n\t"
      "v_cvt_pk_f16_f32 v240, v192, v196\n\t"
      "v_cvt_pk_f16_f32 v241, v200, v204\n\t"
      "ds_write_b128 %[vlw], v[238:241] offset:4096\n\t"
      "s_nop 1\n\t"
      "v_cvt_pk_f16_f32 v238, v177, v181\n\t"
      "v_cvt_pk_f16_f32 v239, v185, v189\n\t"
      "v_cvt_pk_f16_f32 v240, v193, v197\n\t"
      "v_cvt_pk_f16_f32 v241, v201, v205\n\t"
      "ds_write_b128 %[vlw], v[238:241] offset:6144\n\t"
      "s_nop 1\n\t"
      "s_waitcnt vmcnt(8)\n\t"
      "s_mov_b32 s67, 34816\n\t"
      "s_cmp_lg_u32 %[swave], 0\n\t"
      "s_cbranch_scc1 Lpolled_%=\n\t"
      "Lq0_%=:\n\t"
      "s_memrealtime s[80:81]\n\t"
      "s_waitcnt lgkmcnt(0)\n\t"
      "s_sub_u32 s81, s80, s78\n\t"
      "s_cmp_lt_u32 s81, 275\n\t"
      "s_cbranch_scc1 Lq0_%=\n\t"
      "Lq_%=:\n\t"
      "buffer_load_dword v254, v134, %[srdws], s67 offen sc0 sc1\n\t"
      "s_memrealtime s[80:81]\n\t"
      "s_waitcnt vmcnt(0) lgkmcnt(0)\n\t"
      "v_cmp_gt_u32 vcc, s73, v254\n\t"
      "s_nop 4\n\t"
      "s_cbranch_vccz Lpolled_%=\n\t"
      "s_sub_u32 s81, s80, s78\n\t"
      "s_cmp_gt_u32 s81, 50000000\n\t"
      "s_cbranch_scc1 Lqbad_%=\n\t"
      "s_sleep 10\n\t"
      "s_branch Lq_%=\n\t"
      "Lqbad_%=:\n\t"
      "s_mov_b32 s84, 36864\n\t"
      "s_call_b64 s[82:83], Lpbad_%=\n\t"
      "Lpolled_%=:\n\t"
      "s_waitcnt lgkmcnt(0)\n\t"
      "s_barrier\n\t"
      "s_memrealtime s[94:95]\n\t"
      "v_readfirstlane_b32 s72, v133\n\t"
      "buffer_load_dword v254, off, %[srdws], s67 sc0 sc1\n\t"
      "s_lshr_b32 s60, %[sg0], 6\n\t"
      "buffer_load_dwordx4 v[14:17], %[vx0], s[68:71], s60 offen offset:0 sc0 sc1\n\t"
      "buffer_load_dwordx4 v[18:21], %[vx0], s[68:71], s60 offen offset:1024 sc0 sc1\n\t"
      "buffer_load_dwordx4 v[22:25], %[vx0], s[68:71], s60 offen offset:2048 sc0 sc1\n\t"
      "buffer_load_dwordx4 v[26:29], %[vx0], s[68:71], s60 offen offset:3072 sc0 sc1\n\t"
      "buffer_load_dwordx4 v[30:33], %[vx1], s[68:71], s60 offen offset:0 sc0 sc1\n\t"
      "buffer_load_dwordx4 v[34:37], %[vx1], s[68:71], s60 offen offset:1024 sc0 sc1\n\t"
      "buffer_load_dwordx4 v[38:41], %[vx1], s[68:71], s60 offen offset:2048 sc0 sc1\n\t"
      "buffer_load_dwordx4 v[42:45], %[vx1], s[68:71], s60 offen offset:3072 sc0 sc1\n\t"
      "s_add_u32 s60, s60, 0x2000\n\t"
      "buffer_load_dwordx4 v[46:49], %[vx0], s[68:71], s60 offen offset:0 sc0 sc1\n\t"
      "buffer_load_dwordx4 v[50:53], %[vx0], s[68:71], s60 offen offset:1024 sc0 sc1\n\t"
      "buffer_load_dwordx4 v[54:57], %[vx0], s[68:71], s60 offen offset:2048 sc0 sc1\n\t"
      "buffer_load_dwordx4 v[58:61], %[vx0], s[68:71], s60 offen offset:3072 sc0 sc1\n\t"
      "buffer_load_dwordx4 v[62:65], %[vx1], s[68:71], s60 offen offset:0 sc0 sc1\n\t"
      "buffer_load_dwordx4 v[66:69], %[vx1], s[68:71], s60 offen offset:1024 sc0 sc1\n\t"
      "buffer_load_dwordx4 v[70:73], %[vx1], s[68:71], s60 offen offset:2048 sc0 sc1\n\t"
      "buffer_load_dwordx4 v[74:77], %[vx1], s[68:71], s60 offen offset:3072 sc0 sc1\n\t"
      "s_add_u32 s60, s60, 122880\n\t"
      "ds_read_b128 v[206:209], %[vlr] offset:0\n\t"
      "ds_read_b128 v[210:213], %[vlr] offset:1024\n\t"
      "ds_read_b128 v[214:217], %[vlr] offset:2048\n\t"
      "ds_read_b128 v[218:221], %[vlr] offset:3072\n\t"
      "s_add_u32 s61, %[sg0], 0x1000000\n\t"
      "buffer_load_dwordx4 v[174:177], %[vw0], %[srdw], s61 offen nt\n\t"
      "buffer_load_dwordx4 v[178:181], %[vw1], %[srdw], s61 offen nt\n\t"
      "buffer_load_dwordx4 v[182:185], %[vw2], %[srdw], s61 offen nt\n\t"
      "buffer_load_dwordx4 v[186:189], %[vw3], %[srdw], s61 offen nt\n\t"
      "s_add_u32 s61, s61, 0x80000\n\t"
      "buffer_load_dwordx4 v[190:193], %[vw0], %[srdw], s61 offen nt\n\t"
      "buffer_load_dwordx4 v[194:197], %[vw1], %[srdw], s61 offen nt\n\t"
      "buffer_load_dwordx4 v[198:201], %[vw2], %[srdw], s61 offen nt\n\t"
      "buffer_load_dwordx4 v[202:205], %[vw3], %[srdw], s61 offen nt\n\t"
      "s_add_u32 s61, s61, 7864320\n\t"
      "s_add_u32 s64, %[sg0], 0x1000000\n\t"
      "s_add_u32 s65, %[sg0], 0x2000000\n\t"
      "s_bfe_u32 s85, %[spk], 0x80010\n\t"
      "s_mov_b32 s89, 0\n\t"
      "s_bfe_u32 s88, %[spk], 0x80018\n\t"
      "s_cmp_eq_u32 s85, 29\n\t"
      "s_cselect_b32 s63, 0, 0\n\t"
      "s_cmp_eq_u32 s72, 0\n\t"
      "s_cselect_b32 s63, s63, 0\n\t"
      "s_sub_u32 s85, s85, s63\n\t"
      "s_lshl_b32 s66, s72, 3\n\t"
      "s_lshr_b32 s66, %[spk], s66\n\t"
      "s_and_b32 s66, s66, 0xff\n\t"
      "s_waitcnt vmcnt(8) lgkmcnt(0)\n\t"
      "s_sub_u32 s94, s94, s99\n\t"
      "s_min_u32 s94, s94, 340\n\t"
      "s_add_u32 s94, s94, s99\n\t"
      "s_add_u32 s93, %[sstag], s94\n\t"
      "s_sub_u32 s96, s94, 256\n\t"
      "v_readlane_b32 s92, %[vpad], 0\n\t"
      "s_mov_b32 s57, 0\n\t"
      "s_mov_b32 s50, 0\n\t"
      "s_mov_b32 s86, 0\n\t"
      "s_mov_b32 s98, 0\n\t"
      "s_mov_b32 s87, 0\n\t"
      "s_mov_b32 s52, 0\n\t"
      "s_mov_b32 s55, 0\n\t"
      "s_mov_b32 s59, -1\n\t"
      "s_bitcmp0_b32 %[sseq], 10\n\t"
      "s_cbranch_scc1 Lph2_%=\n\t"
      "s_cmp_eq_u32 %[snself], 0\n\t"
      "s_cbranch_scc1 Lp4b_%=\n\t"
      "global_store_dwordx4 %[vpd], v[110:113], s[76:77] sc0 sc1\n\t"
      "s_add_u32 s76, s76, 262144\n\t"
      "s_addc_u32 s77, s77, 0\n\t"
      "Lp4b_%=:\n\t"
      "Lph2_%=:\n\t"
      "s_sub_u32 s66, s66, 1\n\t"
      "s_mov_b32 s91, 0\n\t"
      "s_cmp_lg_u32 s72, 0\n\t"
      "s_cbranch_scc1 Lloop2H_%=\n\t"
      "s_branch Lloop2C_%=\n\t"
      ".p2align 6\n\t"
      "LloopC_%=:\n\t"
      "s_waitcnt vmcnt(7) lgkmcnt(0)\n\t"
      "Lloop2C_%=:\n\t"
      "v_mfma_f32_16x16x32_f16 a[0:3], v[14:17], v[206:209], a[0:3]\n\t"
      "v_cvt_pk_f16_f32 v238, v142, v146\n\t"
      "v_mfma_f32_16x16x32_f16 a[0:3], v[18:21], v[210:213], a[0:3]\n\t"
      "buffer_load_dwordx4 v[78:81], %[vx0], s[68:71], s60 offen offset:0 sc0 sc1\n\t"
      "v_mfma_f32_16x16x32_f16 a[4:7], v[18:21], v[218:221], a[4:7]\n\t"
      "ds_read_b128 v[222:225], %[vlr] offset:4096\n\t"
      "v_mfma_f32_16x16x32_f16 a[4:7], v[14:17], v[214:217], a[4:7]\n\t"
      "v_cvt_pk_f16_f32 v239, v150, v154\n\t"
      "v_mfma_f32_16x16x32_f16 a[36:39], v[22:25], v[214:217], a[36:39]\n\t"
      "ds_read_b128 v[226:229], %[vlr] offset:5120\n\t"
      "v_mfma_f32_16x16x32_f16 a[36:39], v[26:29], v[218:221], a[36:39]\n\t"
      "buffer_load_dwordx4 v[82:85], %[vx0], s[68:71], s60 offen offset:1024 sc0 sc1\n\t"
      "v_mfma_f32_16x16x32_f16 a[32:35], v[26:29], v[210:213], a[32:35]\n\t"
      "v_cvt_pk_f16_f32 v240, v158, v162\n\t"
      "v_mfma_f32_16x16x32_f16 a[32:35], v[22:25], v[206:209], a[32:35]\n\t"
      "ds_read_b128 v[230:233], %[vlr] offset:6144\n\t"
      "v_mfma_f32_16x16x32_f16 a[64:67], v[30:33], v[206:209], a[64:67]\n\t"
      "v_cvt_pk_f16_f32 v241, v166, v170\n\t"
      "v_mfma_f32_16x16x32_f16 a[64:67], v[34:37], v[210:213], a[64:67]\n\t"
      "buffer_load_dwordx4 v[86:89], %[vx0], s[68:71], s60 offen offset:2048 sc0 sc1\n\t"
      "v_mfma_f32_16x16x32_f16 a[68:71], v[34:37], v[218:221], a[68:71]\n\t"
      "ds_read_b128 v[234:237], %[vlr] offset:7168\n\t"
      "v_mfma_f32_16x16x32_f16 a[68:71], v[30:33], v[214:217], a[68:71]\n\t"
      "ds_write_b128 %[vlw], v[238:241] offset:16384\n\t"
      "v_mfma_f32_16x16x32_f16 a[100:103], v[38:41], v[214:217], a[100:103]\n\t"
      "v_cvt_pk_f16_f32 v238, v143, v147\n\t"
      "v_mfma_f32_16x16x32_f16 a[100:103], v[42:45], v[218:221], a[100:103]\n\t"
      "buffer_load_dwordx4 v[90:93], %[vx0], s[68:71], s60 offen offset:3072 sc0 sc1\n\t"
      "v_mfma_f32_16x16x32_f16 a[96:99], v[42:45], v[210:213], a[96:99]\n\t"
      "v_cvt_pk_f16_f32 v239, v151, v155\n\t"
      "v_mfma_f32_16x16x32_f16 a[96:99], v[38:41], v[206:209], a[96:99]\n\t"
      "v_cvt_pk_f16_f32 v240, v159, v163\n\t"
      "v_mfma_f32_16x16x32_f16 a[128:131], v[46:49], v[206:209], a[128:131]\n\t"
      "v_cvt_pk_f16_f32 v241, v167, v171\n\t"
      "v_mfma_f32_16x16x32_f16 a[128:131], v[50:53], v[210:213], a[128:131]\n\t"
      "buffer_load_dwordx4 v[94:97], %[vx1], s[68:71], s60 offen offset:0 sc0 sc1\n\t"
      "v_mfma_f32_16x16x32_f16 a[132:135], v[50:53], v[218:221], a[132:135]\n\t"
      "ds_write_b128 %[vlw], v[238:241] offset:18432\n\t"
      "v_mfma_f32_16x16x32_f16 a[132:135], v[46:49], v[214:217], a[132:135]\n\t"
      "v_cvt_pk_f16_f32 v238, v144, v148\n\t"
      "v_mfma_f32_16x16x32_f16 a[164:167], v[54:57], v[214:217], a[164:167]\n\t"
      "v_cvt_pk_f16_f32 v239, v152, v156\n\t"
      "v_mfma_f32_16x16x32_f16 a[164:167], v[58:61], v[218:221], a[164:167]\n\t"
      "v_cvt_pk_f16_f32 v240, v160, v164\n\t"
      "v_mfma_f32_16x16x32_f16 a[160:163], v[58:61], v[210:213], a[160:163]\n\t"
      "buffer_load_dwordx4 v[98:101], %[vx1], s[68:71], s60 offen offset:1024 sc0 sc1\n\t"
      "v_mfma_f32_16x16x32_f16 a[160:163], v[54:57], v[206:209], a[160:163]\n\t"
      "v_cvt_pk_f16_f32 v241, v168, v172\n\t"
      "v_mfma_f32_16x16x32_f16 a[192:195], v[62:65], v[206:209], a[192:195]\n\t"
      "ds_write_b128 %[vlw], v[238:241] offset:20480\n\t"
      "v_mfma_f32_16x16x32_f16 a[192:195], v[66:69], v[210:213], a[192:195]\n\t"
      "v_cvt_pk_f16_f32 v238, v145, v149\n\t"
      "v_mfma_f32_16x16x32_f16 a[196:199], v[66:69], v[218:221], a[196:199]\n\t"
      "buffer_load_dwordx4 v[102:105], %[vx1], s[68:71], s60 offen offset:2048 sc0 sc1\n\t"
      "v_mfma_f32_16x16x32_f16 a[196:199], v[62:65], v[214:217], a[196:199]\n\t"
      "v_cvt_pk_f16_f32 v239, v153, v157\n\t"
      "v_mfma_f32_16x16x32_f16 a[228:231], v[70:73], v[214:217], a[228:231]\n\t"
      "v_cvt_pk_f16_f32 v240, v161, v165\n\t"
      "v_mfma_f32_16x16x32_f16 a[228:231], v[74:77], v[218:221], a[228:231]\n\t"
      "v_cvt_pk_f16_f32 v241, v169, v173\n\t"
      "v_mfma_f32_16x16x32_f16 a[224:227], v[74:77], v[210:213], a[224:227]\n\t"
      "buffer_load_dwordx4 v[106:109], %[vx1], s[68:71], s60 offen offset:3072 sc0 sc1\n\t"
      "v_mfma_f32_16x16x32_f16 a[224:227], v[70:73], v[206:209], a[224:227]\n\t"
      "s_add_u32 s60, s60, 0x2000\n\t"
      "s_waitcnt lgkmcnt(3)\n\t"
      "v_mfma_f32_16x16x32_f16 a[8:11], v[14:17], v[222:225], a[8:11]\n\t"
      "ds_write_b128 %[vlw], v[238:241] offset:22528\n\t"
      "v_mfma_f32_16x16x32_f16 a[8:11], v[18:21], v[226:229], a[8:11]\n\t"
      "ds_read_b128 v[206:209], %[vlr] offset:8192\n\t"
      "v_mfma_f32_16x16x32_f16 a[12:15], v[18:21], v[234:237], a[12:15]\n\t"
      "buffer_load_dwordx4 v[110:113], %[vx0], s[68:71], s60 offen offset:0 sc0 sc1\n\t"
      "v_mfma_f32_16x16x32_f16 a[12:15], v[14:17], v[230:233], a[12:15]\n\t"
      "v_mfma_f32_16x16x32_f16 a[44:47], v[22:25], v[230:233], a[44:47]\n\t"
      "ds_read_b128 v[210:213], %[vlr] offset:9216\n\t"
      "v_mfma_f32_16x16x32_f16 a[44:47], v[26:29], v[234:237], a[44:47]\n\t"
      "s_cmp_lt_u32 s98, s52\n\t"
      "s_cselect_b32 s59, 55, -1\n\t"
      "v_mfma_f32_16x16x32_f16 a[40:43], v[26:29], v[226:229], a[40:43]\n\t"
      "buffer_load_dwordx4 v[114:117], %[vx0], s[68:71], s60 offen offset:1024 sc0 sc1\n\t"
      "v_mfma_f32_16x16x32_f16 a[40:43], v[22:25], v[222:225], a[40:43]\n\t"
      "ds_read_b128 v[214:217], %[vlr] offset:10240\n\t"
      "v_mfma_f32_16x16x32_f16 a[72:75], v[30:33], v[222:225], a[72:75]\n\t"
      "v_mfma_f32_16x16x32_f16 a[72:75], v[34:37], v[226:229], a[72:75]\n\t"
      "v_mfma_f32_16x16x32_f16 a[76:79], v[34:37], v[234:237], a[76:79]\n\t"
      "ds_read_b128 v[218:221], %[vlr] offset:11264\n\t"
      "v_mfma_f32_16x16x32_f16 a[76:79], v[30:33], v[230:233], a[76:79]\n\t"
      "buffer_load_dwordx4 v[118:121], %[vx0], s[68:71], s60 offen offset:2048 sc0 sc1\n\t"
      "v_mfma_f32_16x16x32_f16 a[108:111], v[38:41], v[230:233], a[108:111]\n\t"
      "s_memrealtime s[94:95]\n\t"
      "v_mfma_f32_16x16x32_f16 a[108:111], v[42:45], v[234:237], a[108:111]\n\t"
      "v_mfma_f32_16x16x32_f16 a[104:107], v[42:45], v[226:229], a[104:107]\n\t"
      "v_mfma_f32_16x16x32_f16 a[104:107], v[38:41], v[222:225], a[104:107]\n\t"
      "s_cmp_eq_u32 s62, s88\n\t"
      "s_cbranch_scc1 LextC_%=\n\t"
      "LextrC_%=:\n\t"
      "v_mfma_f32_16x16x32_f16 a[136:139], v[46:49], v[222:225], a[136:139]\n\t"
      "s_cmp_eq_u32 s62, s66\n\t"
      "s_cbranch_scc1 Lpush0C_%=\n\t"
      "Lpushr0C_%=:\n\t"
      "v_mfma_f32_16x16x32_f16 a[136:139], v[50:53], v[226:229], a[136:139]\n\t"
      "buffer_load_dwordx4 v[142:145], %[vw0], %[srdw], s61 offen nt\n\t"
      "v_mfma_f32_16x16x32_f16 a[140:143], v[50:53], v[234:237], a[140:143]\n\t"
      "s_lshl1_add_u32 s93, s92, s93\n\t"
      "v_mfma_f32_16x16x32_f16 a[140:143], v[46:49], v[230:233], a[140:143]\n\t"
      "v_mfma_f32_16x16x32_f16 a[172:175], v[54:57], v[230:233], a[172:175]\n\t"
      "v_mfma_f32_16x16x32_f16 a[172:175], v[58:61], v[234:237], a[172:175]\n\t"
      "buffer_load_dwordx4 v[146:149], %[vw1], %[srdw], s61 offen nt\n\t"
      "v_mfma_f32_16x16x32_f16 a[168:171], v[58:61], v[226:229], a[168:171]\n\t"
      "v_mfma_f32_16x16x32_f16 a[168:171], v[54:57], v[222:225], a[168:171]\n\t"
      "s_waitcnt lgkmcnt(0)\n\t"
      "v_mfma_f32_16x16x32_f16 a[200:203], v[62:65], v[222:225], a[200:203]\n\t"
      "s_sub_u32 s95, s94, s93\n\t"
      "v_mfma_f32_16x16x32_f16 a[200:203], v[66:69], v[226:229], a[200:203]\n\t"
      "buffer_load_dwordx4 v[150:153], %[vw2], %[srdw], s61 offen nt\n\t"
      "v_mfma_f32_16x16x32_f16 a[204:207], v[66:69], v[234:237], a[204:207]\n\t"
      "s_cmp_lt_i32 s95, 0\n\t"
      "s_cbranch_scc1 Lzl0C_%=\n\t"
      "Lzzr0C_%=:\n\t"
      "v_mfma_f32_16x16x32_f16 a[204:207], v[62:65], v[230:233], a[204:207]\n\t"
      "s_sub_u32 s95, s95, 30\n\t"
      "s_max_i32 s95, s95, 0\n\t"
      "v_mfma_f32_16x16x32_f16 a[236:239], v[70:73], v[230:233], a[236:239]\n\t"
      "s_mul_i32 s86, s86, 3\n\t"
      "s_lshr_b32 s86, s86, 2\n\t"
      "v_mfma_f32_16x16x32_f16 a[236:239], v[74:77], v[234:237], a[236:239]\n\t"
      "buffer_load_dwordx4 v[154:157], %[vw3], %[srdw], s61 offen nt\n\t"
      "v_mfma_f32_16x16x32_f16 a[232:235], v[74:77], v[226:229], a[232:235]\n\t"
      "s_add_u32 s61, s61, 0x80000\n\t"
      "v_mfma_f32_16x16x32_f16 a[232:235], v[70:73], v[222:225], a[232:235]\n\t"
      "s_cmp_ge_u32 s95, s59\n\t"
      "s_cselect_b32 s86, s55, s86\n\t"
      "s_waitcnt lgkmcnt(0)\n\t"
      "v_mfma_f32_16x16x32_f16 a[16:19], v[14:17], v[206:209], a[16:19]\n\t"
      "s_add_u32 s95, s95, s86\n\t"
      "s_add_u32 s93, s93, s95\n\t"
      "v_mfma_f32_16x16x32_f16 a[16:19], v[18:21], v[210:213], a[16:19]\n\t"
      "ds_read_b128 v[222:225], %[vlr] offset:12288\n\t"
      "s_add_u32 s98, s98, s95\n\t"
      "v_mfma_f32_16x16x32_f16 a[20:23], v[18:21], v[218:221], a[20:23]\n\t"
      "buffer_load_dwordx4 v[158:161], %[vw0], %[srdw], s61 offen nt\n\t"
      "v_mfma_f32_16x16x32_f16 a[20:23], v[14:17], v[214:217], a[20:23]\n\t"
      "s_sub_u32 s97, s93, s96\n\t"
      "s_lshr_b32 s97, s97, 8\n\t"
      "v_mfma_f32_16x16x32_f16 a[52:55], v[22:25], v[214:217], a[52:55]\n\t"
      "ds_read_b128 v[226:229], %[vlr] offset:13312\n\t"
      "s_min_u32 s97, s97, 63\n\t"
      "v_mfma_f32_16x16x32_f16 a[52:55], v[26:29], v[218:221], a[52:55]\n\t"
      "s_mov_b32 s54, s92\n\t"
      "v_readlane_b32 s90, %[vpad], s97\n\t"
      "v_mfma_f32_16x16x32_f16 a[48:51], v[26:29], v[210:213], a[48:51]\n\t"
      "s_min_u32 s98, s98, 600\n\t"
      "s_lshr_b32 s94, s98, 2\n\t"
      "v_mfma_f32_16x16x32_f16 a[48:51], v[22:25], v[206:209], a[48:51]\n\t"
      "ds_read_b128 v[230:233], %[vlr] offset:14336\n\t"
      "s_and_b32 s92, s90, 0xff\n\t"
      "v_mfma_f32_16x16x32_f16 a[80:83], v[30:33], v[206:209], a[80:83]\n\t"
      "s_lshr_b32 s97, s90, 24\n\t"
      "s_min_u32 s97, s97, s94\n\t"
      "v_mfma_f32_16x16x32_f16 a[80:83], v[34:37], v[210:213], a[80:83]\n\t"
      "s_cmp_gt_u32 s86, 3\n\t"
      "s_cselect_b32 s97, 0, s97\n\t"
      "v_mfma_f32_16x16x32_f16 a[84:87], v[34:37], v[218:221], a[84:87]\n\t"
      "ds_read_b128 v[234:237], %[vlr] offset:15360\n\t"
      "v_mfma_f32_16x16x32_f16 a[84:87], v[30:33], v[214:217], a[84:87]\n\t"
      "s_bfe_u32 s90, s90, 0x80008\n\t"
      "s_cbranch_scc1 Lbk0C_%=\n\t"
      "Lbkr0C_%=:\n\t"
      "v_mfma_f32_16x16x32_f16 a[116:119], v[38:41], v[214:217], a[116:119]\n\t"
      "s_sub_u32 s92, s92, s97\n\t"
      "s_lshl_b32 s97, s97, 2\n\t"
      "v_mfma_f32_16x16x32_f16 a[116:119], v[42:45], v[218:221], a[116:119]\n\t"
      "s_sub_u32 s98, s98, s97\n\t"
      "s_mov_b32 s87, s98\n\t"
      "v_mfma_f32_16x16x32_f16 a[112:115], v[42:45], v[210:213], a[112:115]\n\t"
      "v_mfma_f32_16x16x32_f16 a[112:115], v[38:41], v[206:209], a[112:115]\n\t"
      "v_mfma_f32_16x16x32_f16 a[144:147], v[46:49], v[206:209], a[144:147]\n\t"
      "v_mfma_f32_16x16x32_f16 a[144:147], v[50:53], v[210:213], a[144:147]\n\t"
      "v_mfma_f32_16x16x32_f16 a[148:151], v[50:53], v[218:221], a[148:151]\n\t"
      "v_mfma_f32_16x16x32_f16 a[148:151], v[46:49], v[214:217], a[148:151]\n\t"
      "v_mfma_f32_16x16x32_f16 a[180:183], v[54:57], v[214:217], a[180:183]\n\t"
      "v_add_u32 %[vlw], 0x8000, %[vlw]\n\t"
      "v_mfma_f32_16x16x32_f16 a[180:183], v[58:61], v[218:221], a[180:183]\n\t"
      "v_and_b32 %[vlw], 0x1ffff, %[vlw]\n\t"
      "v_mfma_f32_16x16x32_f16 a[176:179], v[58:61], v[210:213], a[176:179]\n\t"
      "buffer_load_dwordx4 v[122:125], %[vx0], s[68:71], s60 offen offset:3072 sc0 sc1\n\t"
      "v_mfma_f32_16x16x32_f16 a[176:179], v[54:57], v[206:209], a[176:179]\n\t"
      "v_mfma_f32_16x16x32_f16 a[208:211], v[62:65], v[206:209], a[208:211]\n\t"
      "v_mfma_f32_16x16x32_f16 a[208:211], v[66:69], v[210:213], a[208:211]\n\t"
      "v_mfma_f32_16x16x32_f16 a[212:215], v[66:69], v[218:221], a[212:215]\n\t"
      "buffer_load_dwordx4 v[126:129], %[vx1], s[68:71], s60 offen offset:0 sc0 sc1\n\t"
      "v_mfma_f32_16x16x32_f16 a[212:215], v[62:65], v[214:217], a[212:215]\n\t"
      "v_mfma_f32_16x16x32_f16 a[244:247], v[70:73], v[214:217], a[244:247]\n\t"
      "v_mfma_f32_16x16x32_f16 a[244:247], v[74:77], v[218:221], a[244:247]\n\t"
      "v_mfma_f32_16x16x32_f16 a[240:243], v[74:77], v[210:213], a[240:243]\n\t"
      "buffer_load_dwordx4 v[130:133], %[vx1], s[68:71], s60 offen offset:1024 sc0 sc1\n\t"
      "v_mfma_f32_16x16x32_f16 a[240:243], v[70:73], v[206:209], a[240:243]\n\t"
      "s_waitcnt vmcnt(19) lgkmcnt(0)\n\t"
      "s_barrier\n\t"
      "v_mfma_f32_16x16x32_f16 a[24:27], v[14:17], v[222:225], a[24:27]\n\t"
      "v_readfirstlane_b32 s63, v254\n\t"
      "v_mfma_f32_16x16x32_f16 a[24:27], v[18:21], v[226:229], a[24:27]\n\t"
      "s_cmp_lt_u32 s63, s73\n\t"
      "s_cbranch_scc1 LslowC_%=\n\t"
      "LslowrC_%=:\n\t"
      "v_mfma_f32_16x16x32_f16 a[28:31], v[18:21], v[234:237], a[28:31]\n\t"
      "buffer_load_dwordx4 v[134:137], %[vx1], s[68:71], s60 offen offset:2048 sc0 sc1\n\t"
      "v_mfma_f32_16x16x32_f16 a[28:31], v[14:17], v[230:233], a[28:31]\n\t"
      "ds_read_b128 v[206:209], %[vlr] offset:16384\n\t"
      "v_mfma_f32_16x16x32_f16 a[60:63], v[22:25], v[230:233], a[60:63]\n\t"
      "ds_read_b128 v[210:213], %[vlr] offset:17408\n\t"
      "v_mfma_f32_16x16x32_f16 a[60:63], v[26:29], v[234:237], a[60:63]\n\t"
      "ds_read_b128 v[214:217], %[vlr] offset:18432\n\t"
      "v_mfma_f32_16x16x32_f16 a[56:59], v[26:29], v[226:229], a[56:59]\n\t"
      "buffer_load_dwordx4 v[138:141], %[vx1], s[68:71], s60 offen offset:3072 sc0 sc1\n\t"
      "v_mfma_f32_16x16x32_f16 a[56:59], v[22:25], v[222:225], a[56:59]\n\t"
      "s_lshr_b32 s60, s64, 6\n\t"
      "v_mfma_f32_16x16x32_f16 a[88:91], v[30:33], v[222:225], a[88:91]\n\t"
      "ds_read_b128 v[218:221], %[vlr] offset:19456\n\t"
      "v_mfma_f32_16x16x32_f16 a[88:91], v[34:37], v[226:229], a[88:91]\n\t"
      "v_cvt_pk_f16_f32 v238, v174, v178\n\t"
      "v_mfma_f32_16x16x32_f16 a[92:95], v[34:37], v[234:237], a[92:95]\n\t"
      "v_cvt_pk_f16_f32 v239, v182, v186\n\t"
      "v_mfma_f32_16x16x32_f16 a[92:95], v[30:33], v[230:233], a[92:95]\n\t"
      "buffer_load_dwordx4 v[14:17], %[vx0], s[68:71], s60 offen offset:0 sc0 sc1\n\t"
      "v_mfma_f32_16x16x32_f16 a[124:127], v[38:41], v[230:233], a[124:127]\n\t"
      "v_cvt_pk_f16_f32 v240, v190, v194\n\t"
      "s_memrealtime s[94:95]\n\t"
      "v_mfma_f32_16x16x32_f16 a[124:127], v[42:45], v[234:237], a[124:127]\n\t"
      "v_cvt_pk_f16_f32 v241, v198, v202\n\t"
      "v_mfma_f32_16x16x32_f16 a[120:123], v[42:45], v[226:229], a[120:123]\n\t"
      "ds_write_b128 %[vlw], v[238:241] offset:0\n\t"
      "v_mfma_f32_16x16x32_f16 a[120:123], v[38:41], v[222:225], a[120:123]\n\t"
      "buffer_load_dwordx4 v[18:21], %[vx0], s[68:71], s60 offen offset:1024 sc0 sc1\n\t"
      "v_mfma_f32_16x16x32_f16 a[152:155], v[46:49], v[222:225], a[152:155]\n\t"
      "v_cvt_pk_f16_f32 v238, v175, v179\n\t"
      "s_lshl1_add_u32 s93, s92, s93\n\t"
      "v_mfma_f32_16x16x32_f16 a[152:155], v[50:53], v[226:229], a[152:155]\n\t"
      "v_cvt_pk_f16_f32 v239, v183, v187\n\t"
      "v_mfma_f32_16x16x32_f16 a[156:159], v[50:53], v[234:237], a[156:159]\n\t"
      "v_cvt_pk_f16_f32 v240, v191, v195\n\t"
      "v_mfma_f32_16x16x32_f16 a[156:159], v[46:49], v[230:233], a[156:159]\n\t"
      "buffer_load_dwordx4 v[22:25], %[vx0], s[68:71], s60 offen offset:2048 sc0 sc1\n\t"
      "v_mfma_f32_16x16x32_f16 a[188:191], v[54:57], v[230:233], a[188:191]\n\t"
      "v_cvt_pk_f16_f32 v241, v199, v203\n\t"
      "v_mfma_f32_16x16x32_f16 a[188:191], v[58:61], v[234:237], a[188:191]\n\t"
      "ds_write_b128 %[vlw], v[238:241] offset:2048\n\t"
      "v_mfma_f32_16x16x32_f16 a[184:187], v[58:61], v[226:229], a[184:187]\n\t"
      "v_cvt_pk_f16_f32 v238, v176, v180\n\t"
      "v_mfma_f32_16x16x32_f16 a[184:187], v[54:57], v[222:225], a[184:187]\n\t"
      "buffer_load_dwordx4 v[26:29], %[vx0], s[68:71], s60 offen offset:3072 sc0 sc1\n\t"
      "s_waitcnt lgkmcnt(0)\n\t"
      "v_mfma_f32_16x16x32_f16 a[216:219], v[62:65], v[222:225], a[216:219]\n\t"
      "v_cvt_pk_f16_f32 v239, v184, v188\n\t"
      "s_sub_u32 s95, s94, s93\n\t"
      "v_mfma_f32_16x16x32_f16 a[216:219], v[66:69], v[226:229], a[216:219]\n\t"
      "v_cvt_pk_f16_f32 v240, v192, v196\n\t"
      "s_cmp_lt_i32 s95, 0\n\t"
      "s_cbranch_scc1 Lzl1C_%=\n\t"
      "Lzzr1C_%=:\n\t"
      "v_mfma_f32_16x16x32_f16 a[220:223], v[66:69], v[234:237], a[220:223]\n\t"
      "v_cvt_pk_f16_f32 v241, v200, v204\n\t"
      "s_sub_u32 s95, s95, 30\n\t"
      "v_mfma_f32_16x16x32_f16 a[220:223], v[62:65], v[230:233], a[220:223]\n\t"
      "buffer_load_dwordx4 v[30:33], %[vx1], s[68:71], s60 offen offset:0 sc0 sc1\n\t"
      "v_mfma_f32_16x16x32_f16 a[252:255], v[70:73], v[230:233], a[252:255]\n\t"
      "ds_write_b128 %[vlw], v[238:241] offset:4096\n\t"
      "v_mfma_f32_16x16x32_f16 a[252:255], v[74:77], v[234:237], a[252:255]\n\t"
      "v_cvt_pk_f16_f32 v238, v177, v181\n\t"
      "s_max_i32 s95, s95, 0\n\t"
      "v_mfma_f32_16x16x32_f16 a[248:251], v[74:77], v[226:229], a[248:251]\n\t"
      "v_cvt_pk_f16_f32 v239, v185, v189\n\t"
      "s_mul_i32 s86, s86, 3\n\t"
      "v_mfma_f32_16x16x32_f16 a[248:251], v[70:73], v[222:225], a[248:251]\n\t"
      "v_cvt_pk_f16_f32 v240, v193, v197\n\t"
      "s_lshr_b32 s86, s86, 2\n\t"
      "s_waitcnt vmcnt(15) lgkmcnt(3)\n\t"
      "v_mfma_f32_16x16x32_f16 a[0:3], v[78:81], v[206:209], a[0:3]\n\t"
      "buffer_load_dwordx4 v[34:37], %[vx1], s[68:71], s60 offen offset:1024 sc0 sc1\n\t"
      "v_mfma_f32_16x16x32_f16 a[0:3], v[82:85], v[210:213], a[0:3]\n\t"
      "ds_read_b128 v[222:225], %[vlr] offset:20480\n\t"
      "v_mfma_f32_16x16x32_f16 a[4:7], v[82:85], v[218:221], a[4:7]\n\t"
      "v_cvt_pk_f16_f32 v241, v201, v205\n\t"
      "v_mfma_f32_16x16x32_f16 a[4:7], v[78:81], v[214:217], a[4:7]\n\t"
      "ds_write_b128 %[vlw], v[238:241] offset:6144\n\t"
      "v_mfma_f32_16x16x32_f16 a[36:39], v[86:89], v[214:217], a[36:39]\n\t"
      "s_cmp_eq_u32 s62, s66\n\t"
      "s_cbranch_scc1 Lpush1C_%=\n\t"
      "Lpushr1C_%=:\n\t"
      "v_mfma_f32_16x16x32_f16 a[36:39], v[90:93], v[218:221], a[36:39]\n\t"
      "s_cmp_eq_u32 s62, s85\n\t"
      "s_cbranch_scc1 LsigC_%=\n\t"
      "LsigrC_%=:\n\t"
      "v_mfma_f32_16x16x32_f16 a[32:35], v[90:93], v[210:213], a[32:35]\n\t"
      "buffer_load_dwordx4 v[162:165], %[vw1], %[srdw], s61 offen nt\n\t"
      "v_mfma_f32_16x16x32_f16 a[32:35], v[86:89], v[206:209], a[32:35]\n\t"
      "ds_read_b128 v[226:229], %[vlr] offset:21504\n\t"
      "v_mfma_f32_16x16x32_f16 a[64:67], v[94:97], v[206:209], a[64:67]\n\t"
      "ds_read_b128 v[230:233], %[vlr] offset:22528\n\t"
      "v_mfma_f32_16x16x32_f16 a[64:67], v[98:101], v[210:213], a[64:67]\n\t"
      "s_cmp_ge_u32 s95, s59\n\t"
      "s_cselect_b32 s86, s55, s86\n\t"
      "v_mfma_f32_16x16x32_f16 a[68:71], v[98:101], v[218:221], a[68:71]\n\t"
      "buffer_load_dwordx4 v[166:169], %[vw2], %[srdw], s61 offen nt\n\t"
      "v_mfma_f32_16x16x32_f16 a[68:71], v[94:97], v[214:217], a[68:71]\n\t"
      "ds_read_b128 v[234:237], %[vlr] offset:23552\n\t"
      "s_add_u32 s95, s95, s86\n\t"
      "v_mfma_f32_16x16x32_f16 a[100:103], v[102:105], v[214:217], a[100:103]\n\t"
      "s_add_u32 s93, s93, s95\n\t"
      "s_add_u32 s98, s98, s95\n\t"
      "v_mfma_f32_16x16x32_f16 a[100:103], v[106:109], v[218:221], a[100:103]\n\t"
      "v_mfma_f32_16x16x32_f16 a[96:99], v[106:109], v[210:213], a[96:99]\n\t"
      "buffer_load_dwordx4 v[170:173], %[vw3], %[srdw], s61 offen nt\n\t"
      "v_mfma_f32_16x16x32_f16 a[96:99], v[102:105], v[206:209], a[96:99]\n\t"
      "s_mov_b32 s61, s65\n\t"
      "v_mfma_f32_16x16x32_f16 a[128:131], v[110:113], v[206:209], a[128:131]\n\t"
      "v_readlane_b32 s84, %[vtab], s62\n\t"
      "v_mfma_f32_16x16x32_f16 a[128:131], v[114:117], v[210:213], a[128:131]\n\t"
      "v_readlane_b32 s63, %[vtab], s89\n\t"
      "v_mfma_f32_16x16x32_f16 a[132:135], v[114:117], v[218:221], a[132:135]\n\t"
      "buffer_load_dwordx4 v[174:177], %[vw0], %[srdw], s61 offen nt\n\t"
      "v_mfma_f32_16x16x32_f16 a[132:135], v[110:113], v[214:217], a[132:135]\n\t"
      "s_lshl_b32 s64, s84, 8\n\t"
      "s_waitcnt vmcnt(10)\n\t"
      "v_mfma_f32_16x16x32_f16 a[164:167], v[118:121], v[214:217], a[164:167]\n\t"
      "s_and_b32 s64, s64, 0xff000000\n\t"
      "v_mfma_f32_16x16x32_f16 a[164:167], v[122:125], v[218:221], a[164:167]\n\t"
      "s_and_b32 s65, s63, 0xff000000\n\t"
      "v_mfma_f32_16x16x32_f16 a[160:163], v[122:125], v[210:213], a[160:163]\n\t"
      "buffer_load_dwordx4 v[178:181], %[vw1], %[srdw], s61 offen nt\n\t"
      "v_mfma_f32_16x16x32_f16 a[160:163], v[118:121], v[206:209], a[160:163]\n\t"
      "s_add_u32 s89, s89, 1\n\t"
      "v_mfma_f32_16x16x32_f16 a[192:195], v[126:129], v[206:209], a[192:195]\n\t"
      "s_and_b32 s67, s84, 0xffff\n\t"
      "v_mfma_f32_16x16x32_f16 a[192:195], v[130:133], v[210:213], a[192:195]\n\t"
      "v_mfma_f32_16x16x32_f16 a[196:199], v[130:133], v[218:221], a[196:199]\n\t"
      "v_mfma_f32_16x16x32_f16 a[196:199], v[126:129], v[214:217], a[196:199]\n\t"
      "v_mfma_f32_16x16x32_f16 a[228:231], v[134:137], v[214:217], a[228:231]\n\t"
      "v_mfma_f32_16x16x32_f16 a[228:231], v[138:141], v[218:221], a[228:231]\n\t"
      "v_mfma_f32_16x16x32_f16 a[224:227], v[138:141], v[210:213], a[224:227]\n\t"
      "v_mfma_f32_16x16x32_f16 a[224:227], v[134:137], v[206:209], a[224:227]\n\t"
      "s_waitcnt lgkmcnt(0)\n\t"
      "v_mfma_f32_16x16x32_f16 a[8:11], v[78:81], v[222:225], a[8:11]\n\t"
      "v_mfma_f32_16x16x32_f16 a[8:11], v[82:85], v[226:229], a[8:11]\n\t"
      "ds_read_b128 v[206:209], %[vlr] offset:24576\n\t"
      "v_mfma_f32_16x16x32_f16 a[12:15], v[82:85], v[234:237], a[12:15]\n\t"
      "v_mfma_f32_16x16x32_f16 a[12:15], v[78:81], v[230:233], a[12:15]\n\t"
      "v_mfma_f32_16x16x32_f16 a[44:47], v[86:89], v[230:233], a[44:47]\n\t"
      "ds_read_b128 v[210:213], %[vlr] offset:25600\n\t"
      "v_mfma_f32_16x16x32_f16 a[44:47], v[90:93], v[234:237], a[44:47]\n\t"
      "v_mfma_f32_16x16x32_f16 a[40:43], v[90:93], v[226:229], a[40:43]\n\t"
      "v_mfma_f32_16x16x32_f16 a[40:43], v[86:89], v[222:225], a[40:43]\n\t"
      "ds_read_b128 v[214:217], %[vlr] offset:26624\n\t"
      "v_mfma_f32_16x16x32_f16 a[72:75], v[94:97], v[222:225], a[72:75]\n\t"
      "v_mfma_f32_16x16x32_f16 a[72:75], v[98:101], v[226:229], a[72:75]\n\t"
      "v_mfma_f32_16x16x32_f16 a[76:79], v[98:101], v[234:237], a[76:79]\n\t"
      "ds_read_b128 v[218:221], %[vlr] offset:27648\n\t"
      "v_mfma_f32_16x16x32_f16 a[76:79], v[94:97], v[230:233], a[76:79]\n\t"
      "buffer_load_dwordx4 v[38:41], %[vx1], s[68:71], s60 offen offset:2048 sc0 sc1\n\t"
      "v_mfma_f32_16x16x32_f16 a[108:111], v[102:105], v[230:233], a[108:111]\n\t"
      "s_memrealtime s[94:95]\n\t"
      "v_mfma_f32_16x16x32_f16 a[108:111], v[106:109], v[234:237], a[108:111]\n\t"
      "v_mfma_f32_16x16x32_f16 a[104:107], v[106:109], v[226:229], a[104:107]\n\t"
      "v_mfma_f32_16x16x32_f16 a[104:107], v[102:105], v[222:225], a[104:107]\n\t"
      "buffer_load_dwordx4 v[42:45], %[vx1], s[68:71], s60 offen offset:3072 sc0 sc1\n\t"
      "v_mfma_f32_16x16x32_f16 a[136:139], v[110:113], v[222:225], a[136:139]\n\t"
      "s_add_u32 s60, s60, 0x2000\n\t"
      "s_lshl1_add_u32 s93, s92, s93\n\t"
      "v_mfma_f32_16x16x32_f16 a[136:139], v[114:117], v[226:229], a[136:139]\n\t"
      "v_mfma_f32_16x16x32_f16 a[140:143], v[114:117], v[234:237], a[140:143]\n\t"
      "v_mfma_f32_16x16x32_f16 a[140:143], v[110:113], v[230:233], a[140:143]\n\t"
      "buffer_load_dwordx4 v[46:49], %[vx0], s[68:71], s60 offen offset:0 sc0 sc1\n\t"
      "v_mfma_f32_16x16x32_f16 a[172:175], v[118:121], v[230:233], a[172:175]\n\t"
      "v_mfma_f32_16x16x32_f16 a[172:175], v[122:125], v[234:237], a[172:175]\n\t"
      "v_mfma_f32_16x16x32_f16 a[168:171], v[122:125], v[226:229], a[168:171]\n\t"
      "v_mfma_f32_16x16x32_f16 a[168:171], v[118:121], v[222:225], a[168:171]\n\t"
      "buffer_load_dwordx4 v[50:53], %[vx0], s[68:71], s60 offen offset:1024 sc0 sc1\n\t"
      "s_waitcnt lgkmcnt(0)\n\t"
      "v_mfma_f32_16x16x32_f16 a[200:203], v[126:129], v[222:225], a[200:203]\n\t"
      "s_sub_u32 s95, s94, s93\n\t"
      "v_mfma_f32_16x16x32_f16 a[200:203], v[130:133], v[226:229], a[200:203]\n\t"
      "s_cmp_lt_i32 s95, 0\n\t"
      "s_cbranch_scc1 Lzl2C_%=\n\t"
      "Lzzr2C_%=:\n\t"
      "v_mfma_f32_16x16x32_f16 a[204:207], v[130:133], v[234:237], a[204:207]\n\t"
      "s_sub_u32 s95, s95, 30\n\t"
      "s_max_i32 s95, s95, 0\n\t"
      "v_mfma_f32_16x16x32_f16 a[204:207], v[126:129], v[230:233], a[204:207]\n\t"
      "buffer_load_dwordx4 v[54:57], %[vx0], s[68:71], s60 offen offset:2048 sc0 sc1\n\t"
      "v_mfma_f32_16x16x32_f16 a[236:239], v[134:137], v[230:233], a[236:239]\n\t"
      "s_mul_i32 s86, s86, 3\n\t"
      "s_lshr_b32 s86, s86, 2\n\t"
      "v_mfma_f32_16x16x32_f16 a[236:239], v[138:141], v[234:237], a[236:239]\n\t"
      "s_cmp_ge_u32 s95, s59\n\t"
      "s_cselect_b32 s86, s55, s86\n\t"
      "v_mfma_f32_16x16x32_f16 a[232:235], v[138:141], v[226:229], a[232:235]\n\t"
      "s_add_u32 s95, s95, s86\n\t"
      "s_add_u32 s93, s93, s95\n\t"
      "v_mfma_f32_16x16x32_f16 a[232:235], v[134:137], v[222:225], a[232:235]\n\t"
      "buffer_load_dwordx4 v[58:61], %[vx0], s[68:71], s60 offen offset:3072 sc0 sc1\n\t"
      "s_waitcnt lgkmcnt(0)\n\t"
      "v_mfma_f32_16x16x32_f16 a[16:19], v[78:81], v[206:209], a[16:19]\n\t"
      "s_add_u32 s98, s98, s95\n\t"
      "s_sub_u32 s97, s93, s96\n\t"
      "v_mfma_f32_16x16x32_f16 a[16:19], v[82:85], v[210:213], a[16:19]\n\t"
      "ds_read_b128 v[222:225], %[vlr] offset:28672\n\t"
      "s_lshr_b32 s97, s97, 8\n\t"
      "v_mfma_f32_16x16x32_f16 a[20:23], v[82:85], v[218:221], a[20:23]\n\t"
      "s_min_u32 s97, s97, 63\n\t"
      "s_mov_b32 s54, s92\n\t"
      "v_mfma_f32_16x16x32_f16 a[20:23], v[78:81], v[214:217], a[20:23]\n\t"
      "v_readlane_b32 s90, %[vpad], s97\n\t"
      "v_mfma_f32_16x16x32_f16 a[52:55], v[86:89], v[214:217], a[52:55]\n\t"
      "buffer_load_dwordx4 v[62:65], %[vx1], s[68:71], s60 offen offset:0 sc0 sc1\n\t"
      "v_mfma_f32_16x16x32_f16 a[52:55], v[90:93], v[218:221], a[52:55]\n\t"
      "ds_read_b128 v[226:229], %[vlr] offset:29696\n\t"
      "s_min_u32 s98, s98, 600\n\t"
      "v_mfma_f32_16x16x32_f16 a[48:51], v[90:93], v[210:213], a[48:51]\n\t"
      "s_lshr_b32 s94, s98, 2\n\t"
      "s_and_b32 s92, s90, 0xff\n\t"
      "v_mfma_f32_16x16x32_f16 a[48:51], v[86:89], v[206:209], a[48:51]\n\t"
      "ds_read_b128 v[230:233], %[vlr] offset:30720\n\t"
      "s_lshr_b32 s97, s90, 24\n\t"
      "v_mfma_f32_16x16x32_f16 a[80:83], v[94:97], v[206:209], a[80:83]\n\t"
      "buffer_load_dwordx4 v[66:69], %[vx1], s[68:71], s60 offen offset:1024 sc0 sc1\n\t"
      "v_mfma_f32_16x16x32_f16 a[80:83], v[98:101], v[210:213], a[80:83]\n\t"
      "s_min_u32 s97, s97, s94\n\t"
      "v_mfma_f32_16x16x32_f16 a[84:87], v[98:101], v[218:221], a[84:87]\n\t"
      "ds_read_b128 v[234:237], %[vlr] offset:31744\n\t"
      "v_mfma_f32_16x16x32_f16 a[84:87], v[94:97], v[214:217], a[84:87]\n\t"
      "s_cmp_gt_u32 s86, 3\n\t"
      "s_cselect_b32 s97, 0, s97\n\t"
      "v_mfma_f32_16x16x32_f16 a[116:119], v[102:105], v[214:217], a[116:119]\n\t"
      "buffer_load_dwordx4 v[70:73], %[vx1], s[68:71], s60 offen offset:2048 sc0 sc1\n\t"
      "v_mfma_f32_16x16x32_f16 a[116:119], v[106:109], v[218:221], a[116:119]\n\t"
      "s_bfe_u32 s90, s90, 0x80008\n\t"
      "s_cbranch_scc1 Lbk2C_%=\n\t"
      "Lbkr2C_%=:\n\t"
      "v_mfma_f32_16x16x32_f16 a[112:115], v[106:109], v[210:213], a[112:115]\n\t"
      "s_sub_u32 s92, s92, s97\n\t"
      "s_lshl_b32 s97, s97, 2\n\t"
      "v_mfma_f32_16x16x32_f16 a[112:115], v[102:105], v[206:209], a[112:115]\n\t"
      "s_sub_u32 s98, s98, s97\n\t"
      "s_mov_b32 s87, s98\n\t"
      "v_mfma_f32_16x16x32_f16 a[144:147], v[110:113], v[206:209], a[144:147]\n\t"
      "buffer_load_dwordx4 v[74:77], %[vx1], s[68:71], s60 offen offset:3072 sc0 sc1\n\t"
      "v_mfma_f32_16x16x32_f16 a[144:147], v[114:117], v[210:213], a[144:147]\n\t"
      "s_add_u32 s60, s60, 122880\n\t"
      "v_mfma_f32_16x16x32_f16 a[148:151], v[114:117], v[218:221], a[148:151]\n\t"
      "v_mfma_f32_16x16x32_f16 a[148:151], v[110:113], v[214:217], a[148:151]\n\t"
      "v_mfma_f32_16x16x32_f16 a[180:183], v[118:121], v[214:217], a[180:183]\n\t"
      "s_cmp_eq_u32 s62, s66\n\t"
      "s_cbranch_scc1 Lpush2C_%=\n\t"
      "Lpushr2C_%=:\n\t"
      "v_mfma_f32_16x16x32_f16 a[180:183], v[122:125], v[218:221], a[180:183]\n\t"
      "buffer_load_dword v254, off, %[srdws], s67 sc0 sc1\n\t"
      "v_mfma_f32_16x16x32_f16 a[176:179], v[122:125], v[210:213], a[176:179]\n\t"
      "v_mfma_f32_16x16x32_f16 a[176:179], v[118:121], v[206:209], a[176:179]\n\t"
      "buffer_load_dwordx4 v[182:185], %[vw2], %[srdw], s61 offen nt\n\t"
      "v_mfma_f32_16x16x32_f16 a[208:211], v[126:129], v[206:209], a[208:211]\n\t"
      "v_mfma_f32_16x16x32_f16 a[208:211], v[130:133], v[210:213], a[208:211]\n\t"
      "v_add_u32 %[vlr], 0x8000, %[vlr]\n\t"
      "v_mfma_f32_16x16x32_f16 a[212:215], v[130:133], v[218:221], a[212:215]\n\t"
      "v_and_b32 %[vlr], 0x1ffff, %[vlr]\n\t"
      "v_mfma_f32_16x16x32_f16 a[212:215], v[126:129], v[214:217], a[212:215]\n\t"
      "v_mfma_f32_16x16x32_f16 a[244:247], v[134:137], v[214:217], a[244:247]\n\t"
      "buffer_load_dwordx4 v[186:189], %[vw3], %[srdw], s61 offen nt\n\t"
      "v_mfma_f32_16x16x32_f16 a[244:247], v[138:141], v[218:221], a[244:247]\n\t"
      "s_add_u32 s61, s61, 0x80000\n\t"
      "v_mfma_f32_16x16x32_f16 a[240:243], v[138:141], v[210:213], a[240:243]\n\t"
      "v_mfma_f32_16x16x32_f16 a[240:243], v[134:137], v[206:209], a[240:243]\n\t"
      "s_waitcnt lgkmcnt(0)\n\t"
      "s_barrier\n\t"
      "s_cmp_eq_u32 s62, 29\n\t"
      "s_cbranch_scc1 LexitC_%=\n\t"
      "v_mfma_f32_16x16x32_f16 a[24:27], v[78:81], v[222:225], a[24:27]\n\t"
      "buffer_load_dwordx4 v[190:193], %[vw0], %[srdw], s61 offen nt\n\t"
      "v_mfma_f32_16x16x32_f16 a[24:27], v[82:85], v[226:229], a[24:27]\n\t"
      "ds_read_b128 v[206:209], %[vlr] offset:0\n\t"
      "v_mfma_f32_16x16x32_f16 a[28:31], v[82:85], v[234:237], a[28:31]\n\t"
      "ds_read_b128 v[210:213], %[vlr] offset:1024\n\t"
      "v_mfma_f32_16x16x32_f16 a[28:31], v[78:81], v[230:233], a[28:31]\n\t"
      "v_mfma_f32_16x16x32_f16 a[60:63], v[86:89], v[230:233], a[60:63]\n\t"
      "buffer_load_dwordx4 v[194:197], %[vw1], %[srdw], s61 offen nt\n\t"
      "v_mfma_f32_16x16x32_f16 a[60:63], v[90:93], v[234:237], a[60:63]\n\t"
      "ds_read_b128 v[214:217], %[vlr] offset:2048\n\t"
      "v_mfma_f32_16x16x32_f16 a[56:59], v[90:93], v[226:229], a[56:59]\n\t"
      "ds_read_b128 v[218:221], %[vlr] offset:3072\n\t"
      "v_mfma_f32_16x16x32_f16 a[56:59], v[86:89], v[222:225], a[56:59]\n\t"
      "v_mfma_f32_16x16x32_f16 a[88:91], v[94:97], v[222:225], a[88:91]\n\t"
      "buffer_load_dwordx4 v[198:201], %[vw2], %[srdw], s61 offen nt\n\t"
      "v_mfma_f32_16x16x32_f16 a[88:91], v[98:101], v[226:229], a[88:91]\n\t"
      "s_memrealtime s[94:95]\n\t"
      "v_mfma_f32_16x16x32_f16 a[92:95], v[98:101], v[234:237], a[92:95]\n\t"
      "v_mfma_f32_16x16x32_f16 a[92:95], v[94:97], v[230:233], a[92:95]\n\t"
      "v_mfma_f32_16x16x32_f16 a[124:127], v[102:105], v[230:233], a[124:127]\n\t"
      "buffer_load_dwordx4 v[202:205], %[vw3], %[srdw], s61 offen nt\n\t"
      "v_mfma_f32_16x16x32_f16 a[124:127], v[106:109], v[234:237], a[124:127]\n\t"
      "s_add_u32 s61, s61, 7864320\n\t"
      "s_lshl1_add_u32 s93, s92, s93\n\t"
      "v_mfma_f32_16x16x32_f16 a[120:123], v[106:109], v[226:229], a[120:123]\n\t"
      "v_mfma_f32_16x16x32_f16 a[120:123], v[102:105], v[222:225], a[120:123]\n\t"
      "v_mfma_f32_16x16x32_f16 a[152:155], v[110:113], v[222:225], a[152:155]\n\t"
      "v_mfma_f32_16x16x32_f16 a[152:155], v[114:117], v[226:229], a[152:155]\n\t"
      "v_mfma_f32_16x16x32_f16 a[156:159], v[114:117], v[234:237], a[156:159]\n\t"
      "v_mfma_f32_16x16x32_f16 a[156:159], v[110:113], v[230:233], a[156:159]\n\t"
      "s_waitcnt lgkmcnt(0)\n\t"
      "v_mfma_f32_16x16x32_f16 a[188:191], v[118:121], v[230:233], a[188:191]\n\t"
      "s_sub_u32 s95, s94, s93\n\t"
      "v_mfma_f32_16x16x32_f16 a[188:191], v[122:125], v[234:237], a[188:191]\n\t"
      "s_cmp_lt_i32 s95, 0\n\t"
      "s_cbranch_scc1 Lzl3C_%=\n\t"
      "Lzzr3C_%=:\n\t"
      "v_mfma_f32_16x16x32_f16 a[184:187], v[122:125], v[226:229], a[184:187]\n\t"
      "s_sub_u32 s95, s95, 30\n\t"
      "s_max_i32 s95, s95, 0\n\t"
      "v_mfma_f32_16x16x32_f16 a[184:187], v[118:121], v[222:225], a[184:187]\n\t"
      "s_mul_i32 s86, s86, 3\n\t"
      "s_lshr_b32 s86, s86, 2\n\t"
      "v_mfma_f32_16x16x32_f16 a[216:219], v[126:129], v[222:225], a[216:219]\n\t"
      "s_cmp_ge_u32 s95, s59\n\t"
      "s_cselect_b32 s86, s55, s86\n\t"
      "v_mfma_f32_16x16x32_f16 a[216:219], v[130:133], v[226:229], a[216:219]\n\t"
      "s_add_u32 s95, s95, s86\n\t"
      "s_add_u32 s93, s93, s95\n\t"
      "v_mfma_f32_16x16x32_f16 a[220:223], v[130:133], v[234:237], a[220:223]\n\t"
      "s_add_u32 s98, s98, s95\n\t"
      "v_mfma_f32_16x16x32_f16 a[220:223], v[126:129], v[230:233], a[220:223]\n\t"
      "s_add_u32 s62, s62, 1\n\t"
      "v_mfma_f32_16x16x32_f16 a[252:255], v[134:137], v[230:233], a[252:255]\n\t"
      "v_mfma_f32_16x16x32_f16 a[252:255], v[138:141], v[234:237], a[252:255]\n\t"
      "v_mfma_f32_16x16x32_f16 a[248:251], v[138:141], v[226:229], a[248:251]\n\t"
      "v_mfma_f32_16x16x32_f16 a[248:251], v[134:137], v[222:225], a[248:251]\n\t"
      "s_branch LloopC_%=\n\t"
      "LexitC_%=:\n\t"
      "s_waitcnt vmcnt(15)\n\t"
      "s_cmp_lg_u32 s85, 30\n\t"
      "s_cbranch_scc1 LepiC_%=\n\t"
      "s_waitcnt vmcnt(0)\n\t"
      "s_mov_b32 s62, 31\n\t"
      "s_call_b64 s[82:83], Lsigf_%=\n\t"
      "s_branch LepiC_%=\n\t"
      "Lpush0C_%=:\n\t"
      "s_cmp_lg_u32 s91, 0\n\t"
      "s_cbranch_scc1 Lpq0C_%=\n\t"
      "s_add_u32 m0, %[sxst], 0\n\t"
      "s_add_u32 s84, %[sxr], 512\n\t"
      "buffer_load_dwordx4 %[vxs], %[srdxs], s84 offen nt lds\n\t"
      "s_add_u32 m0, m0, 1024\n\t"
      "s_add_u32 s84, %[sxr], 528\n\t"
      "buffer_load_dwordx4 %[vxs], %[srdxs], s84 offen nt lds\n\t"
      "s_add_u32 m0, m0, 1024\n\t"
      "s_branch Lpushr0C_%=\n\t"
      "Lpq0C_%=:\n\t"
      "v_and_b32 v250, 0x3ff, %[vlr]\n\t"
      "v_add_u32 v250, %[sxst], v250\n\t"
      "ds_read_b128 v[242:245], v250 offset:0\n\t"
      "ds_read_b128 v[246:249], v250 offset:1024\n\t"
      "s_waitcnt lgkmcnt(0)\n\t"
      "v_cvt_pk_f16_f32 v250, v242, v243\n\t"
      "v_cvt_pk_f16_f32 v251, v244, v245\n\t"
      "v_cvt_pk_f16_f32 v252, v246, v247\n\t"
      "v_cvt_pk_f16_f32 v253, v248, v249\n\t"
      "global_store_dwordx4 %[vpd], v[250:253], s[76:77] sc0 sc1\n\t"
      "s_add_u32 s76, s76, 262144\n\t"
      "s_addc_u32 s77, s77, 0\n\t"
      "s_branch Lpushr0C_%=\n\t"
      "Lpush1C_%=:\n\t"
      "s_cmp_lg_u32 s91, 0\n\t"
      "s_cbranch_scc1 Lpq1C_%=\n\t"
      "s_add_u32 m0, %[sxst], 2048\n\t"
      "s_add_u32 s84, %[sxr], 1024\n\t"
      "buffer_load_dwordx4 %[vxs], %[srdxs], s84 offen nt lds\n\t"
      "s_add_u32 m0, m0, 1024\n\t"
      "s_add_u32 s84, %[sxr], 1040\n\t"
      "buffer_load_dwordx4 %[vxs], %[srdxs], s84 offen nt lds\n\t"
      "s_add_u32 m0, m0, 1024\n\t"
      "s_branch Lpushr1C_%=\n\t"
      "Lpq1C_%=:\n\t"
      "v_and_b32 v250, 0x3ff, %[vlr]\n\t"
      "v_add_u32 v250, %[sxst], v250\n\t"
      "ds_read_b128 v[242:245], v250 offset:2048\n\t"
      "ds_read_b128 v[246:249], v250 offset:3072\n\t"
      "s_waitcnt lgkmcnt(0)\n\t"
      "v_cvt_pk_f16_f32 v250, v242, v243\n\t"
      "v_cvt_pk_f16_f32 v251, v244, v245\n\t"
      "v_cvt_pk_f16_f32 v252, v246, v247\n\t"
      "v_cvt_pk_f16_f32 v253, v248, v249\n\t"
      "global_store_dwordx4 %[vpd], v[250:253], s[76:77] sc0 sc1\n\t"
      "s_add_u32 s76, s76, 262144\n\t"
      "s_addc_u32 s77, s77, 0\n\t"
      "s_branch Lpushr1C_%=\n\t"
      "Lpush2C_%=:\n\t"
      "s_cmp_lg_u32 s91, 0\n\t"
      "s_cbranch_scc1 Lpq2C_%=\n\t"
      "s_add_u32 m0, %[sxst], 4096\n\t"
      "s_add_u32 s84, %[sxr], 1536\n\t"
      "buffer_load_dwordx4 %[vxs], %[srdxs], s84 offen nt lds\n\t"
      "s_add_u32 m0, m0, 1024\n\t"
      "s_add_u32 s84, %[sxr], 1552\n\t"
      "buffer_load_dwordx4 %[vxs], %[srdxs], s84 offen nt lds\n\t"
      "s_add_u32 m0, m0, 1024\n\t"
      "s_mov_b32 s91, 1\n\t"
      "s_add_u32 s66, s66, 1\n\t"
      "s_branch Lpushr2C_%=\n\t"
      "Lpq2C_%=:\n\t"
      "v_and_b32 v250, 0x3ff, %[vlr]\n\t"
      "v_add_u32 v250, %[sxst], v250\n\t"
      "ds_read_b128 v[242:245], v250 offset:4096\n\t"
      "ds_read_b128 v[246:249], v250 offset:5120\n\t"
      "s_waitcnt lgkmcnt(0)\n\t"
      "v_cvt_pk_f16_f32 v250, v242, v243\n\t"
      "v_cvt_pk_f16_f32 v251, v244, v245\n\t"
      "v_cvt_pk_f16_f32 v252, v246, v247\n\t"
      "v_cvt_pk_f16_f32 v253, v248, v249\n\t"
      "global_store_dwordx4 %[vpd], v[250:253], s[76:77] sc0 sc1\n\t"
      "s_add_u32 s76, s76, 262144\n\t"
      "s_addc_u32 s77, s77, 0\n\t"
      "s_branch Lpushr2C_%=\n\t"
      "LextC_%=:\n\t"
      "buffer_load_dwordx4 v[142:145], %[vw0], %[srdw], s61 offen nt\n\t"
      "buffer_load_dwordx4 v[146:149], %[vw1], %[srdw], s61 offen nt\n\t"
      "buffer_load_dwordx4 v[150:153], %[vw2], %[srdw], s61 offen nt\n\t"
      "buffer_load_dwordx4 v[154:157], %[vw3], %[srdw], s61 offen nt\n\t"
      "s_add_u32 s61, s61, 0x80000\n\t"
      "buffer_load_dwordx4 v[158:161], %[vw0], %[srdw], s61 offen nt\n\t"
      "buffer_load_dwordx4 v[162:165], %[vw1], %[srdw], s61 offen nt\n\t"
      "buffer_load_dwordx4 v[166:169], %[vw2], %[srdw], s61 offen nt\n\t"
      "buffer_load_dwordx4 v[170:173], %[vw3], %[srdw], s61 offen nt\n\t"
      "s_add_u32 s61, s61, 7864320\n\t"
      "v_readlane_b32 s63, %[vtab], s89\n\t"
      "v_add_u32 %[vlw], 0x8000, %[vlw]\n\t"
      "v_and_b32 %[vlw], 0x1ffff, %[vlw]\n\t"
      "s_add_u32 s93, s93, 150\n\t"
      "s_waitcnt vmcnt(19)\n\t"
      "v_cvt_pk_f16_f32 v238, v174, v178\n\t"
      "v_cvt_pk_f16_f32 v239, v182, v186\n\t"
      "v_cvt_pk_f16_f32 v240, v190, v194\n\t"
      "v_cvt_pk_f16_f32 v241, v198, v202\n\t"
      "ds_write_b128 %[vlw], v[238:241] offset:0\n\t"
      "s_nop 1\n\t"
      "v_cvt_pk_f16_f32 v238, v175, v179\n\t"
      "v_cvt_pk_f16_f32 v239, v183, v187\n\t"
      "v_cvt_pk_f16_f32 v240, v191, v195\n\t"
      "v_cvt_pk_f16_f32 v241, v199, v203\n\t"
      "ds_write_b128 %[vlw], v[238:241] offset:2048\n\t"
      "s_nop 1\n\t"
      "v_cvt_pk_f16_f32 v238, v176, v180\n\t"
      "v_cvt_pk_f16_f32 v239, v184, v188\n\t"
      "v_cvt_pk_f16_f32 v240, v192, v196\n\t"
      "v_cvt_pk_f16_f32 v241, v200, v204\n\t"
      "ds_write_b128 %[vlw], v[238:241] offset:4096\n\t"
      "s_nop 1\n\t"
      "v_cvt_pk_f16_f32 v238, v177, v181\n\t"
      "v_cvt_pk_f16_f32 v239, v185, v189\n\t"
      "v_cvt_pk_f16_f32 v240, v193, v197\n\t"
      "v_cvt_pk_f16_f32 v241, v201, v205\n\t"
      "ds_write_b128 %[vlw], v[238:241] offset:6144\n\t"
      "s_nop 1\n\t"
      "s_mov_b32 s61, s65\n\t"
      "buffer_load_dwordx4 v[174:177], %[vw0], %[srdw], s61 offen nt\n\t"
      "buffer_load_dwordx4 v[178:181], %[vw1], %[srdw], s61 offen nt\n\t"
      "buffer_load_dwordx4 v[182:185], %[vw2], %[srdw], s61 offen nt\n\t"
      "buffer_load_dwordx4 v[186:189], %[vw3], %[srdw], s61 offen nt\n\t"
      "s_add_u32 s61, s61, 0x80000\n\t"
      "buffer_load_dwordx4 v[190:193], %[vw0], %[srdw], s61 offen nt\n\t"
      "buffer_load_dwordx4 v[194:197], %[vw1], %[srdw], s61 offen nt\n\t"
      "buffer_load_dwordx4 v[198:201], %[vw2], %[srdw], s61 offen nt\n\t"
      "buffer_load_dwordx4 v[202:205], %[vw3], %[srdw], s61 offen nt\n\t"
      "s_add_u32 s61, s61, 7864320\n\t"
      "s_and_b32 s65, s63, 0xff000000\n\t"
      "s_add_u32 s89, s89, 1\n\t"
      "s_add_u32 s88, s88, 5\n\t"
      "s_sub_u32 s63, s89, s62\n\t"
      "s_cmp_ge_u32 s63, 1\n\t"
      "s_cselect_b32 s88, 0xff, s88\n\t"
      "v_mov_b32 v254, 0\n\t"
      "s_waitcnt vmcnt(8)\n\t"
      "v_cvt_pk_f16_f32 v238, v142, v146\n\t"
      "v_cvt_pk_f16_f32 v239, v150, v154\n\t"
      "v_cvt_pk_f16_f32 v240, v158, v162\n\t"
      "v_cvt_pk_f16_f32 v241, v166, v170\n\t"
      "ds_write_b128 %[vlw], v[238:241] offset:16384\n\t"
      "s_nop 1\n\t"
      "v_cvt_pk_f16_f32 v238, v143, v147\n\t"
      "v_cvt_pk_f16_f32 v239, v151, v155\n\t"
      "v_cvt_pk_f16_f32 v240, v159, v163\n\t"
      "v_cvt_pk_f16_f32 v241, v167, v171\n\t"
      "ds_write_b128 %[vlw], v[238:241] offset:18432\n\t"
      "s_nop 1\n\t"
      "v_cvt_pk_f16_f32 v238, v144, v148\n\t"
      "v_cvt_pk_f16_f32 v239, v152, v156\n\t"
      "v_cvt_pk_f16_f32 v240, v160, v164\n\t"
      "v_cvt_pk_f16_f32 v241, v168, v172\n\t"
      "ds_write_b128 %[vlw], v[238:241] offset:20480\n\t"
      "s_nop 1\n\t"
      "v_cvt_pk_f16_f32 v238, v145, v149\n\t"
      "v_cvt_pk_f16_f32 v239, v153, v157\n\t"
      "v_cvt_pk_f16_f32 v240, v161, v165\n\t"
      "v_cvt_pk_f16_f32 v241, v169, v173\n\t"
      "ds_write_b128 %[vlw], v[238:241] offset:22528\n\t"
      "s_nop 1\n\t"
      "s_branch LextrC_%=\n\t"
      "LsigC_%=:\n\t"
      "s_call_b64 s[82:83], Lsigf_%=\n\t"
      "s_branch LsigrC_%=\n\t"
      "Lzl0C_%=:\n\t"
      "s_sub_u32 s50, s50, s95\n\t"
      "Lzm0C_%=:\n\t"
      "s_sleep 1\n\t"
      "s_memrealtime s[94:95]\n\t"
      "s_waitcnt lgkmcnt(0)\n\t"
      "s_sub_u32 s95, s94, s93\n\t"
      "s_cmp_lt_i32 s95, 0\n\t"
      "s_cbranch_scc1 Lzm0C_%=\n\t"
      "s_branch Lzzr0C_%=\n\t"
      "Lbk0C_%=:\n\t"
      "s_cmp_lg_u32 s62, 2\n\t"
      "s_cbranch_scc1 Lvd0C_%=\n\t"
      "s_and_b32 s81, s74, 1\n\t"
      "s_lshl_b32 s81, s81, 7\n\t"
      "s_add_u32 s81, s81, 37120\n\t"
      "buffer_load_dword v255, off, %[srdws], s81 sc1\n\t"
      "s_mov_b32 s57, 1\n\t"
      "Lvd0C_%=:\n\t"
      "s_lshl2_add_u32 s54, s54, s98\n\t"
      "s_sub_u32 s54, s54, s87\n\t"
      "s_sub_u32 s54, s54, s50\n\t"
      "s_mov_b32 s50, 0\n\t"
      "s_cmp_lt_u32 s90, 8\n\t"
      "s_cbranch_scc1 Lbkr0C_%=\n\t"
      "s_mov_b32 s55, 33\n\t"
      "s_mov_b32 s52, 100\n\t"
      "s_cmp_eq_u32 s62, s66\n\t"
      "s_cbranch_scc1 Lbkr0C_%=\n\t"
      "s_cmp_lt_i32 s54, 172\n\t"
      "s_cbranch_scc1 Lbkr0C_%=\n\t"
      "s_sub_u32 s97, 0, s90\n\t"
      "s_branch Lbkr0C_%=\n\t"
      "Lzl1C_%=:\n\t"
      "s_sub_u32 s50, s50, s95\n\t"
      "Lzm1C_%=:\n\t"
      "s_sleep 1\n\t"
      "s_memrealtime s[94:95]\n\t"
      "s_waitcnt lgkmcnt(0)\n\t"
      "s_sub_u32 s95, s94, s93\n\t"
      "s_cmp_lt_i32 s95, 0\n\t"
      "s_cbranch_scc1 Lzm1C_%=\n\t"
      "s_branch Lzzr1C_%=\n\t"
      "Lzl2C_%=:\n\t"
      "s_sub_u32 s50, s50, s95\n\t"
      "Lzm2C_%=:\n\t"
      "s_sleep 1\n\t"
      "s_memrealtime s[94:95]\n\t"
      "s_waitcnt lgkmcnt(0)\n\t"
      "s_sub_u32 s95, s94, s93\n\t"
      "s_cmp_lt_i32 s95, 0\n\t"
      "s_cbranch_scc1 Lzm2C_%=\n\t"
      "s_branch Lzzr2C_%=\n\t"
      "Lbk2C_%=:\n\t"
      "s_cmp_lg_u32 s62, 2\n\t"
      "s_cbranch_scc1 Lvd2C_%=\n\t"
      "s_cmp_eq_u32 s57, 0\n\t"
      "s_cbranch_scc1 Lvd2C_%=\n\t"
      "v_readfirstlane_b32 s81, v255\n\t"
      "s_nop 3\n\t"
      "s_lshr_b32 s80, s81, 16\n\t"
      "s_and_b32 s81, s81, 0xffff\n\t"
      "s_cmp_lt_u32 s80, 12\n\t"
      "s_cbranch_scc1 Lvd2C_%=\n\t"
      "s_mul_i32 s80, s80, 162\n\t"
      "s_cmp_ge_u32 s81, s80\n\t"
      "s_cbranch_scc1 Lvd2C_%=\n\t"
      "v_bfe_u32 %[vpad], %[vpad], 16, 8\n\t"
      "v_or_b32 %[vpad], 67108864, %[vpad]\n\t"
      "s_mov_b32 s55, 12\n\t"
      "s_mov_b32 s52, 100\n\t"
      "Lvd2C_%=:\n\t"
      "s_lshl2_add_u32 s54, s54, s98\n\t"
      "s_sub_u32 s54, s54, s87\n\t"
      "s_sub_u32 s54, s54, s50\n\t"
      "s_mov_b32 s50, 0\n\t"
      "s_cmp_lt_u32 s90, 8\n\t"
      "s_cbranch_scc1 Lbkr2C_%=\n\t"
      "s_mov_b32 s55, 33\n\t"
      "s_mov_b32 s52, 100\n\t"
      "s_cmp_eq_u32 s62, s66\n\t"
      "s_cbranch_scc1 Lbkr2C_%=\n\t"
      "s_cmp_lt_i32 s54, 172\n\t"
      "s_cbranch_scc1 Lbkr2C_%=\n\t"
      "s_sub_u32 s97, 0, s90\n\t"
      "s_branch Lbkr2C_%=\n\t"
      "Lzl3C_%=:\n\t"
      "s_sub_u32 s50, s50, s95\n\t"
      "Lzm3C_%=:\n\t"
      "s_sleep 1\n\t"
      "s_memrealtime s[94:95]\n\t"
      "s_waitcnt lgkmcnt(0)\n\t"
      "s_sub_u32 s95, s94, s93\n\t"
      "s_cmp_lt_i32 s95, 0\n\t"
      "s_cbranch_scc1 Lzm3C_%=\n\t"
      "s_branch Lzzr3C_%=\n\t"
      "LslowC_%=:\n\t"
      "s_call_b64 s[82:83], Lpoll_%=\n\t"
      "s_waitcnt vmcnt(0)\n\t"
      "s_sub_u32 s81, s80, s78\n\t"
      "s_cmp_lt_u32 s81, 60\n\t"
      "s_cbranch_scc1 LslowrC_%=\n\t"
      "s_sub_u32 s81, s80, s93\n\t"
      "s_cmp_gt_i32 s81, 0\n\t"
      "s_cselect_b32 s93, s80, s93\n\t"
      "s_branch LslowrC_%=\n\t"
      ".p2align 6\n\t"
      "LloopH_%=:\n\t"
      "s_waitcnt vmcnt(8) lgkmcnt(0)\n\t"
      "Lloop2H_%=:\n\t"
      "v_mfma_f32_16x16x32_f16 a[0:3], v[14:17], v[206:209], a[0:3]\n\t"
      "v_cvt_pk_f16_f32 v238, v142, v146\n\t"
      "v_mfma_f32_16x16x32_f16 a[0:3], v[18:21], v[210:213], a[0:3]\n\t"
      "buffer_load_dwordx4 v[78:81], %[vx0], s[68:71], s60 offen offset:0 sc0 sc1\n\t"
      "v_mfma_f32_16x16x32_f16 a[4:7], v[18:21], v[218:221], a[4:7]\n\t"
      "ds_read_b128 v[222:225], %[vlr] offset:4096\n\t"
      "v_mfma_f32_16x16x32_f16 a[4:7], v[14:17], v[214:217], a[4:7]\n\t"
      "v_cvt_pk_f16_f32 v239, v150, v154\n\t"
      "v_mfma_f32_16x16x32_f16 a[36:39], v[22:25], v[214:217], a[36:39]\n\t"
      "ds_read_b128 v[226:229], %[vlr] offset:5120\n\t"
      "v_mfma_f32_16x16x32_f16 a[36:39], v[26:29], v[218:221], a[36:39]\n\t"
      "buffer_load_dwordx4 v[82:85], %[vx0], s[68:71], s60 offen offset:1024 sc0 sc1\n\t"
      "v_mfma_f32_16x16x32_f16 a[32:35], v[26:29], v[210:213], a[32:35]\n\t"
      "v_cvt_pk_f16_f32 v240, v158, v162\n\t"
      "v_mfma_f32_16x16x32_f16 a[32:35], v[22:25], v[206:209], a[32:35]\n\t"
      "ds_read_b128 v[230:233], %[vlr] offset:6144\n\t"
      "v_mfma_f32_16x16x32_f16 a[64:67], v[30:33], v[206:209], a[64:67]\n\t"
      "v_cvt_pk_f16_f32 v241, v166, v170\n\t"
      "v_mfma_f32_16x16x32_f16 a[64:67], v[34:37], v[210:213], a[64:67]\n\t"
      "buffer_load_dwordx4 v[86:89], %[vx0], s[68:71], s60 offen offset:2048 sc0 sc1\n\t"
      "v_mfma_f32_16x16x32_f16 a[68:71], v[34:37], v[218:221], a[68:71]\n\t"
      "ds_read_b128 v[234:237], %[vlr] offset:7168\n\t"
      "v_mfma_f32_16x16x32_f16 a[68:71], v[30:33], v[214:217], a[68:71]\n\t"
      "ds_write_b128 %[vlw], v[238:241] offset:16384\n\t"
      "v_mfma_f32_16x16x32_f16 a[100:103], v[38:41], v[214:217], a[100:103]\n\t"
      "v_cvt_pk_f16_f32 v238, v143, v147\n\t"
      "v_mfma_f32_16x16x32_f16 a[100:103], v[42:45], v[218:221], a[100:103]\n\t"
      "buffer_load_dwordx4 v[90:93], %[vx0], s[68:71], s60 offen offset:3072 sc0 sc1\n\t"
      "v_mfma_f32_16x16x32_f16 a[96:99], v[42:45], v[210:213], a[96:99]\n\t"
      "v_cvt_pk_f16_f32 v239, v151, v155\n\t"
      "v_mfma_f32_16x16x32_f16 a[96:99], v[38:41], v[206:209], a[96:99]\n\t"
      "v_cvt_pk_f16_f32 v240, v159, v163\n\t"
      "v_mfma_f32_16x16x32_f16 a[128:131], v[46:49], v[206:209], a[128:131]\n\t"
      "v_cvt_pk_f16_f32 v241, v167, v171\n\t"
      "v_mfma_f32_16x16x32_f16 a[128:131], v[50:53], v[210:213], a[128:131]\n\t"
      "buffer_load_dwordx4 v[94:97], %[vx1], s[68:71], s60 offen offset:0 sc0 sc1\n\t"
      "v_mfma_f32_16x16x32_f16 a[132:135], v[50:53], v[218:221], a[132:135]\n\t"
      "ds_write_b128 %[vlw], v[238:241] offset:18432\n\t"
      "v_mfma_f32_16x16x32_f16 a[132:135], v[46:49], v[214:217], a[132:135]\n\t"
      "v_cvt_pk_f16_f32 v238, v144, v148\n\t"
      "v_mfma_f32_16x16x32_f16 a[164:167], v[54:57], v[214:217], a[164:167]\n\t"
      "v_cvt_pk_f16_f32 v239, v152, v156\n\t"
      "v_mfma_f32_16x16x32_f16 a[164:167], v[58:61], v[218:221], a[164:167]\n\t"
      "v_cvt_pk_f16_f32 v240, v160, v164\n\t"
      "v_mfma_f32_16x16x32_f16 a[160:163], v[58:61], v[210:213], a[160:163]\n\t"
      "buffer_load_dwordx4 v[98:101], %[vx1], s[68:71], s60 offen offset:1024 sc0 sc1\n\t"
      "v_mfma_f32_16x16x32_f16 a[160:163], v[54:57], v[206:209], a[160:163]\n\t"
      "v_cvt_pk_f16_f32 v241, v168, v172\n\t"
      "v_mfma_f32_16x16x32_f16 a[192:195], v[62:65], v[206:209], a[192:195]\n\t"
      "ds_write_b128 %[vlw], v[238:241] offset:20480\n\t"
      "v_mfma_f32_16x16x32_f16 a[192:195], v[66:69], v[210:213], a[192:195]\n\t"
      "v_cvt_pk_f16_f32 v238, v145, v149\n\t"
      "v_mfma_f32_16x16x32_f16 a[196:199], v[66:69], v[218:221], a[196:199]\n\t"
      "buffer_load_dwordx4 v[102:105], %[vx1], s[68:71], s60 offen offset:2048 sc0 sc1\n\t"
      "v_mfma_f32_16x16x32_f16 a[196:199], v[62:65], v[214:217], a[196:199]\n\t"
      "v_cvt_pk_f16_f32 v239, v153, v157\n\t"
      "v_mfma_f32_16x16x32_f16 a[228:231], v[70:73], v[214:217], a[228:231]\n\t"
      "v_cvt_pk_f16_f32 v240, v161, v165\n\t"
      "v_mfma_f32_16x16x32_f16 a[228:231], v[74:77], v[218:221], a[228:231]\n\t"
      "v_cvt_pk_f16_f32 v241, v169, v173\n\t"
      "v_mfma_f32_16x16x32_f16 a[224:227], v[74:77], v[210:213], a[224:227]\n\t"
      "buffer_load_dwordx4 v[106:109], %[vx1], s[68:71], s60 offen offset:3072 sc0 sc1\n\t"
      "v_mfma_f32_16x16x32_f16 a[224:227], v[70:73], v[206:209], a[224:227]\n\t"
      "s_add_u32 s60, s60, 0x2000\n\t"
      "s_waitcnt lgkmcnt(3)\n\t"
      "v_mfma_f32_16x16x32_f16 a[8:11], v[14:17], v[222:225], a[8:11]\n\t"
      "ds_write_b128 %[vlw], v[238:241] offset:22528\n\t"
      "v_mfma_f32_16x16x32_f16 a[8:11], v[18:21], v[226:229], a[8:11]\n\t"
      "ds_read_b128 v[206:209], %[vlr] offset:8192\n\t"
      "v_mfma_f32_16x16x32_f16 a[12:15], v[18:21], v[234:237], a[12:15]\n\t"
      "buffer_load_dwordx4 v[110:113], %[vx0], s[68:71], s60 offen offset:0 sc0 sc1\n\t"
      "v_mfma_f32_16x16x32_f16 a[12:15], v[14:17], v[230:233], a[12:15]\n\t"
      "v_mfma_f32_16x16x32_f16 a[44:47], v[22:25], v[230:233], a[44:47]\n\t"
      "ds_read_b128 v[210:213], %[vlr] offset:9216\n\t"
      "v_mfma_f32_16x16x32_f16 a[44:47], v[26:29], v[234:237], a[44:47]\n\t"
      "v_mfma_f32_16x16x32_f16 a[40:43], v[26:29], v[226:229], a[40:43]\n\t"
      "buffer_load_dwordx4 v[114:117], %[vx0], s[68:71], s60 offen offset:1024 sc0 sc1\n\t"
      "v_mfma_f32_16x16x32_f16 a[40:43], v[22:25], v[222:225], a[40:43]\n\t"
      "ds_read_b128 v[214:217], %[vlr] offset:10240\n\t"
      "v_mfma_f32_16x16x32_f16 a[72:75], v[30:33], v[222:225], a[72:75]\n\t"
      "v_mfma_f32_16x16x32_f16 a[72:75], v[34:37], v[226:229], a[72:75]\n\t"
      "v_mfma_f32_16x16x32_f16 a[76:79], v[34:37], v[234:237], a[76:79]\n\t"
      "ds_read_b128 v[218:221], %[vlr] offset:11264\n\t"
      "v_mfma_f32_16x16x32_f16 a[76:79], v[30:33], v[230:233], a[76:79]\n\t"
      "s_cmp_eq_u32 s62, s66\n\t"
      "s_cbranch_scc1 Lpush0H_%=\n\t"
      "Lpushr0H_%=:\n\t"
      "v_mfma_f32_16x16x32_f16 a[108:111], v[38:41], v[230:233], a[108:111]\n\t"
      "buffer_load_dwordx4 v[142:145], %[vw0], %[srdw], s61 offen nt\n\t"
      "v_mfma_f32_16x16x32_f16 a[108:111], v[42:45], v[234:237], a[108:111]\n\t"
      "v_mfma_f32_16x16x32_f16 a[104:107], v[42:45], v[226:229], a[104:107]\n\t"
      "v_mfma_f32_16x16x32_f16 a[104:107], v[38:41], v[222:225], a[104:107]\n\t"
      "v_mfma_f32_16x16x32_f16 a[136:139], v[46:49], v[222:225], a[136:139]\n\t"
      "buffer_load_dwordx4 v[146:149], %[vw1], %[srdw], s61 offen nt\n\t"
      "v_mfma_f32_16x16x32_f16 a[136:139], v[50:53], v[226:229], a[136:139]\n\t"
      "v_mfma_f32_16x16x32_f16 a[140:143], v[50:53], v[234:237], a[140:143]\n\t"
      "v_mfma_f32_16x16x32_f16 a[140:143], v[46:49], v[230:233], a[140:143]\n\t"
      "v_mfma_f32_16x16x32_f16 a[172:175], v[54:57], v[230:233], a[172:175]\n\t"
      "buffer_load_dwordx4 v[150:153], %[vw2], %[srdw], s61 offen nt\n\t"
      "v_mfma_f32_16x16x32_f16 a[172:175], v[58:61], v[234:237], a[172:175]\n\t"
      "v_mfma_f32_16x16x32_f16 a[168:171], v[58:61], v[226:229], a[168:171]\n\t"
      "v_mfma_f32_16x16x32_f16 a[168:171], v[54:57], v[222:225], a[168:171]\n\t"
      "v_mfma_f32_16x16x32_f16 a[200:203], v[62:65], v[222:225], a[200:203]\n\t"
      "buffer_load_dwordx4 v[154:157], %[vw3], %[srdw], s61 offen nt\n\t"
      "v_mfma_f32_16x16x32_f16 a[200:203], v[66:69], v[226:229], a[200:203]\n\t"
      "s_add_u32 s61, s61, 0x80000\n\t"
      "v_mfma_f32_16x16x32_f16 a[204:207], v[66:69], v[234:237], a[204:207]\n\t"
      "v_mfma_f32_16x16x32_f16 a[204:207], v[62:65], v[230:233], a[204:207]\n\t"
      "v_mfma_f32_16x16x32_f16 a[236:239], v[70:73], v[230:233], a[236:239]\n\t"
      "v_mfma_f32_16x16x32_f16 a[236:239], v[74:77], v[234:237], a[236:239]\n\t"
      "v_mfma_f32_16x16x32_f16 a[232:235], v[74:77], v[226:229], a[232:235]\n\t"
      "v_mfma_f32_16x16x32_f16 a[232:235], v[70:73], v[222:225], a[232:235]\n\t"
      "s_waitcnt lgkmcnt(0)\n\t"
      "v_mfma_f32_16x16x32_f16 a[16:19], v[14:17], v[206:209], a[16:19]\n\t"
      "v_mfma_f32_16x16x32_f16 a[16:19], v[18:21], v[210:213], a[16:19]\n\t"
      "ds_read_b128 v[222:225], %[vlr] offset:12288\n\t"
      "v_mfma_f32_16x16x32_f16 a[20:23], v[18:21], v[218:221], a[20:23]\n\t"
      "v_mfma_f32_16x16x32_f16 a[20:23], v[14:17], v[214:217], a[20:23]\n\t"
      "v_mfma_f32_16x16x32_f16 a[52:55], v[22:25], v[214:217], a[52:55]\n\t"
      "ds_read_b128 v[226:229], %[vlr] offset:13312\n\t"
      "v_mfma_f32_16x16x32_f16 a[52:55], v[26:29], v[218:221], a[52:55]\n\t"
      "v_mfma_f32_16x16x32_f16 a[48:51], v[26:29], v[210:213], a[48:51]\n\t"
      "v_mfma_f32_16x16x32_f16 a[48:51], v[22:25], v[206:209], a[48:51]\n\t"
      "ds_read_b128 v[230:233], %[vlr] offset:14336\n\t"
      "v_mfma_f32_16x16x32_f16 a[80:83], v[30:33], v[206:209], a[80:83]\n\t"
      "v_mfma_f32_16x16x32_f16 a[80:83], v[34:37], v[210:213], a[80:83]\n\t"
      "v_mfma_f32_16x16x32_f16 a[84:87], v[34:37], v[218:221], a[84:87]\n\t"
      "ds_read_b128 v[234:237], %[vlr] offset:15360\n\t"
      "v_mfma_f32_16x16x32_f16 a[84:87], v[30:33], v[214:217], a[84:87]\n\t"
      "v_mfma_f32_16x16x32_f16 a[116:119], v[38:41], v[214:217], a[116:119]\n\t"
      "v_mfma_f32_16x16x32_f16 a[116:119], v[42:45], v[218:221], a[116:119]\n\t"
      "v_mfma_f32_16x16x32_f16 a[112:115], v[42:45], v[210:213], a[112:115]\n\t"
      "v_mfma_f32_16x16x32_f16 a[112:115], v[38:41], v[206:209], a[112:115]\n\t"
      "buffer_load_dwordx4 v[118:121], %[vx0], s[68:71], s60 offen offset:2048 sc0 sc1\n\t"
      "v_mfma_f32_16x16x32_f16 a[144:147], v[46:49], v[206:209], a[144:147]\n\t"
      "v_mfma_f32_16x16x32_f16 a[144:147], v[50:53], v[210:213], a[144:147]\n\t"
      "v_mfma_f32_16x16x32_f16 a[148:151], v[50:53], v[218:221], a[148:151]\n\t"
      "v_mfma_f32_16x16x32_f16 a[148:151], v[46:49], v[214:217], a[148:151]\n\t"
      "buffer_load_dwordx4 v[122:125], %[vx0], s[68:71], s60 offen offset:3072 sc0 sc1\n\t"
      "v_mfma_f32_16x16x32_f16 a[180:183], v[54:57], v[214:217], a[180:183]\n\t"
      "v_mfma_f32_16x16x32_f16 a[180:183], v[58:61], v[218:221], a[180:183]\n\t"
      "v_mfma_f32_16x16x32_f16 a[176:179], v[58:61], v[210:213], a[176:179]\n\t"
      "v_mfma_f32_16x16x32_f16 a[176:179], v[54:57], v[206:209], a[176:179]\n\t"
      "buffer_load_dwordx4 v[126:129], %[vx1], s[68:71], s60 offen offset:0 sc0 sc1\n\t"
      "v_mfma_f32_16x16x32_f16 a[208:211], v[62:65], v[206:209], a[208:211]\n\t"
      "v_mfma_f32_16x16x32_f16 a[208:211], v[66:69], v[210:213], a[208:211]\n\t"
      "v_mfma_f32_16x16x32_f16 a[212:215], v[66:69], v[218:221], a[212:215]\n\t"
      "v_mfma_f32_16x16x32_f16 a[212:215], v[62:65], v[214:217], a[212:215]\n\t"
      "buffer_load_dwordx4 v[130:133], %[vx1], s[68:71], s60 offen offset:1024 sc0 sc1\n\t"
      "v_mfma_f32_16x16x32_f16 a[244:247], v[70:73], v[214:217], a[244:247]\n\t"
      "v_mfma_f32_16x16x32_f16 a[244:247], v[74:77], v[218:221], a[244:247]\n\t"
      "v_mfma_f32_16x16x32_f16 a[240:243], v[74:77], v[210:213], a[240:243]\n\t"
      "v_mfma_f32_16x16x32_f16 a[240:243], v[70:73], v[206:209], a[240:243]\n\t"
      "s_waitcnt vmcnt(18) lgkmcnt(0)\n\t"
      "s_barrier\n\t"
      "v_mfma_f32_16x16x32_f16 a[24:27], v[14:17], v[222:225], a[24:27]\n\t"
      "buffer_load_dwordx4 v[134:137], %[vx1], s[68:71], s60 offen offset:2048 sc0 sc1\n\t"
      "v_mfma_f32_16x16x32_f16 a[24:27], v[18:21], v[226:229], a[24:27]\n\t"
      "v_readfirstlane_b32 s63, v254\n\t"
      "v_mfma_f32_16x16x32_f16 a[28:31], v[18:21], v[234:237], a[28:31]\n\t"
      "s_cmp_lt_u32 s63, s73\n\t"
      "s_cbranch_scc1 LslowH_%=\n\t"
      "LslowrH_%=:\n\t"
      "v_mfma_f32_16x16x32_f16 a[28:31], v[14:17], v[230:233], a[28:31]\n\t"
      "ds_read_b128 v[206:209], %[vlr] offset:16384\n\t"
      "v_mfma_f32_16x16x32_f16 a[60:63], v[22:25], v[230:233], a[60:63]\n\t"
      "buffer_load_dwordx4 v[138:141], %[vx1], s[68:71], s60 offen offset:3072 sc0 sc1\n\t"
      "v_mfma_f32_16x16x32_f16 a[60:63], v[26:29], v[234:237], a[60:63]\n\t"
      "s_lshr_b32 s60, s64, 6\n\t"
      "v_mfma_f32_16x16x32_f16 a[56:59], v[26:29], v[226:229], a[56:59]\n\t"
      "ds_read_b128 v[210:213], %[vlr] offset:17408\n\t"
      "v_mfma_f32_16x16x32_f16 a[56:59], v[22:25], v[222:225], a[56:59]\n\t"
      "ds_read_b128 v[214:217], %[vlr] offset:18432\n\t"
      "v_mfma_f32_16x16x32_f16 a[88:91], v[30:33], v[222:225], a[88:91]\n\t"
      "buffer_load_dwordx4 v[14:17], %[vx0], s[68:71], s60 offen offset:0 sc0 sc1\n\t"
      "v_mfma_f32_16x16x32_f16 a[88:91], v[34:37], v[226:229], a[88:91]\n\t"
      "ds_read_b128 v[218:221], %[vlr] offset:19456\n\t"
      "v_mfma_f32_16x16x32_f16 a[92:95], v[34:37], v[234:237], a[92:95]\n\t"
      "v_cvt_pk_f16_f32 v238, v174, v178\n\t"
      "v_mfma_f32_16x16x32_f16 a[92:95], v[30:33], v[230:233], a[92:95]\n\t"
      "v_cvt_pk_f16_f32 v239, v182, v186\n\t"
      "v_mfma_f32_16x16x32_f16 a[124:127], v[38:41], v[230:233], a[124:127]\n\t"
      "buffer_load_dwordx4 v[18:21], %[vx0], s[68:71], s60 offen offset:1024 sc0 sc1\n\t"
      "v_mfma_f32_16x16x32_f16 a[124:127], v[42:45], v[234:237], a[124:127]\n\t"
      "v_cvt_pk_f16_f32 v240, v190, v194\n\t"
      "v_mfma_f32_16x16x32_f16 a[120:123], v[42:45], v[226:229], a[120:123]\n\t"
      "v_cvt_pk_f16_f32 v241, v198, v202\n\t"
      "v_mfma_f32_16x16x32_f16 a[120:123], v[38:41], v[222:225], a[120:123]\n\t"
      "ds_write_b128 %[vlw], v[238:241] offset:0\n\t"
      "v_mfma_f32_16x16x32_f16 a[152:155], v[46:49], v[222:225], a[152:155]\n\t"
      "buffer_load_dwordx4 v[22:25], %[vx0], s[68:71], s60 offen offset:2048 sc0 sc1\n\t"
      "v_mfma_f32_16x16x32_f16 a[152:155], v[50:53], v[226:229], a[152:155]\n\t"
      "v_cvt_pk_f16_f32 v238, v175, v179\n\t"
      "v_mfma_f32_16x16x32_f16 a[156:159], v[50:53], v[234:237], a[156:159]\n\t"
      "v_cvt_pk_f16_f32 v239, v183, v187\n\t"
      "v_mfma_f32_16x16x32_f16 a[156:159], v[46:49], v[230:233], a[156:159]\n\t"
      "v_cvt_pk_f16_f32 v240, v191, v195\n\t"
      "v_mfma_f32_16x16x32_f16 a[188:191], v[54:57], v[230:233], a[188:191]\n\t"
      "buffer_load_dwordx4 v[26:29], %[vx0], s[68:71], s60 offen offset:3072 sc0 sc1\n\t"
      "v_mfma_f32_16x16x32_f16 a[188:191], v[58:61], v[234:237], a[188:191]\n\t"
      "v_cvt_pk_f16_f32 v241, v199, v203\n\t"
      "v_mfma_f32_16x16x32_f16 a[184:187], v[58:61], v[226:229], a[184:187]\n\t"
      "ds_write_b128 %[vlw], v[238:241] offset:2048\n\t"
      "v_mfma_f32_16x16x32_f16 a[184:187], v[54:57], v[222:225], a[184:187]\n\t"
      "v_cvt_pk_f16_f32 v238, v176, v180\n\t"
      "v_mfma_f32_16x16x32_f16 a[216:219], v[62:65], v[222:225], a[216:219]\n\t"
      "v_cvt_pk_f16_f32 v239, v184, v188\n\t"
      "v_mfma_f32_16x16x32_f16 a[216:219], v[66:69], v[226:229], a[216:219]\n\t"
      "s_cmp_eq_u32 s62, s66\n\t"
      "s_cbranch_scc1 Lpush1H_%=\n\t"
      "Lpushr1H_%=:\n\t"
      "v_mfma_f32_16x16x32_f16 a[220:223], v[66:69], v[234:237], a[220:223]\n\t"
      "s_cmp_eq_u32 s62, s85\n\t"
      "s_cbranch_scc1 LsigH_%=\n\t"
      "LsigrH_%=:\n\t"
      "v_mfma_f32_16x16x32_f16 a[220:223], v[62:65], v[230:233], a[220:223]\n\t"
      "buffer_load_dwordx4 v[158:161], %[vw0], %[srdw], s61 offen nt\n\t"
      "v_mfma_f32_16x16x32_f16 a[252:255], v[70:73], v[230:233], a[252:255]\n\t"
      "v_cvt_pk_f16_f32 v240, v192, v196\n\t"
      "v_mfma_f32_16x16x32_f16 a[252:255], v[74:77], v[234:237], a[252:255]\n\t"
      "v_cvt_pk_f16_f32 v241, v200, v204\n\t"
      "v_mfma_f32_16x16x32_f16 a[248:251], v[74:77], v[226:229], a[248:251]\n\t"
      "ds_write_b128 %[vlw], v[238:241] offset:4096\n\t"
      "v_mfma_f32_16x16x32_f16 a[248:251], v[70:73], v[222:225], a[248:251]\n\t"
      "buffer_load_dwordx4 v[162:165], %[vw1], %[srdw], s61 offen nt\n\t"
      "s_waitcnt vmcnt(11) lgkmcnt(3)\n\t"
      "v_mfma_f32_16x16x32_f16 a[0:3], v[78:81], v[206:209], a[0:3]\n\t"
      "v_cvt_pk_f16_f32 v238, v177, v181\n\t"
      "v_mfma_f32_16x16x32_f16 a[0:3], v[82:85], v[210:213], a[0:3]\n\t"
      "ds_read_b128 v[222:225], %[vlr] offset:20480\n\t"
      "v_mfma_f32_16x16x32_f16 a[4:7], v[82:85], v[218:221], a[4:7]\n\t"
      "v_cvt_pk_f16_f32 v239, v185, v189\n\t"
      "v_mfma_f32_16x16x32_f16 a[4:7], v[78:81], v[214:217], a[4:7]\n\t"
      "buffer_load_dwordx4 v[166:169], %[vw2], %[srdw], s61 offen nt\n\t"
      "v_mfma_f32_16x16x32_f16 a[36:39], v[86:89], v[214:217], a[36:39]\n\t"
      "ds_read_b128 v[226:229], %[vlr] offset:21504\n\t"
      "v_mfma_f32_16x16x32_f16 a[36:39], v[90:93], v[218:221], a[36:39]\n\t"
      "v_cvt_pk_f16_f32 v240, v193, v197\n\t"
      "v_mfma_f32_16x16x32_f16 a[32:35], v[90:93], v[210:213], a[32:35]\n\t"
      "v_cvt_pk_f16_f32 v241, v201, v205\n\t"
      "v_mfma_f32_16x16x32_f16 a[32:35], v[86:89], v[206:209], a[32:35]\n\t"
      "buffer_load_dwordx4 v[170:173], %[vw3], %[srdw], s61 offen nt\n\t"
      "v_mfma_f32_16x16x32_f16 a[64:67], v[94:97], v[206:209], a[64:67]\n\t"
      "s_mov_b32 s61, s65\n\t"
      "v_mfma_f32_16x16x32_f16 a[64:67], v[98:101], v[210:213], a[64:67]\n\t"
      "ds_read_b128 v[230:233], %[vlr] offset:22528\n\t"
      "v_mfma_f32_16x16x32_f16 a[68:71], v[98:101], v[218:221], a[68:71]\n\t"
      "ds_read_b128 v[234:237], %[vlr] offset:23552\n\t"
      "v_mfma_f32_16x16x32_f16 a[68:71], v[94:97], v[214:217], a[68:71]\n\t"
      "buffer_load_dwordx4 v[174:177], %[vw0], %[srdw], s61 offen nt\n\t"
      "v_mfma_f32_16x16x32_f16 a[100:103], v[102:105], v[214:217], a[100:103]\n\t"
      "ds_write_b128 %[vlw], v[238:241] offset:6144\n\t"
      "v_mfma_f32_16x16x32_f16 a[100:103], v[106:109], v[218:221], a[100:103]\n\t"
      "v_readlane_b32 s84, %[vtab], s62\n\t"
      "v_mfma_f32_16x16x32_f16 a[96:99], v[106:109], v[210:213], a[96:99]\n\t"
      "s_mov_b32 s64, s65\n\t"
      "v_mfma_f32_16x16x32_f16 a[96:99], v[102:105], v[206:209], a[96:99]\n\t"
      "s_and_b32 s65, s84, 0xff000000\n\t"
      "v_mfma_f32_16x16x32_f16 a[128:131], v[110:113], v[206:209], a[128:131]\n\t"
      "s_and_b32 s67, s84, 0xffff\n\t"
      "v_mfma_f32_16x16x32_f16 a[128:131], v[114:117], v[210:213], a[128:131]\n\t"
      "v_mfma_f32_16x16x32_f16 a[132:135], v[114:117], v[218:221], a[132:135]\n\t"
      "v_mfma_f32_16x16x32_f16 a[132:135], v[110:113], v[214:217], a[132:135]\n\t"
      "s_waitcnt vmcnt(9)\n\t"
      "v_mfma_f32_16x16x32_f16 a[164:167], v[118:121], v[214:217], a[164:167]\n\t"
      "v_mfma_f32_16x16x32_f16 a[164:167], v[122:125], v[218:221], a[164:167]\n\t"
      "v_mfma_f32_16x16x32_f16 a[160:163], v[122:125], v[210:213], a[160:163]\n\t"
      "v_mfma_f32_16x16x32_f16 a[160:163], v[118:121], v[206:209], a[160:163]\n\t"
      "v_mfma_f32_16x16x32_f16 a[192:195], v[126:129], v[206:209], a[192:195]\n\t"
      "v_mfma_f32_16x16x32_f16 a[192:195], v[130:133], v[210:213], a[192:195]\n\t"
      "v_mfma_f32_16x16x32_f16 a[196:199], v[130:133], v[218:221], a[196:199]\n\t"
      "v_mfma_f32_16x16x32_f16 a[196:199], v[126:129], v[214:217], a[196:199]\n\t"
      "v_mfma_f32_16x16x32_f16 a[228:231], v[134:137], v[214:217], a[228:231]\n\t"
      "v_mfma_f32_16x16x32_f16 a[228:231], v[138:141], v[218:221], a[228:231]\n\t"
      "v_mfma_f32_16x16x32_f16 a[224:227], v[138:141], v[210:213], a[224:227]\n\t"
      "v_mfma_f32_16x16x32_f16 a[224:227], v[134:137], v[206:209], a[224:227]\n\t"
      "buffer_load_dwordx4 v[30:33], %[vx1], s[68:71], s60 offen offset:0 sc0 sc1\n\t"
      "s_waitcnt lgkmcnt(1)\n\t"
      "v_mfma_f32_16x16x32_f16 a[8:11], v[78:81], v[222:225], a[8:11]\n\t"
      "v_mfma_f32_16x16x32_f16 a[8:11], v[82:85], v[226:229], a[8:11]\n\t"
      "ds_read_b128 v[206:209], %[vlr] offset:24576\n\t"
      "v_mfma_f32_16x16x32_f16 a[12:15], v[82:85], v[234:237], a[12:15]\n\t"
      "v_mfma_f32_16x16x32_f16 a[12:15], v[78:81], v[230:233], a[12:15]\n\t"
      "buffer_load_dwordx4 v[34:37], %[vx1], s[68:71], s60 offen offset:1024 sc0 sc1\n\t"
      "v_mfma_f32_16x16x32_f16 a[44:47], v[86:89], v[230:233], a[44:47]\n\t"
      "ds_read_b128 v[210:213], %[vlr] offset:25600\n\t"
      "v_mfma_f32_16x16x32_f16 a[44:47], v[90:93], v[234:237], a[44:47]\n\t"
      "v_mfma_f32_16x16x32_f16 a[40:43], v[90:93], v[226:229], a[40:43]\n\t"
      "v_mfma_f32_16x16x32_f16 a[40:43], v[86:89], v[222:225], a[40:43]\n\t"
      "buffer_load_dwordx4 v[38:41], %[vx1], s[68:71], s60 offen offset:2048 sc0 sc1\n\t"
      "v_mfma_f32_16x16x32_f16 a[72:75], v[94:97], v[222:225], a[72:75]\n\t"
      "ds_read_b128 v[214:217], %[vlr] offset:26624\n\t"
      "v_mfma_f32_16x16x32_f16 a[72:75], v[98:101], v[226:229], a[72:75]\n\t"
      "v_mfma_f32_16x16x32_f16 a[76:79], v[98:101], v[234:237], a[76:79]\n\t"
      "ds_read_b128 v[218:221], %[vlr] offset:27648\n\t"
      "v_mfma_f32_16x16x32_f16 a[76:79], v[94:97], v[230:233], a[76:79]\n\t"
      "buffer_load_dwordx4 v[42:45], %[vx1], s[68:71], s60 offen offset:3072 sc0 sc1\n\t"
      "v_mfma_f32_16x16x32_f16 a[108:111], v[102:105], v[230:233], a[108:111]\n\t"
      "s_add_u32 s60, s60, 0x2000\n\t"
      "v_mfma_f32_16x16x32_f16 a[108:111], v[106:109], v[234:237], a[108:111]\n\t"
      "v_mfma_f32_16x16x32_f16 a[104:107], v[106:109], v[226:229], a[104:107]\n\t"
      "v_mfma_f32_16x16x32_f16 a[104:107], v[102:105], v[222:225], a[104:107]\n\t"
      "buffer_load_dwordx4 v[46:49], %[vx0], s[68:71], s60 offen offset:0 sc0 sc1\n\t"
      "v_mfma_f32_16x16x32_f16 a[136:139], v[110:113], v[222:225], a[136:139]\n\t"
      "v_mfma_f32_16x16x32_f16 a[136:139], v[114:117], v[226:229], a[136:139]\n\t"
      "v_mfma_f32_16x16x32_f16 a[140:143], v[114:117], v[234:237], a[140:143]\n\t"
      "v_mfma_f32_16x16x32_f16 a[140:143], v[110:113], v[230:233], a[140:143]\n\t"
      "buffer_load_dwordx4 v[50:53], %[vx0], s[68:71], s60 offen offset:1024 sc0 sc1\n\t"
      "v_mfma_f32_16x16x32_f16 a[172:175], v[118:121], v[230:233], a[172:175]\n\t"
      "v_mfma_f32_16x16x32_f16 a[172:175], v[122:125], v[234:237], a[172:175]\n\t"
      "v_mfma_f32_16x16x32_f16 a[168:171], v[122:125], v[226:229], a[168:171]\n\t"
      "v_mfma_f32_16x16x32_f16 a[168:171], v[118:121], v[222:225], a[168:171]\n\t"
      "v_mfma_f32_16x16x32_f16 a[200:203], v[126:129], v[222:225], a[200:203]\n\t"
      "buffer_load_dwordx4 v[54:57], %[vx0], s[68:71], s60 offen offset:2048 sc0 sc1\n\t"
      "v_mfma_f32_16x16x32_f16 a[200:203], v[130:133], v[226:229], a[200:203]\n\t"
      "v_mfma_f32_16x16x32_f16 a[204:207], v[130:133], v[234:237], a[204:207]\n\t"
      "v_mfma_f32_16x16x32_f16 a[204:207], v[126:129], v[230:233], a[204:207]\n\t"
      "v_mfma_f32_16x16x32_f16 a[236:239], v[134:137], v[230:233], a[236:239]\n\t"
      "buffer_load_dwordx4 v[58:61], %[vx0], s[68:71], s60 offen offset:3072 sc0 sc1\n\t"
      "v_mfma_f32_16x16x32_f16 a[236:239], v[138:141], v[234:237], a[236:239]\n\t"
      "v_mfma_f32_16x16x32_f16 a[232:235], v[138:141], v[226:229], a[232:235]\n\t"
      "v_mfma_f32_16x16x32_f16 a[232:235], v[134:137], v[222:225], a[232:235]\n\t"
      "s_waitcnt lgkmcnt(0)\n\t"
      "v_mfma_f32_16x16x32_f16 a[16:19], v[78:81], v[206:209], a[16:19]\n\t"
      "buffer_load_dwordx4 v[62:65], %[vx1], s[68:71], s60 offen offset:0 sc0 sc1\n\t"
      "v_mfma_f32_16x16x32_f16 a[16:19], v[82:85], v[210:213], a[16:19]\n\t"
      "ds_read_b128 v[222:225], %[vlr] offset:28672\n\t"
      "v_mfma_f32_16x16x32_f16 a[20:23], v[82:85], v[218:221], a[20:23]\n\t"
      "v_mfma_f32_16x16x32_f16 a[20:23], v[78:81], v[214:217], a[20:23]\n\t"
      "v_mfma_f32_16x16x32_f16 a[52:55], v[86:89], v[214:217], a[52:55]\n\t"
      "buffer_load_dwordx4 v[66:69], %[vx1], s[68:71], s60 offen offset:1024 sc0 sc1\n\t"
      "v_mfma_f32_16x16x32_f16 a[52:55], v[90:93], v[218:221], a[52:55]\n\t"
      "ds_read_b128 v[226:229], %[vlr] offset:29696\n\t"
      "v_mfma_f32_16x16x32_f16 a[48:51], v[90:93], v[210:213], a[48:51]\n\t"
      "v_mfma_f32_16x16x32_f16 a[48:51], v[86:89], v[206:209], a[48:51]\n\t"
      "ds_read_b128 v[230:233], %[vlr] offset:30720\n\t"
      "v_mfma_f32_16x16x32_f16 a[80:83], v[94:97], v[206:209], a[80:83]\n\t"
      "buffer_load_dwordx4 v[70:73], %[vx1], s[68:71], s60 offen offset:2048 sc0 sc1\n\t"
      "v_mfma_f32_16x16x32_f16 a[80:83], v[98:101], v[210:213], a[80:83]\n\t"
      "v_mfma_f32_16x16x32_f16 a[84:87], v[98:101], v[218:221], a[84:87]\n\t"
      "ds_read_b128 v[234:237], %[vlr] offset:31744\n\t"
      "v_mfma_f32_16x16x32_f16 a[84:87], v[94:97], v[214:217], a[84:87]\n\t"
      "v_mfma_f32_16x16x32_f16 a[116:119], v[102:105], v[214:217], a[116:119]\n\t"
      "v_mfma_f32_16x16x32_f16 a[116:119], v[106:109], v[218:221], a[116:119]\n\t"
      "buffer_load_dwordx4 v[74:77], %[vx1], s[68:71], s60 offen offset:3072 sc0 sc1\n\t"
      "v_mfma_f32_16x16x32_f16 a[112:115], v[106:109], v[210:213], a[112:115]\n\t"
      "s_add_u32 s60, s60, 122880\n\t"
      "v_mfma_f32_16x16x32_f16 a[112:115], v[102:105], v[206:209], a[112:115]\n\t"
      "v_mfma_f32_16x16x32_f16 a[144:147], v[110:113], v[206:209], a[144:147]\n\t"
      "v_mfma_f32_16x16x32_f16 a[144:147], v[114:117], v[210:213], a[144:147]\n\t"
      "s_cmp_eq_u32 s62, s66\n\t"
      "s_cbranch_scc1 Lpush2H_%=\n\t"
      "Lpushr2H_%=:\n\t"
      "v_mfma_f32_16x16x32_f16 a[148:151], v[114:117], v[218:221], a[148:151]\n\t"
      "buffer_load_dword v254, off, %[srdws], s67 sc0 sc1\n\t"
      "v_mfma_f32_16x16x32_f16 a[148:151], v[110:113], v[214:217], a[148:151]\n\t"
      "v_mfma_f32_16x16x32_f16 a[180:183], v[118:121], v[214:217], a[180:183]\n\t"
      "buffer_load_dwordx4 v[178:181], %[vw1], %[srdw], s61 offen\n\t"
      "v_mfma_f32_16x16x32_f16 a[180:183], v[122:125], v[218:221], a[180:183]\n\t"
      "v_mfma_f32_16x16x32_f16 a[176:179], v[122:125], v[210:213], a[176:179]\n\t"
      "v_mfma_f32_16x16x32_f16 a[176:179], v[118:121], v[206:209], a[176:179]\n\t"
      "v_mfma_f32_16x16x32_f16 a[208:211], v[126:129], v[206:209], a[208:211]\n\t"
      "buffer_load_dwordx4 v[182:185], %[vw2], %[srdw], s61 offen\n\t"
      "v_mfma_f32_16x16x32_f16 a[208:211], v[130:133], v[210:213], a[208:211]\n\t"
      "v_mfma_f32_16x16x32_f16 a[212:215], v[130:133], v[218:221], a[212:215]\n\t"
      "v_mfma_f32_16x16x32_f16 a[212:215], v[126:129], v[214:217], a[212:215]\n\t"
      "v_mfma_f32_16x16x32_f16 a[244:247], v[134:137], v[214:217], a[244:247]\n\t"
      "buffer_load_dwordx4 v[186:189], %[vw3], %[srdw], s61 offen\n\t"
      "v_mfma_f32_16x16x32_f16 a[244:247], v[138:141], v[218:221], a[244:247]\n\t"
      "s_add_u32 s61, s61, 0x80000\n\t"
      "v_mfma_f32_16x16x32_f16 a[240:243], v[138:141], v[210:213], a[240:243]\n\t"
      "v_mfma_f32_16x16x32_f16 a[240:243], v[134:137], v[206:209], a[240:243]\n\t"
      "s_waitcnt lgkmcnt(0)\n\t"
      "s_barrier\n\t"
      "s_cmp_eq_u32 s62, 30\n\t"
      "s_cbranch_scc1 LexitH_%=\n\t"
      "v_mfma_f32_16x16x32_f16 a[24:27], v[78:81], v[222:225], a[24:27]\n\t"
      "buffer_load_dwordx4 v[190:193], %[vw0], %[srdw], s61 offen\n\t"
      "v_mfma_f32_16x16x32_f16 a[24:27], v[82:85], v[226:229], a[24:27]\n\t"
      "ds_read_b128 v[206:209], %[vlr] offset:0\n\t"
      "v_mfma_f32_16x16x32_f16 a[28:31], v[82:85], v[234:237], a[28:31]\n\t"
      "ds_read_b128 v[210:213], %[vlr] offset:1024\n\t"
      "v_mfma_f32_16x16x32_f16 a[28:31], v[78:81], v[230:233], a[28:31]\n\t"
      "v_mfma_f32_16x16x32_f16 a[60:63], v[86:89], v[230:233], a[60:63]\n\t"
      "ds_read_b128 v[214:217], %[vlr] offset:2048\n\t"
      "v_mfma_f32_16x16x32_f16 a[60:63], v[90:93], v[234:237], a[60:63]\n\t"
      "buffer_load_dwordx4 v[194:197], %[vw1], %[srdw], s61 offen\n\t"
      "v_mfma_f32_16x16x32_f16 a[56:59], v[90:93], v[226:229], a[56:59]\n\t"
      "ds_read_b128 v[218:221], %[vlr] offset:3072\n\t"
      "v_mfma_f32_16x16x32_f16 a[56:59], v[86:89], v[222:225], a[56:59]\n\t"
      "v_mfma_f32_16x16x32_f16 a[88:91], v[94:97], v[222:225], a[88:91]\n\t"
      "v_mfma_f32_16x16x32_f16 a[88:91], v[98:101], v[226:229], a[88:91]\n\t"
      "buffer_load_dwordx4 v[198:201], %[vw2], %[srdw], s61 offen\n\t"
      "v_mfma_f32_16x16x32_f16 a[92:95], v[98:101], v[234:237], a[92:95]\n\t"
      "v_mfma_f32_16x16x32_f16 a[92:95], v[94:97], v[230:233], a[92:95]\n\t"
      "v_mfma_f32_16x16x32_f16 a[124:127], v[102:105], v[230:233], a[124:127]\n\t"
      "v_mfma_f32_16x16x32_f16 a[124:127], v[106:109], v[234:237], a[124:127]\n\t"
      "buffer_load_dwordx4 v[202:205], %[vw3], %[srdw], s61 offen\n\t"
      "v_mfma_f32_16x16x32_f16 a[120:123], v[106:109], v[226:229], a[120:123]\n\t"
      "s_add_u32 s61, s61, 7864320\n\t"
      "v_mfma_f32_16x16x32_f16 a[120:123], v[102:105], v[222:225], a[120:123]\n\t"
      "v_mfma_f32_16x16x32_f16 a[152:155], v[110:113], v[222:225], a[152:155]\n\t"
      "v_mfma_f32_16x16x32_f16 a[152:155], v[114:117], v[226:229], a[152:155]\n\t"
      "v_mfma_f32_16x16x32_f16 a[156:159], v[114:117], v[234:237], a[156:159]\n\t"
      "v_mfma_f32_16x16x32_f16 a[156:159], v[110:113], v[230:233], a[156:159]\n\t"
      "v_mfma_f32_16x16x32_f16 a[188:191], v[118:121], v[230:233], a[188:191]\n\t"
      "v_mfma_f32_16x16x32_f16 a[188:191], v[122:125], v[234:237], a[188:191]\n\t"
      "v_mfma_f32_16x16x32_f16 a[184:187], v[122:125], v[226:229], a[184:187]\n\t"
      "v_mfma_f32_16x16x32_f16 a[184:187], v[118:121], v[222:225], a[184:187]\n\t"
      "v_mfma_f32_16x16x32_f16 a[216:219], v[126:129], v[222:225], a[216:219]\n\t"
      "v_mfma_f32_16x16x32_f16 a[216:219], v[130:133], v[226:229], a[216:219]\n\t"
      "v_mfma_f32_16x16x32_f16 a[220:223], v[130:133], v[234:237], a[220:223]\n\t"
      "s_add_u32 s62, s62, 1\n\t"
      "v_mfma_f32_16x16x32_f16 a[220:223], v[126:129], v[230:233], a[220:223]\n\t"
      "v_mfma_f32_16x16x32_f16 a[252:255], v[134:137], v[230:233], a[252:255]\n\t"
      "v_mfma_f32_16x16x32_f16 a[252:255], v[138:141], v[234:237], a[252:255]\n\t"
      "v_mfma_f32_16x16x32_f16 a[248:251], v[138:141], v[226:229], a[248:251]\n\t"
      "v_mfma_f32_16x16x32_f16 a[248:251], v[134:137], v[222:225], a[248:251]\n\t"
      "s_branch LloopH_%=\n\t"
      "LexitH_%=:\n\t"
      "s_waitcnt vmcnt(17)\n\t"
      "s_branch LepiH_%=\n\t"
      "Lpush0H_%=:\n\t"
      "s_cmp_lg_u32 s91, 0\n\t"
      "s_cbranch_scc1 Lpq0H_%=\n\t"
      "s_add_u32 m0, %[sxst], 0\n\t"
      "s_add_u32 s84, %[sxr], 512\n\t"
      "buffer_load_dwordx4 %[vxs], %[srdxs], s84 offen nt lds\n\t"
      "s_add_u32 m0, m0, 1024\n\t"
      "s_add_u32 s84, %[sxr], 528\n\t"
      "buffer_load_dwordx4 %[vxs], %[srdxs], s84 offen nt lds\n\t"
      "s_add_u32 m0, m0, 1024\n\t"
      "s_branch Lpushr0H_%=\n\t"
      "Lpq0H_%=:\n\t"
      "v_and_b32 v250, 0x3ff, %[vlr]\n\t"
      "v_add_u32 v250, %[sxst], v250\n\t"
      "ds_read_b128 v[242:245], v250 offset:0\n\t"
      "ds_read_b128 v[246:249], v250 offset:1024\n\t"
      "s_waitcnt lgkmcnt(0)\n\t"
      "v_cvt_pk_f16_f32 v250, v242, v243\n\t"
      "v_cvt_pk_f16_f32 v251, v244, v245\n\t"
      "v_cvt_pk_f16_f32 v252, v246, v247\n\t"
      "v_cvt_pk_f16_f32 v253, v248, v249\n\t"
      "global_store_dwordx4 %[vpd], v[250:253], s[76:77] sc0 sc1\n\t"
      "s_add_u32 s76, s76, 262144\n\t"
      "s_addc_u32 s77, s77, 0\n\t"
      "s_branch Lpushr0H_%=\n\t"
      "Lpush1H_%=:\n\t"
      "s_cmp_lg_u32 s91, 0\n\t"
      "s_cbranch_scc1 Lpq1H_%=\n\t"
      "s_add_u32 m0, %[sxst], 2048\n\t"
      "s_add_u32 s84, %[sxr], 1024\n\t"
      "buffer_load_dwordx4 %[vxs], %[srdxs], s84 offen nt lds\n\t"
      "s_add_u32 m0, m0, 1024\n\t"
      "s_add_u32 s84, %[sxr], 1040\n\t"
      "buffer_load_dwordx4 %[vxs], %[srdxs], s84 offen nt lds\n\t"
      "s_add_u32 m0, m0, 1024\n\t"
      "s_branch Lpushr1H_%=\n\t"
      "Lpq1H_%=:\n\t"
      "v_and_b32 v250, 0x3ff, %[vlr]\n\t"
      "v_add_u32 v250, %[sxst], v250\n\t"
      "ds_read_b128 v[242:245], v250 offset:2048\n\t"
      "ds_read_b128 v[246:249], v250 offset:3072\n\t"
      "s_waitcnt lgkmcnt(0)\n\t"
      "v_cvt_pk_f16_f32 v250, v242, v243\n\t"
      "v_cvt_pk_f16_f32 v251, v244, v245\n\t"
      "v_cvt_pk_f16_f32 v252, v246, v247\n\t"
      "v_cvt_pk_f16_f32 v253, v248, v249\n\t"
      "global_store_dwordx4 %[vpd], v[250:253], s[76:77] sc0 sc1\n\t"
      "s_add_u32 s76, s76, 262144\n\t"
      "s_addc_u32 s77, s77, 0\n\t"
      "s_branch Lpushr1H_%=\n\t"
      "Lpush2H_%=:\n\t"
      "s_cmp_lg_u32 s91, 0\n\t"
      "s_cbranch_scc1 Lpq2H_%=\n\t"
      "s_add_u32 m0, %[sxst], 4096\n\t"
      "s_add_u32 s84, %[sxr], 1536\n\t"
      "buffer_load_dwordx4 %[vxs], %[srdxs], s84 offen nt lds\n\t"
      "s_add_u32 m0, m0, 1024\n\t"
      "s_add_u32 s84, %[sxr], 1552\n\t"
      "buffer_load_dwordx4 %[vxs], %[srdxs], s84 offen nt lds\n\t"
      "s_add_u32 m0, m0, 1024\n\t"
      "s_mov_b32 s91, 1\n\t"
      "s_add_u32 s66, s66, 1\n\t"
      "s_branch Lpushr2H_%=\n\t"
      "Lpq2H_%=:\n\t"
      "v_and_b32 v250, 0x3ff, %[vlr]\n\t"
      "v_add_u32 v250, %[sxst], v250\n\t"
      "ds_read_b128 v[242:245], v250 offset:4096\n\t"
      "ds_read_b128 v[246:249], v250 offset:5120\n\t"
      "s_waitcnt lgkmcnt(0)\n\t"
      "v_cvt_pk_f16_f32 v250, v242, v243\n\t"
      "v_cvt_pk_f16_f32 v251, v244, v245\n\t"
      "v_cvt_pk_f16_f32 v252, v246, v247\n\t"
      "v_cvt_pk_f16_f32 v253, v248, v249\n\t"
      "global_store_dwordx4 %[vpd], v[250:253], s[76:77] sc0 sc1\n\t"
      "s_add_u32 s76, s76, 262144\n\t"
      "s_addc_u32 s77, s77, 0\n\t"
      "s_branch Lpushr2H_%=\n\t"
      "LsigH_%=:\n\t"
      "s_call_b64 s[82:83], Lsigf_%=\n\t"
      "s_branch LsigrH_%=\n\t"
      "LslowH_%=:\n\t"
      "s_call_b64 s[82:83], Lpoll_%=\n\t"
      "s_waitcnt vmcnt(0)\n\t"
      "s_branch LslowrH_%=\n\t"
      "Lsigf_%=:\n\t"
      "v_mov_b32 v255, 1\n\t"
      "v_mov_b32 v254, 0\n\t"
      "s_mov_b64 exec, 1\n\t"
      "s_cmp_eq_u32 s72, 0\n\t"
      "s_cselect_b32 s81, 29, 29\n\t"
      "s_cmp_eq_u32 s62, s81\n\t"
      "s_cbranch_scc1 Lsig2_%=\n\t"
      "s_cmp_gt_u32 s62, s81\n\t"
      "s_cbranch_scc1 Lsig4_%=\n\t"
      "s_cmp_eq_u32 s62, 0\n\t"
      "s_cbranch_scc1 Lsig1_%=\n\t"
      "s_cmp_eq_u32 s62, 1\n\t"
      "s_cbranch_scc1 Lsig5_%=\n\t"
      "global_atomic_add v254, v255, %[sflag] offset:1024 sc1\n\t"
      "s_mov_b32 s85, s81\n\t"
      "s_branch Lsig3_%=\n\t"
      "Lsig1_%=:\n\t"
      "global_atomic_add v254, v255, %[sflag] sc1\n\t"
      "s_add_u32 s85, s66, 2\n\t"
      "s_lshr_b32 s63, %[sseq], 10\n\t"
      "s_and_b32 s63, s63, 25\n\t"
      "s_xor_b32 s63, s63, 1\n\t"
      "s_or_b32 s63, s63, s72\n\t"
      "s_cbranch_scc1 Lsig3_%=\n\t"
      "s_mov_b32 s51, s85\n\t"
      "s_mov_b32 s85, 1\n\t"
      "s_memtime s[80:81]\n\t"
      "s_memrealtime s[78:79]\n\t"
      "s_waitcnt lgkmcnt(0)\n\t"
      "s_mov_b32 s58, s80\n\t"
      "s_mov_b32 s53, s78\n\t"
      "s_branch Lsig3_%=\n\t"
      "Lsig5_%=:\n\t"
      "s_mov_b32 s85, s51\n\t"
      "s_memtime s[80:81]\n\t"
      "s_memrealtime s[78:79]\n\t"
      "s_waitcnt lgkmcnt(0)\n\t"
      "s_sub_u32 s58, s80, s58\n\t"
      "s_sub_u32 s53, s78, s53\n\t"
      "s_sub_u32 s63, s53, 345\n\t"
      "s_cmp_gt_u32 s53, 430\n\t"
      "s_cselect_b32 s63, s63, 0\n\t"
      "s_sub_u32 s53, s53, s63\n\t"
      "s_mul_i32 s63, s63, 250\n\t"
      "s_mul_i32 s58, s58, 10\n\t"
      "s_sub_u32 s58, s58, s63\n\t"
      "v_cvt_f32_u32 v255, s53\n\t"
      "v_cvt_f32_u32 v254, s58\n\t"
      "v_rcp_f32 v255, v255\n\t"
      "s_nop 1\n\t"
      "v_mul_f32 v255, v255, v254\n\t"
      "v_add_f32 v255, 0.5, v255\n\t"
      "v_cvt_u32_f32 v255, v255\n\t"
      "v_min_u32 v255, 0xff, v255\n\t"
      "v_add_u32 v255, 0x10000, v255\n\t"
      "s_and_b32 s63, s74, 1\n\t"
      "s_lshl_b32 s63, s63, 7\n\t"
      "s_add_u32 s63, s63, 37120\n\t"
      "v_mov_b32 v254, s63\n\t"
      "global_atomic_add v254, v255, %[sws] sc1\n\t"
      "s_branch Lsig3_%=\n\t"
      "Lsig2_%=:\n\t"
      "v_mov_b32 v254, %[sfin]\n\t"
      "global_atomic_add v242, v254, v255, %[sws] sc0 sc1\n\t"
      "s_add_u32 s85, s62, 1\n\t"
      "s_mov_b64 exec, -1\n\t"
      "s_getpc_b64 s[80:81]\n\t"
      "Lpc_%=:\n\t"
      "s_mul_i32 s63, s72, LepiH_%=-LepiC_%=\n\t"
      "s_add_u32 s63, s63, LepiC_%=-Lpc_%=\n\t"
      "s_add_u32 s80, s80, s63\n\t"
      "s_addc_u32 s81, s81, 0\n\t"
      "v_mbcnt_lo_u32_b32 v254, -1, 0\n\t"
      "v_mbcnt_hi_u32_b32 v254, -1, v254\n\t"
      "v_mul_u32_u24 v254, 104, v254\n\t"
      "global_load_dword v243, v254, s[80:81]\n\t"
      "s_cmp_lg_u32 s72, 0\n\t"
      "s_cbranch_scc1 Lsig3_%=\n\t"
      "v_add_u32 v254, 6656, v254\n\t"
      "global_load_dword v243, v254, s[80:81]\n\t"
      "s_branch Lsig3_%=\n\t"
      "Lsig4_%=:\n\t"
      "v_readfirstlane_b32 s63, v242\n\t"
      "s_lshl_b32 s91, s74, 5\n\t"
      "s_sub_u32 s91, s91, 1\n\t"
      "s_nop 2\n\t"
      "s_cmp_lg_u32 s63, s91\n\t"
      "s_cbranch_scc1 Lsig3_%=\n\t"
      "v_mov_b32 v254, %[sfin]\n\t"
      "v_mov_b32 v255, s74\n\t"
      "global_store_dword v254, v255, %[sws] offset:128 sc1\n\t"
      "Lsig3_%=:\n\t"
      "s_mov_b64 exec, -1\n\t"
      "s_nop 3\n\t"
      "s_setpc_b64 s[82:83]\n\t"
      "Lpoll_%=:\n\t"
      "s_cmp_lg_u32 s75, 0\n\t"
      "s_cbranch_scc1 Lpret_%=\n\t"
      "s_memrealtime s[78:79]\n\t"
      "Lspin_%=:\n\t"
      "buffer_load_dword v254, off, %[srdws], s67 sc0 sc1\n\t"
      "s_mov_b32 s84, 36864\n\t"
      "buffer_load_dword v255, off, %[srdws], s84 sc1\n\t"
      "s_memrealtime s[80:81]\n\t"
      "s_waitcnt vmcnt(0) lgkmcnt(0)\n\t"
      "v_readfirstlane_b32 s63, v254\n\t"
      "v_readfirstlane_b32 s81, v255\n\t"
      "s_nop 3\n\t"
      "s_cmp_ge_u32 s63, s73\n\t"
      "s_cbranch_scc1 Lpret_%=\n\t"
      "s_cmp_eq_u32 s81, s74\n\t"
      "s_cbranch_scc1 Lpbad_%=\n\t"
      "s_sub_u32 s81, s80, s78\n\t"
      "s_cmp_gt_u32 s81, 50000000\n\t"
      "s_cbranch_scc1 Lpbad_%=\n\t"
      "s_cmp_gt_u32 s81, 6000\n\t"
      "s_cbranch_scc1 Lspz_%=\n\t"
      "s_cmp_lg_u32 s72, 0\n\t"
      "s_cbranch_scc1 Lspz_%=\n\t"
      "s_mov_b32 s81, 8\n\t"
      "Lspb_%=:\n\t"
      "v_mfma_f32_16x16x32_f16 v[238:241], v[14:17], v[222:225], 0\n\t"
      "v_mfma_f32_16x16x32_f16 v[238:241], v[22:25], v[226:229], 0\n\t"
      "v_mfma_f32_16x16x32_f16 v[238:241], v[30:33], v[230:233], 0\n\t"
      "v_mfma_f32_16x16x32_f16 v[238:241], v[38:41], v[234:237], 0\n\t"
      "v_mfma_f32_16x16x32_f16 v[238:241], v[46:49], v[222:225], 0\n\t"
      "v_mfma_f32_16x16x32_f16 v[238:241], v[54:57], v[226:229], 0\n\t"
      "v_mfma_f32_16x16x32_f16 v[238:241], v[62:65], v[230:233], 0\n\t"
      "v_mfma_f32_16x16x32_f16 v[238:241], v[70:73], v[234:237], 0\n\t"
      "s_sub_u32 s81, s81, 1\n\t"
      "s_cmp_lg_u32 s81, 0\n\t"
      "s_cbranch_scc1 Lspb_%=\n\t"
      "s_branch Lspin_%=\n\t"
      "Lspz_%=:\n\t"
      "s_sleep 16\n\t"
      "s_branch Lspin_%=\n\t"
      "Lpbad_%=:\n\t"
      "s_mov_b32 s75, 1\n\t"
      "v_mov_b32 v255, s74\n\t"
      "buffer_store_dword v255, off, %[srdws], s84 sc1\n\t"
      "s_waitcnt vmcnt(0)\n\t"
      "Lpret_%=:\n\t"
      "s_setpc_b64 s[82:83]\n\t"
      ".p2align 6\n\t"
      "LepiC_%=:\n\t"
      "s_mov_b32 s61, s60\n\t"
      "s_lshr_b32 s64, s64, 6\n\t"
      "v_mfma_f32_16x16x32_f16 a[24:27], v[78:81], v[222:225], a[24:27]\n\t"
      "v_cvt_pk_f16_f32 v238, v142, v146\n\t"
      "v_mfma_f32_16x16x32_f16 a[24:27], v[82:85], v[226:229], a[24:27]\n\t"
      "s_add_u32 s60, s61, 0\n\t"
      "v_cvt_pk_f16_f32 v239, v150, v154\n\t"
      "v_mfma_f32_16x16x32_f16 a[28:31], v[82:85], v[234:237], a[28:31]\n\t"
      "v_cvt_pk_f16_f32 v240, v158, v162\n\t"
      "v_mfma_f32_16x16x32_f16 a[28:31], v[78:81], v[230:233], a[28:31]\n\t"
      "v_cvt_pk_f16_f32 v241, v166, v170\n\t"
      "v_mfma_f32_16x16x32_f16 a[60:63], v[86:89], v[230:233], a[60:63]\n\t"
      "buffer_load_dwordx4 v[78:81], %[vx0], s[68:71], s60 offen offset:0 sc0 sc1\n\t"
      "ds_write_b128 %[vlw], v[238:241] offset:16384\n\t"
      "v_mfma_f32_16x16x32_f16 a[60:63], v[90:93], v[234:237], a[60:63]\n\t"
      "v_cvt_pk_f16_f32 v238, v143, v147\n\t"
      "v_mfma_f32_16x16x32_f16 a[56:59], v[90:93], v[226:229], a[56:59]\n\t"
      "v_cvt_pk_f16_f32 v239, v151, v155\n\t"
      "v_mfma_f32_16x16x32_f16 a[56:59], v[86:89], v[222:225], a[56:59]\n\t"
      "buffer_load_dwordx4 v[82:85], %[vx0], s[68:71], s60 offen offset:1024 sc0 sc1\n\t"
      "v_cvt_pk_f16_f32 v240, v159, v163\n\t"
      "v_mfma_f32_16x16x32_f16 a[88:91], v[94:97], v[222:225], a[88:91]\n\t"
      "v_cvt_pk_f16_f32 v241, v167, v171\n\t"
      "v_mfma_f32_16x16x32_f16 a[88:91], v[98:101], v[226:229], a[88:91]\n\t"
      "ds_write_b128 %[vlw], v[238:241] offset:18432\n\t"
      "v_mfma_f32_16x16x32_f16 a[92:95], v[98:101], v[234:237], a[92:95]\n\t"
      "buffer_load_dwordx4 v[86:89], %[vx0], s[68:71], s60 offen offset:2048 sc0 sc1\n\t"
      "v_cvt_pk_f16_f32 v238, v144, v148\n\t"
      "v_mfma_f32_16x16x32_f16 a[92:95], v[94:97], v[230:233], a[92:95]\n\t"
      "v_cvt_pk_f16_f32 v239, v152, v156\n\t"
      "s_waitcnt vmcnt(3)\n\t"
      "v_mfma_f32_16x16x32_f16 a[124:127], v[102:105], v[230:233], a[124:127]\n\t"
      "v_cvt_pk_f16_f32 v240, v160, v164\n\t"
      "v_mfma_f32_16x16x32_f16 a[124:127], v[106:109], v[234:237], a[124:127]\n\t"
      "buffer_load_dwordx4 v[90:93], %[vx0], s[68:71], s60 offen offset:3072 sc0 sc1\n\t"
      "s_add_u32 s60, s64, 0\n\t"
      "v_mfma_f32_16x16x32_f16 a[120:123], v[106:109], v[226:229], a[120:123]\n\t"
      "v_cvt_pk_f16_f32 v241, v168, v172\n\t"
      "v_mfma_f32_16x16x32_f16 a[120:123], v[102:105], v[222:225], a[120:123]\n\t"
      "ds_write_b128 %[vlw], v[238:241] offset:20480\n\t"
      "v_mfma_f32_16x16x32_f16 a[152:155], v[110:113], v[222:225], a[152:155]\n\t"
      "buffer_load_dwordx4 v[94:97], %[vx0], s[68:71], s60 offen offset:0 sc0 sc1\n\t"
      "v_cvt_pk_f16_f32 v238, v145, v149\n\t"
      "v_mfma_f32_16x16x32_f16 a[152:155], v[114:117], v[226:229], a[152:155]\n\t"
      "v_cvt_pk_f16_f32 v239, v153, v157\n\t"
      "v_mfma_f32_16x16x32_f16 a[156:159], v[114:117], v[234:237], a[156:159]\n\t"
      "v_cvt_pk_f16_f32 v240, v161, v165\n\t"
      "v_mfma_f32_16x16x32_f16 a[156:159], v[110:113], v[230:233], a[156:159]\n\t"
      "buffer_load_dwordx4 v[98:101], %[vx0], s[68:71], s60 offen offset:1024 sc0 sc1\n\t"
      "v_cvt_pk_f16_f32 v241, v169, v173\n\t"
      "v_mfma_f32_16x16x32_f16 a[188:191], v[118:121], v[230:233], a[188:191]\n\t"
      "ds_write_b128 %[vlw], v[238:241] offset:22528\n\t"
      "v_mfma_f32_16x16x32_f16 a[188:191], v[122:125], v[234:237], a[188:191]\n\t"
      "ds_read_b128 v[206:209], %[vlr] offset:0\n\t"
      "v_mfma_f32_16x16x32_f16 a[184:187], v[122:125], v[226:229], a[184:187]\n\t"
      "buffer_load_dwordx4 v[102:105], %[vx0], s[68:71], s60 offen offset:2048 sc0 sc1\n\t"
      "v_mfma_f32_16x16x32_f16 a[184:187], v[118:121], v[222:225], a[184:187]\n\t"
      "ds_read_b128 v[210:213], %[vlr] offset:1024\n\t"
      "v_mfma_f32_16x16x32_f16 a[216:219], v[126:129], v[222:225], a[216:219]\n\t"
      "ds_read_b128 v[214:217], %[vlr] offset:2048\n\t"
      "v_mfma_f32_16x16x32_f16 a[216:219], v[130:133], v[226:229], a[216:219]\n\t"
      "buffer_load_dwordx4 v[106:109], %[vx0], s[68:71], s60 offen offset:3072 sc0 sc1\n\t"
      "s_add_u32 s60, s64, 131072\n\t"
      "v_mfma_f32_16x16x32_f16 a[220:223], v[130:133], v[234:237], a[220:223]\n\t"
      "ds_read_b128 v[218:221], %[vlr] offset:3072\n\t"
      "v_mfma_f32_16x16x32_f16 a[220:223], v[126:129], v[230:233], a[220:223]\n\t"
      "v_mfma_f32_16x16x32_f16 a[252:255], v[134:137], v[230:233], a[252:255]\n\t"
      "buffer_load_dwordx4 v[110:113], %[vx0], s[68:71], s60 offen offset:0 sc0 sc1\n\t"
      "v_mfma_f32_16x16x32_f16 a[252:255], v[138:141], v[234:237], a[252:255]\n\t"
      "v_mfma_f32_16x16x32_f16 a[248:251], v[138:141], v[226:229], a[248:251]\n\t"
      "s_waitcnt lgkmcnt(0)\n\t"
      "s_barrier\n\t"
      "v_mfma_f32_16x16x32_f16 a[248:251], v[134:137], v[222:225], a[248:251]\n\t"
      "buffer_load_dwordx4 v[114:117], %[vx0], s[68:71], s60 offen offset:1024 sc0 sc1\n\t"
      "s_waitcnt lgkmcnt(0)\n\t"
      "v_mfma_f32_16x16x32_f16 a[0:3], v[14:17], v[206:209], a[0:3]\n\t"
      "v_mfma_f32_16x16x32_f16 a[0:3], v[18:21], v[210:213], a[0:3]\n\t"
      "v_mfma_f32_16x16x32_f16 a[4:7], v[18:21], v[218:221], a[4:7]\n\t"
      "buffer_load_dwordx4 v[118:121], %[vx0], s[68:71], s60 offen offset:2048 sc0 sc1\n\t"
      "v_add_u32 v246, 0x10000, %[vlr]\n\t"
      "v_mfma_f32_16x16x32_f16 a[4:7], v[14:17], v[214:217], a[4:7]\n\t"
      "v_bfe_u32 v244, %[vlr], 8, 2\n\t"
      "v_mfma_f32_16x16x32_f16 a[36:39], v[22:25], v[214:217], a[36:39]\n\t"
      "v_lshlrev_b32 v244, 19, v244\n\t"
      "ds_read_b128 v[174:177], %[vlr] offset:4096\n\t"
      "v_mfma_f32_16x16x32_f16 a[36:39], v[26:29], v[218:221], a[36:39]\n\t"
      "buffer_load_dwordx4 v[122:125], %[vx0], s[68:71], s60 offen offset:3072 sc0 sc1\n\t"
      "s_add_u32 s60, s61, 4096\n\t"
      "v_mfma_f32_16x16x32_f16 a[32:35], v[26:29], v[210:213], a[32:35]\n\t"
      "v_and_b32 v245, 0xff, %[vlr]\n\t"
      "ds_read_b128 v[178:181], %[vlr] offset:5120\n\t"
      "v_mfma_f32_16x16x32_f16 a[32:35], v[22:25], v[206:209], a[32:35]\n\t"
      "v_add3_u32 v244, v244, v245, %[sob]\n\t"
      "ds_read_b128 v[182:185], %[vlr] offset:6144\n\t"
      "v_mfma_f32_16x16x32_f16 a[64:67], v[30:33], v[206:209], a[64:67]\n\t"
      "buffer_load_dwordx4 v[126:129], %[vx0], s[68:71], s60 offen offset:0 sc0 sc1\n\t"
      "ds_read_b128 v[186:189], %[vlr] offset:7168\n\t"
      "v_mfma_f32_16x16x32_f16 a[64:67], v[34:37], v[210:213], a[64:67]\n\t"
      "v_mfma_f32_16x16x32_f16 a[68:71], v[34:37], v[218:221], a[68:71]\n\t"
      "v_mfma_f32_16x16x32_f16 a[68:71], v[30:33], v[214:217], a[68:71]\n\t"
      "buffer_load_dwordx4 v[130:133], %[vx0], s[68:71], s60 offen offset:1024 sc0 sc1\n\t"
      "v_mfma_f32_16x16x32_f16 a[100:103], v[38:41], v[214:217], a[100:103]\n\t"
      "ds_read_b128 v[190:193], %[vlr] offset:8192\n\t"
      "ds_read_b128 v[194:197], %[vlr] offset:9216\n\t"
      "v_mfma_f32_16x16x32_f16 a[100:103], v[42:45], v[218:221], a[100:103]\n\t"
      "ds_read_b128 v[198:201], %[vlr] offset:10240\n\t"
      "ds_read_b128 v[202:205], %[vlr] offset:11264\n\t"
      "v_mfma_f32_16x16x32_f16 a[96:99], v[42:45], v[210:213], a[96:99]\n\t"
      "buffer_load_dwordx4 v[134:137], %[vx0], s[68:71], s60 offen offset:2048 sc0 sc1\n\t"
      "v_mfma_f32_16x16x32_f16 a[96:99], v[38:41], v[206:209], a[96:99]\n\t"
      "v_mfma_f32_16x16x32_f16 a[128:131], v[46:49], v[206:209], a[128:131]\n\t"
      "v_mfma_f32_16x16x32_f16 a[128:131], v[50:53], v[210:213], a[128:131]\n\t"
      "buffer_load_dwordx4 v[138:141], %[vx0], s[68:71], s60 offen offset:3072 sc0 sc1\n\t"
      "s_add_u32 s60, s64, 4096\n\t"
      "v_mfma_f32_16x16x32_f16 a[132:135], v[50:53], v[218:221], a[132:135]\n\t"
      "v_mfma_f32_16x16x32_f16 a[132:135], v[46:49], v[214:217], a[132:135]\n\t"
      "v_mfma_f32_16x16x32_f16 a[164:167], v[54:57], v[214:217], a[164:167]\n\t"
      "ds_read_b128 v[222:225], %[vlr] offset:12288\n\t"
      "ds_read_b128 v[226:229], %[vlr] offset:13312\n\t"
      "v_mfma_f32_16x16x32_f16 a[164:167], v[58:61], v[218:221], a[164:167]\n\t"
      "ds_read_b128 v[230:233], %[vlr] offset:14336\n\t"
      "ds_read_b128 v[234:237], %[vlr] offset:15360\n\t"
      "v_mfma_f32_16x16x32_f16 a[160:163], v[58:61], v[210:213], a[160:163]\n\t"
      "v_mfma_f32_16x16x32_f16 a[160:163], v[54:57], v[206:209], a[160:163]\n\t"
      "v_mfma_f32_16x16x32_f16 a[192:195], v[62:65], v[206:209], a[192:195]\n\t"
      "v_mfma_f32_16x16x32_f16 a[192:195], v[66:69], v[210:213], a[192:195]\n\t"
      "v_mfma_f32_16x16x32_f16 a[196:199], v[66:69], v[218:221], a[196:199]\n\t"
      "v_mfma_f32_16x16x32_f16 a[196:199], v[62:65], v[214:217], a[196:199]\n\t"
      "v_mfma_f32_16x16x32_f16 a[228:231], v[70:73], v[214:217], a[228:231]\n\t"
      "v_mfma_f32_16x16x32_f16 a[228:231], v[74:77], v[218:221], a[228:231]\n\t"
      "v_mfma_f32_16x16x32_f16 a[224:227], v[74:77], v[210:213], a[224:227]\n\t"
      "v_mfma_f32_16x16x32_f16 a[224:227], v[70:73], v[206:209], a[224:227]\n\t"
      "ds_read_b128 v[206:209], %[vlr] offset:16384\n\t"
      "ds_read_b128 v[210:213], %[vlr] offset:17408\n\t"
      "s_waitcnt lgkmcnt(10)\n\t"
      "v_mfma_f32_16x16x32_f16 a[8:11], v[14:17], v[174:177], a[8:11]\n\t"
      "ds_read_b128 v[214:217], %[vlr] offset:18432\n\t"
      "ds_read_b128 v[218:221], %[vlr] offset:19456\n\t"
      "v_mfma_f32_16x16x32_f16 a[8:11], v[18:21], v[178:181], a[8:11]\n\t"
      "v_mfma_f32_16x16x32_f16 a[12:15], v[18:21], v[186:189], a[12:15]\n\t"
      "v_mfma_f32_16x16x32_f16 a[12:15], v[14:17], v[182:185], a[12:15]\n\t"
      "v_mfma_f32_16x16x32_f16 a[44:47], v[22:25], v[182:185], a[44:47]\n\t"
      "v_mfma_f32_16x16x32_f16 a[44:47], v[26:29], v[186:189], a[44:47]\n\t"
      "v_mfma_f32_16x16x32_f16 a[40:43], v[26:29], v[178:181], a[40:43]\n\t"
      "v_mfma_f32_16x16x32_f16 a[40:43], v[22:25], v[174:177], a[40:43]\n\t"
      "ds_read_b128 v[174:177], %[vlr] offset:20480\n\t"
      "ds_read_b128 v[178:181], %[vlr] offset:21504\n\t"
      "s_waitcnt lgkmcnt(10)\n\t"
      "v_mfma_f32_16x16x32_f16 a[16:19], v[14:17], v[190:193], a[16:19]\n\t"
      "ds_read_b128 v[182:185], %[vlr] offset:22528\n\t"
      "ds_read_b128 v[186:189], %[vlr] offset:23552\n\t"
      "v_mfma_f32_16x16x32_f16 a[16:19], v[18:21], v[194:197], a[16:19]\n\t"
      "v_mfma_f32_16x16x32_f16 a[20:23], v[18:21], v[202:205], a[20:23]\n\t"
      "v_mfma_f32_16x16x32_f16 a[20:23], v[14:17], v[198:201], a[20:23]\n\t"
      "v_mfma_f32_16x16x32_f16 a[52:55], v[22:25], v[198:201], a[52:55]\n\t"
      "v_mfma_f32_16x16x32_f16 a[52:55], v[26:29], v[202:205], a[52:55]\n\t"
      "v_mfma_f32_16x16x32_f16 a[48:51], v[26:29], v[194:197], a[48:51]\n\t"
      "v_mfma_f32_16x16x32_f16 a[48:51], v[22:25], v[190:193], a[48:51]\n\t"
      "ds_read_b128 v[190:193], %[vlr] offset:24576\n\t"
      "ds_read_b128 v[194:197], %[vlr] offset:25600\n\t"
      "s_waitcnt lgkmcnt(10)\n\t"
      "v_mfma_f32_16x16x32_f16 a[24:27], v[14:17], v[222:225], a[24:27]\n\t"
      "ds_read_b128 v[198:201], %[vlr] offset:26624\n\t"
      "ds_read_b128 v[202:205], %[vlr] offset:27648\n\t"
      "v_mfma_f32_16x16x32_f16 a[24:27], v[18:21], v[226:229], a[24:27]\n\t"
      "v_mfma_f32_16x16x32_f16 a[28:31], v[18:21], v[234:237], a[28:31]\n\t"
      "v_mfma_f32_16x16x32_f16 a[28:31], v[14:17], v[230:233], a[28:31]\n\t"
      "buffer_load_dwordx4 v[14:17], %[vx0], s[68:71], s60 offen offset:0 sc0 sc1\n\t"
      "v_mfma_f32_16x16x32_f16 a[60:63], v[22:25], v[230:233], a[60:63]\n\t"
      "v_mfma_f32_16x16x32_f16 a[60:63], v[26:29], v[234:237], a[60:63]\n\t"
      "v_mfma_f32_16x16x32_f16 a[56:59], v[26:29], v[226:229], a[56:59]\n\t"
      "buffer_load_dwordx4 v[18:21], %[vx0], s[68:71], s60 offen offset:1024 sc0 sc1\n\t"
      "v_mfma_f32_16x16x32_f16 a[56:59], v[22:25], v[222:225], a[56:59]\n\t"
      "ds_read_b128 v[222:225], %[vlr] offset:28672\n\t"
      "ds_read_b128 v[226:229], %[vlr] offset:29696\n\t"
      "s_waitcnt vmcnt(14) lgkmcnt(10)\n\t"
      "v_mfma_f32_16x16x32_f16 a[0:3], v[78:81], v[206:209], a[0:3]\n\t"
      "ds_read_b128 v[230:233], %[vlr] offset:30720\n\t"
      "ds_read_b128 v[234:237], %[vlr] offset:31744\n\t"
      "v_mfma_f32_16x16x32_f16 a[0:3], v[82:85], v[210:213], a[0:3]\n\t"
      "buffer_load_dwordx4 v[22:25], %[vx0], s[68:71], s60 offen offset:2048 sc0 sc1\n\t"
      "v_mfma_f32_16x16x32_f16 a[4:7], v[82:85], v[218:221], a[4:7]\n\t"
      "v_mfma_f32_16x16x32_f16 a[4:7], v[78:81], v[214:217], a[4:7]\n\t"
      "v_mfma_f32_16x16x32_f16 a[36:39], v[86:89], v[214:217], a[36:39]\n\t"
      "buffer_load_dwordx4 v[26:29], %[vx0], s[68:71], s60 offen offset:3072 sc0 sc1\n\t"
      "s_add_u32 s60, s64, 135168\n\t"
      "v_mfma_f32_16x16x32_f16 a[36:39], v[90:93], v[218:221], a[36:39]\n\t"
      "v_mfma_f32_16x16x32_f16 a[32:35], v[90:93], v[210:213], a[32:35]\n\t"
      "v_mfma_f32_16x16x32_f16 a[32:35], v[86:89], v[206:209], a[32:35]\n\t"
      "ds_read_b128 v[206:209], %[vlr] offset:32768\n\t"
      "ds_read_b128 v[210:213], %[vlr] offset:33792\n\t"
      "s_waitcnt lgkmcnt(10)\n\t"
      "v_mfma_f32_16x16x32_f16 a[8:11], v[78:81], v[174:177], a[8:11]\n\t"
      "ds_read_b128 v[214:217], %[vlr] offset:34816\n\t"
      "ds_read_b128 v[218:221], %[vlr] offset:35840\n\t"
      "v_mfma_f32_16x16x32_f16 a[8:11], v[82:85], v[178:181], a[8:11]\n\t"
      "v_mfma_f32_16x16x32_f16 a[12:15], v[82:85], v[186:189], a[12:15]\n\t"
      "v_mfma_f32_16x16x32_f16 a[12:15], v[78:81], v[182:185], a[12:15]\n\t"
      "v_mfma_f32_16x16x32_f16 a[44:47], v[86:89], v[182:185], a[44:47]\n\t"
      "v_mfma_f32_16x16x32_f16 a[44:47], v[90:93], v[186:189], a[44:47]\n\t"
      "v_mfma_f32_16x16x32_f16 a[40:43], v[90:93], v[178:181], a[40:43]\n\t"
      "v_mfma_f32_16x16x32_f16 a[40:43], v[86:89], v[174:177], a[40:43]\n\t"
      "ds_read_b128 v[174:177], %[vlr] offset:36864\n\t"
      "ds_read_b128 v[178:181], %[vlr] offset:37888\n\t"
      "s_waitcnt lgkmcnt(10)\n\t"
      "v_mfma_f32_16x16x32_f16 a[16:19], v[78:81], v[190:193], a[16:19]\n\t"
      "ds_read_b128 v[182:185], %[vlr] offset:38912\n\t"
      "ds_read_b128 v[186:189], %[vlr] offset:39936\n\t"
      "v_mfma_f32_16x16x32_f16 a[16:19], v[82:85], v[194:197], a[16:19]\n\t"
      "s_mov_b32 s67, 36864\n\t"
      "v_mfma_f32_16x16x32_f16 a[20:23], v[82:85], v[202:205], a[20:23]\n\t"
      "buffer_load_dword v254, off, %[srdws], s67 sc0 sc1\n\t"
      "v_mfma_f32_16x16x32_f16 a[20:23], v[78:81], v[198:201], a[20:23]\n\t"
      "v_mfma_f32_16x16x32_f16 a[52:55], v[86:89], v[198:201], a[52:55]\n\t"
      "v_mfma_f32_16x16x32_f16 a[52:55], v[90:93], v[202:205], a[52:55]\n\t"
      "v_mfma_f32_16x16x32_f16 a[48:51], v[90:93], v[194:197], a[48:51]\n\t"
      "v_mfma_f32_16x16x32_f16 a[48:51], v[86:89], v[190:193], a[48:51]\n\t"
      "ds_read_b128 v[190:193], %[vlr] offset:40960\n\t"
      "ds_read_b128 v[194:197], %[vlr] offset:41984\n\t"
      "s_waitcnt lgkmcnt(10)\n\t"
      "v_mfma_f32_16x16x32_f16 a[24:27], v[78:81], v[222:225], a[24:27]\n\t"
      "ds_read_b128 v[198:201], %[vlr] offset:43008\n\t"
      "ds_read_b128 v[202:205], %[vlr] offset:44032\n\t"
      "v_mfma_f32_16x16x32_f16 a[24:27], v[82:85], v[226:229], a[24:27]\n\t"
      "v_mfma_f32_16x16x32_f16 a[28:31], v[82:85], v[234:237], a[28:31]\n\t"
      "v_mfma_f32_16x16x32_f16 a[28:31], v[78:81], v[230:233], a[28:31]\n\t"
      "buffer_load_dwordx4 v[78:81], %[vx0], s[68:71], s60 offen offset:0 sc0 sc1\n\t"
      "v_mfma_f32_16x16x32_f16 a[60:63], v[86:89], v[230:233], a[60:63]\n\t"
      "v_mfma_f32_16x16x32_f16 a[60:63], v[90:93], v[234:237], a[60:63]\n\t"
      "v_mfma_f32_16x16x32_f16 a[56:59], v[90:93], v[226:229], a[56:59]\n\t"
      "buffer_load_dwordx4 v[82:85], %[vx0], s[68:71], s60 offen offset:1024 sc0 sc1\n\t"
      "v_mfma_f32_16x16x32_f16 a[56:59], v[86:89], v[222:225], a[56:59]\n\t"
      "ds_read_b128 v[222:225], %[vlr] offset:45056\n\t"
      "ds_read_b128 v[226:229], %[vlr] offset:46080\n\t"
      "s_waitcnt vmcnt(15) lgkmcnt(10)\n\t"
      "v_mfma_f32_16x16x32_f16 a[0:3], v[94:97], v[206:209], a[0:3]\n\t"
      "ds_read_b128 v[230:233], %[vlr] offset:47104\n\t"
      "ds_read_b128 v[234:237], %[vlr] offset:48128\n\t"
      "v_mfma_f32_16x16x32_f16 a[0:3], v[98:101], v[210:213], a[0:3]\n\t"
      "buffer_load_dwordx4 v[86:89], %[vx0], s[68:71], s60 offen offset:2048 sc0 sc1\n\t"
      "s_add_u32 s84, %[sfin], 128\n\t"
      "v_mfma_f32_16x16x32_f16 a[4:7], v[98:101], v[218:221], a[4:7]\n\t"
      "buffer_load_dword v255, off, %[srdws], s84 sc1\n\t"
      "v_mfma_f32_16x16x32_f16 a[4:7], v[94:97], v[214:217], a[4:7]\n\t"
      "v_mfma_f32_16x16x32_f16 a[36:39], v[102:105], v[214:217], a[36:39]\n\t"
      "buffer_load_dwordx4 v[90:93], %[vx0], s[68:71], s60 offen offset:3072 sc0 sc1\n\t"
      "s_add_u32 s60, s61, 8192\n\t"
      "v_mfma_f32_16x16x32_f16 a[36:39], v[106:109], v[218:221], a[36:39]\n\t"
      "v_mfma_f32_16x16x32_f16 a[32:35], v[106:109], v[210:213], a[32:35]\n\t"
      "v_mfma_f32_16x16x32_f16 a[32:35], v[102:105], v[206:209], a[32:35]\n\t"
      "ds_read_b128 v[206:209], %[vlr] offset:49152\n\t"
      "ds_read_b128 v[210:213], %[vlr] offset:50176\n\t"
      "s_waitcnt lgkmcnt(10)\n\t"
      "v_mfma_f32_16x16x32_f16 a[8:11], v[94:97], v[174:177], a[8:11]\n\t"
      "ds_read_b128 v[214:217], %[vlr] offset:51200\n\t"
      "ds_read_b128 v[218:221], %[vlr] offset:52224\n\t"
      "v_mfma_f32_16x16x32_f16 a[8:11], v[98:101], v[178:181], a[8:11]\n\t"
      "v_mfma_f32_16x16x32_f16 a[12:15], v[98:101], v[186:189], a[12:15]\n\t"
      "v_mfma_f32_16x16x32_f16 a[12:15], v[94:97], v[182:185], a[12:15]\n\t"
      "v_mfma_f32_16x16x32_f16 a[44:47], v[102:105], v[182:185], a[44:47]\n\t"
      "v_mfma_f32_16x16x32_f16 a[44:47], v[106:109], v[186:189], a[44:47]\n\t"
      "v_mfma_f32_16x16x32_f16 a[40:43], v[106:109], v[178:181], a[40:43]\n\t"
      "v_mfma_f32_16x16x32_f16 a[40:43], v[102:105], v[174:177], a[40:43]\n\t"
      "ds_read_b128 v[174:177], %[vlr] offset:53248\n\t"
      "ds_read_b128 v[178:181], %[vlr] offset:54272\n\t"
      "s_waitcnt lgkmcnt(10)\n\t"
      "v_mfma_f32_16x16x32_f16 a[16:19], v[94:97], v[190:193], a[16:19]\n\t"
      "ds_read_b128 v[182:185], %[vlr] offset:55296\n\t"
      "ds_read_b128 v[186:189], %[vlr] offset:56320\n\t"
      "v_mfma_f32_16x16x32_f16 a[16:19], v[98:101], v[194:197], a[16:19]\n\t"
      "v_mfma_f32_16x16x32_f16 a[20:23], v[98:101], v[202:205], a[20:23]\n\t"
      "v_mfma_f32_16x16x32_f16 a[20:23], v[94:97], v[198:201], a[20:23]\n\t"
      "v_mfma_f32_16x16x32_f16 a[52:55], v[102:105], v[198:201], a[52:55]\n\t"
      "v_mfma_f32_16x16x32_f16 a[52:55], v[106:109], v[202:205], a[52:55]\n\t"
      "v_mfma_f32_16x16x32_f16 a[48:51], v[106:109], v[194:197], a[48:51]\n\t"
      "v_mfma_f32_16x16x32_f16 a[48:51], v[102:105], v[190:193], a[48:51]\n\t"
      "ds_read_b128 v[190:193], %[vlr] offset:57344\n\t"
      "ds_read_b128 v[194:197], %[vlr] offset:58368\n\t"
      "s_waitcnt lgkmcnt(10)\n\t"
      "v_mfma_f32_16x16x32_f16 a[24:27], v[94:97], v[222:225], a[24:27]\n\t"
      "ds_read_b128 v[198:201], %[vlr] offset:59392\n\t"
      "ds_read_b128 v[202:205], %[vlr] offset:60416\n\t"
      "v_mfma_f32_16x16x32_f16 a[24:27], v[98:101], v[226:229], a[24:27]\n\t"
      "v_mfma_f32_16x16x32_f16 a[28:31], v[98:101], v[234:237], a[28:31]\n\t"
      "v_mfma_f32_16x16x32_f16 a[28:31], v[94:97], v[230:233], a[28:31]\n\t"
      "buffer_load_dwordx4 v[94:97], %[vx0], s[68:71], s60 offen offset:0 sc0 sc1\n\t"
      "v_mfma_f32_16x16x32_f16 a[60:63], v[102:105], v[230:233], a[60:63]\n\t"
      "v_mfma_f32_16x16x32_f16 a[60:63], v[106:109], v[234:237], a[60:63]\n\t"
      "v_mfma_f32_16x16x32_f16 a[56:59], v[106:109], v[226:229], a[56:59]\n\t"
      "buffer_load_dwordx4 v[98:101], %[vx0], s[68:71], s60 offen offset:1024 sc0 sc1\n\t"
      "v_mfma_f32_16x16x32_f16 a[56:59], v[102:105], v[222:225], a[56:59]\n\t"
      "ds_read_b128 v[222:225], %[vlr] offset:61440\n\t"
      "ds_read_b128 v[226:229], %[vlr] offset:62464\n\t"
      "s_waitcnt vmcnt(16) lgkmcnt(10)\n\t"
      "v_mfma_f32_16x16x32_f16 a[0:3], v[110:113], v[206:209], a[0:3]\n\t"
      "ds_read_b128 v[230:233], %[vlr] offset:63488\n\t"
      "ds_read_b128 v[234:237], %[vlr] offset:64512\n\t"
      "v_mfma_f32_16x16x32_f16 a[0:3], v[114:117], v[210:213], a[0:3]\n\t"
      "buffer_load_dwordx4 v[102:105], %[vx0], s[68:71], s60 offen offset:2048 sc0 sc1\n\t"
      "v_mfma_f32_16x16x32_f16 a[4:7], v[114:117], v[218:221], a[4:7]\n\t"
      "v_mfma_f32_16x16x32_f16 a[4:7], v[110:113], v[214:217], a[4:7]\n\t"
      "v_mfma_f32_16x16x32_f16 a[36:39], v[118:121], v[214:217], a[36:39]\n\t"
      "buffer_load_dwordx4 v[106:109], %[vx0], s[68:71], s60 offen offset:3072 sc0 sc1\n\t"
      "s_add_u32 s60, s61, 12288\n\t"
      "v_mfma_f32_16x16x32_f16 a[36:39], v[122:125], v[218:221], a[36:39]\n\t"
      "v_mfma_f32_16x16x32_f16 a[32:35], v[122:125], v[210:213], a[32:35]\n\t"
      "v_mfma_f32_16x16x32_f16 a[32:35], v[118:121], v[206:209], a[32:35]\n\t"
      "ds_read_b128 v[206:209], %[vlr] offset:4096\n\t"
      "ds_read_b128 v[210:213], %[vlr] offset:5120\n\t"
      "s_waitcnt lgkmcnt(10)\n\t"
      "v_mfma_f32_16x16x32_f16 a[8:11], v[110:113], v[174:177], a[8:11]\n\t"
      "ds_read_b128 v[214:217], %[vlr] offset:6144\n\t"
      "ds_read_b128 v[218:221], %[vlr] offset:7168\n\t"
      "v_mfma_f32_16x16x32_f16 a[8:11], v[114:117], v[178:181], a[8:11]\n\t"
      "v_mfma_f32_16x16x32_f16 a[12:15], v[114:117], v[186:189], a[12:15]\n\t"
      "v_mfma_f32_16x16x32_f16 a[12:15], v[110:113], v[182:185], a[12:15]\n\t"
      "v_mfma_f32_16x16x32_f16 a[44:47], v[118:121], v[182:185], a[44:47]\n\t"
      "s_waitcnt vmcnt(5)\n\t"
      "v_mfma_f32_16x16x32_f16 a[44:47], v[122:125], v[186:189], a[44:47]\n\t"
      "v_readfirstlane_b32 s63, v254\n\t"
      "v_readfirstlane_b32 s91, v255\n\t"
      "v_mfma_f32_16x16x32_f16 a[40:43], v[122:125], v[178:181], a[40:43]\n\t"
      "s_cmp_eq_u32 s63, s74\n\t"
      "s_cselect_b32 s75, 1, s75\n\t"
      "v_mfma_f32_16x16x32_f16 a[40:43], v[118:121], v[174:177], a[40:43]\n\t"
      "ds_read_b128 v[174:177], %[vlr] offset:8192\n\t"
      "ds_read_b128 v[178:181], %[vlr] offset:9216\n\t"
      "s_cmp_lg_u32 s75, 0\n\t"
      "s_cbranch_scc1 Lfbad_%=\n\t"
      "s_waitcnt lgkmcnt(10)\n\t"
      "v_mfma_f32_16x16x32_f16 a[16:19], v[110:113], v[190:193], a[16:19]\n\t"
      "ds_read_b128 v[182:185], %[vlr] offset:10240\n\t"
      "ds_read_b128 v[186:189], %[vlr] offset:11264\n\t"
      "s_cmp_lt_u32 s91, s74\n\t"
      "s_cbranch_scc1 LghC_%=\n\t"
      "Lgh2C_%=:\n\t"
      "v_mfma_f32_16x16x32_f16 a[16:19], v[114:117], v[194:197], a[16:19]\n\t"
      "v_mfma_f32_16x16x32_f16 a[20:23], v[114:117], v[202:205], a[20:23]\n\t"
      "v_mfma_f32_16x16x32_f16 a[20:23], v[110:113], v[198:201], a[20:23]\n\t"
      "v_mfma_f32_16x16x32_f16 a[52:55], v[118:121], v[198:201], a[52:55]\n\t"
      "v_accvgpr_read_b32 v142, a0\n\t"
      "v_accvgpr_read_b32 v143, a4\n\t"
      "v_mfma_f32_16x16x32_f16 a[52:55], v[122:125], v[202:205], a[52:55]\n\t"
      "v_accvgpr_read_b32 v144, a8\n\t"
      "v_accvgpr_read_b32 v145, a12\n\t"
      "v_mfma_f32_16x16x32_f16 a[48:51], v[122:125], v[194:197], a[48:51]\n\t"
      "s_mov_b32 s63, 0\n\t"
      "buffer_store_dwordx4 v[142:145], v244, %[srdo], s63 offen offset:0 sc1\n\t"
      "v_mfma_f32_16x16x32_f16 a[48:51], v[118:121], v[190:193], a[48:51]\n\t"
      "ds_read_b128 v[190:193], %[vlr] offset:12288\n\t"
      "ds_read_b128 v[194:197], %[vlr] offset:13312\n\t"
      "s_waitcnt lgkmcnt(10)\n\t"
      "v_mfma_f32_16x16x32_f16 a[24:27], v[110:113], v[222:225], a[24:27]\n\t"
      "ds_read_b128 v[198:201], %[vlr] offset:14336\n\t"
      "ds_read_b128 v[202:205], %[vlr] offset:15360\n\t"
      "s_sleep 3\n\t"
      "v_mfma_f32_16x16x32_f16 a[24:27], v[114:117], v[226:229], a[24:27]\n\t"
      "v_accvgpr_read_b32 v146, a1\n\t"
      "v_accvgpr_read_b32 v147, a5\n\t"
      "v_mfma_f32_16x16x32_f16 a[28:31], v[114:117], v[234:237], a[28:31]\n\t"
      "v_accvgpr_read_b32 v148, a9\n\t"
      "v_accvgpr_read_b32 v149, a13\n\t"
      "v_mfma_f32_16x16x32_f16 a[28:31], v[110:113], v[230:233], a[28:31]\n\t"
      "buffer_load_dwordx4 v[110:113], %[vx0], s[68:71], s60 offen offset:0 sc0 sc1\n\t"
      "s_mov_b32 s63, 131072\n\t"
      "v_mfma_f32_16x16x32_f16 a[60:63], v[118:121], v[230:233], a[60:63]\n\t"
      "buffer_store_dwordx4 v[146:149], v244, %[srdo], s63 offen offset:0 sc1\n\t"
      "v_mfma_f32_16x16x32_f16 a[60:63], v[122:125], v[234:237], a[60:63]\n\t"
      "v_mfma_f32_16x16x32_f16 a[56:59], v[122:125], v[226:229], a[56:59]\n\t"
      "buffer_load_dwordx4 v[114:117], %[vx0], s[68:71], s60 offen offset:1024 sc0 sc1\n\t"
      "v_mfma_f32_16x16x32_f16 a[56:59], v[118:121], v[222:225], a[56:59]\n\t"
      "ds_read_b128 v[222:225], %[vlr] offset:16384\n\t"
      "ds_read_b128 v[226:229], %[vlr] offset:17408\n\t"
      "s_waitcnt lgkmcnt(10)\n\t"
      "v_mfma_f32_16x16x32_f16 a[72:75], v[30:33], v[206:209], a[72:75]\n\t"
      "ds_read_b128 v[230:233], %[vlr] offset:18432\n\t"
      "ds_read_b128 v[234:237], %[vlr] offset:19456\n\t"
      "v_mfma_f32_16x16x32_f16 a[72:75], v[34:37], v[210:213], a[72:75]\n\t"
      "buffer_load_dwordx4 v[118:121], %[vx0], s[68:71], s60 offen offset:2048 sc0 sc1\n\t"
      "v_accvgpr_read_b32 v150, a2\n\t"
      "v_mfma_f32_16x16x32_f16 a[76:79], v[34:37], v[218:221], a[76:79]\n\t"
      "v_accvgpr_read_b32 v151, a6\n\t"
      "v_accvgpr_read_b32 v152, a10\n\t"
      "v_mfma_f32_16x16x32_f16 a[76:79], v[30:33], v[214:217], a[76:79]\n\t"
      "v_accvgpr_read_b32 v153, a14\n\t"
      "s_mov_b32 s63, 262144\n\t"
      "v_mfma_f32_16x16x32_f16 a[108:111], v[38:41], v[214:217], a[108:111]\n\t"
      "buffer_load_dwordx4 v[122:125], %[vx0], s[68:71], s60 offen offset:3072 sc0 sc1\n\t"
      "s_add_u32 s60, s64, 8192\n\t"
      "v_mfma_f32_16x16x32_f16 a[108:111], v[42:45], v[218:221], a[108:111]\n\t"
      "buffer_store_dwordx4 v[150:153], v244, %[srdo], s63 offen offset:0 sc1\n\t"
      "v_mfma_f32_16x16x32_f16 a[104:107], v[42:45], v[210:213], a[104:107]\n\t"
      "v_mfma_f32_16x16x32_f16 a[104:107], v[38:41], v[206:209], a[104:107]\n\t"
      "ds_read_b128 v[206:209], %[vlr] offset:20480\n\t"
      "ds_read_b128 v[210:213], %[vlr] offset:21504\n\t"
      "s_waitcnt lgkmcnt(10)\n\t"
      "v_mfma_f32_16x16x32_f16 a[80:83], v[30:33], v[174:177], a[80:83]\n\t"
      "ds_read_b128 v[214:217], %[vlr] offset:22528\n\t"
      "ds_read_b128 v[218:221], %[vlr] offset:23552\n\t"
      "v_mfma_f32_16x16x32_f16 a[80:83], v[34:37], v[178:181], a[80:83]\n\t"
      "v_accvgpr_read_b32 v154, a3\n\t"
      "v_accvgpr_read_b32 v155, a7\n\t"
      "v_mfma_f32_16x16x32_f16 a[84:87], v[34:37], v[186:189], a[84:87]\n\t"
      "v_accvgpr_read_b32 v156, a11\n\t"
      "v_accvgpr_read_b32 v157, a15\n\t"
      "v_mfma_f32_16x16x32_f16 a[84:87], v[30:33], v[182:185], a[84:87]\n\t"
      "s_mov_b32 s63, 393216\n\t"
      "buffer_store_dwordx4 v[154:157], v244, %[srdo], s63 offen offset:0 sc1\n\t"
      "v_mfma_f32_16x16x32_f16 a[116:119], v[38:41], v[182:185], a[116:119]\n\t"
      "v_mfma_f32_16x16x32_f16 a[116:119], v[42:45], v[186:189], a[116:119]\n\t"
      "v_mfma_f32_16x16x32_f16 a[112:115], v[42:45], v[178:181], a[112:115]\n\t"
      "v_accvgpr_read_b32 v158, a32\n\t"
      "v_accvgpr_read_b32 v159, a36\n\t"
      "v_mfma_f32_16x16x32_f16 a[112:115], v[38:41], v[174:177], a[112:115]\n\t"
      "ds_read_b128 v[174:177], %[vlr] offset:24576\n\t"
      "ds_read_b128 v[178:181], %[vlr] offset:25600\n\t"
      "s_waitcnt lgkmcnt(10)\n\t"
      "v_mfma_f32_16x16x32_f16 a[88:91], v[30:33], v[190:193], a[88:91]\n\t"
      "ds_read_b128 v[182:185], %[vlr] offset:26624\n\t"
      "ds_read_b128 v[186:189], %[vlr] offset:27648\n\t"
      "v_mfma_f32_16x16x32_f16 a[88:91], v[34:37], v[194:197], a[88:91]\n\t"
      "v_accvgpr_read_b32 v160, a40\n\t"
      "v_accvgpr_read_b32 v161, a44\n\t"
      "v_mfma_f32_16x16x32_f16 a[92:95], v[34:37], v[202:205], a[92:95]\n\t"
      "s_mov_b32 s63, 2097152\n\t"
      "buffer_store_dwordx4 v[158:161], v244, %[srdo], s63 offen offset:0 sc1\n\t"
      "v_mfma_f32_16x16x32_f16 a[92:95], v[30:33], v[198:201], a[92:95]\n\t"
      "buffer_load_dwordx4 v[30:33], %[vx0], s[68:71], s60 offen offset:0 sc0 sc1\n\t"
      "v_mfma_f32_16x16x32_f16 a[124:127], v[38:41], v[198:201], a[124:127]\n\t"
      "v_mfma_f32_16x16x32_f16 a[124:127], v[42:45], v[202:205], a[124:127]\n\t"
      "v_accvgpr_read_b32 v162, a33\n\t"
      "v_accvgpr_read_b32 v163, a37\n\t"
      "v_mfma_f32_16x16x32_f16 a[120:123], v[42:45], v[194:197], a[120:123]\n\t"
      "buffer_load_dwordx4 v[34:37], %[vx0], s[68:71], s60 offen offset:1024 sc0 sc1\n\t"
      "v_accvgpr_read_b32 v164, a41\n\t"
      "v_mfma_f32_16x16x32_f16 a[120:123], v[38:41], v[190:193], a[120:123]\n\t"
      "ds_read_b128 v[190:193], %[vlr] offset:28672\n\t"
      "ds_read_b128 v[194:197], %[vlr] offset:29696\n\t"
      "s_waitcnt vmcnt(25) lgkmcnt(10)\n\t"
      "v_mfma_f32_16x16x32_f16 a[64:67], v[126:129], v[222:225], a[64:67]\n\t"
      "ds_read_b128 v[198:201], %[vlr] offset:30720\n\t"
      "ds_read_b128 v[202:205], %[vlr] offset:31744\n\t"
      "s_sleep 3\n\t"
      "v_mfma_f32_16x16x32_f16 a[64:67], v[130:133], v[226:229], a[64:67]\n\t"
      "buffer_load_dwordx4 v[38:41], %[vx0], s[68:71], s60 offen offset:2048 sc0 sc1\n\t"
      "v_accvgpr_read_b32 v165, a45\n\t"
      "v_mfma_f32_16x16x32_f16 a[68:71], v[130:133], v[234:237], a[68:71]\n\t"
      "s_mov_b32 s63, 2228224\n\t"
      "buffer_store_dwordx4 v[162:165], v244, %[srdo], s63 offen offset:0 sc1\n\t"
      "v_mfma_f32_16x16x32_f16 a[68:71], v[126:129], v[230:233], a[68:71]\n\t"
      "v_mfma_f32_16x16x32_f16 a[100:103], v[134:137], v[230:233], a[100:103]\n\t"
      "buffer_load_dwordx4 v[42:45], %[vx0], s[68:71], s60 offen offset:3072 sc0 sc1\n\t"
      "s_add_u32 s60, s64, 12288\n\t"
      "v_mfma_f32_16x16x32_f16 a[100:103], v[138:141], v[234:237], a[100:103]\n\t"
      "v_accvgpr_read_b32 v166, a34\n\t"
      "v_accvgpr_read_b32 v167, a38\n\t"
      "v_mfma_f32_16x16x32_f16 a[96:99], v[138:141], v[226:229], a[96:99]\n\t"
      "v_accvgpr_read_b32 v168, a42\n\t"
      "v_accvgpr_read_b32 v169, a46\n\t"
      "v_mfma_f32_16x16x32_f16 a[96:99], v[134:137], v[222:225], a[96:99]\n\t"
      "ds_read_b128 v[222:225], %[vlr] offset:32768\n\t"
      "ds_read_b128 v[226:229], %[vlr] offset:33792\n\t"
      "s_waitcnt lgkmcnt(10)\n\t"
      "v_mfma_f32_16x16x32_f16 a[72:75], v[126:129], v[206:209], a[72:75]\n\t"
      "ds_read_b128 v[230:233], %[vlr] offset:34816\n\t"
      "ds_read_b128 v[234:237], %[vlr] offset:35840\n\t"
      "v_mfma_f32_16x16x32_f16 a[72:75], v[130:133], v[210:213], a[72:75]\n\t"
      "s_mov_b32 s63, 2359296\n\t"
      "buffer_store_dwordx4 v[166:169], v244, %[srdo], s63 offen offset:0 sc1\n\t"
      "v_mfma_f32_16x16x32_f16 a[76:79], v[130:133], v[218:221], a[76:79]\n\t"
      "v_mfma_f32_16x16x32_f16 a[76:79], v[126:129], v[214:217], a[76:79]\n\t"
      "v_mfma_f32_16x16x32_f16 a[108:111], v[134:137], v[214:217], a[108:111]\n\t"
      "v_accvgpr_read_b32 v170, a35\n\t"
      "v_accvgpr_read_b32 v171, a39\n\t"
      "v_mfma_f32_16x16x32_f16 a[108:111], v[138:141], v[218:221], a[108:111]\n\t"
      "v_accvgpr_read_b32 v172, a43\n\t"
      "v_accvgpr_read_b32 v173, a47\n\t"
      "v_mfma_f32_16x16x32_f16 a[104:107], v[138:141], v[210:213], a[104:107]\n\t"
      "s_mov_b32 s63, 2490368\n\t"
      "buffer_store_dwordx4 v[170:173], v244, %[srdo], s63 offen offset:0 sc1\n\t"
      "v_mfma_f32_16x16x32_f16 a[104:107], v[134:137], v[206:209], a[104:107]\n\t"
      "ds_read_b128 v[206:209], %[vlr] offset:36864\n\t"
      "ds_read_b128 v[210:213], %[vlr] offset:37888\n\t"
      "s_waitcnt lgkmcnt(10)\n\t"
      "v_mfma_f32_16x16x32_f16 a[80:83], v[126:129], v[174:177], a[80:83]\n\t"
      "ds_read_b128 v[214:217], %[vlr] offset:38912\n\t"
      "ds_read_b128 v[218:221], %[vlr] offset:39936\n\t"
      "v_mfma_f32_16x16x32_f16 a[80:83], v[130:133], v[178:181], a[80:83]\n\t"
      "v_accvgpr_read_b32 v142, a16\n\t"
      "v_accvgpr_read_b32 v143, a20\n\t"
      "v_mfma_f32_16x16x32_f16 a[84:87], v[130:133], v[186:189], a[84:87]\n\t"
      "v_accvgpr_read_b32 v144, a24\n\t"
      "v_accvgpr_read_b32 v145, a28\n\t"
      "v_mfma_f32_16x16x32_f16 a[84:87], v[126:129], v[182:185], a[84:87]\n\t"
      "s_mov_b32 s63, 0\n\t"
      "buffer_store_dwordx4 v[142:145], v244, %[srdo], s63 offen offset:256 sc1\n\t"
      "v_mfma_f32_16x16x32_f16 a[116:119], v[134:137], v[182:185], a[116:119]\n\t"
      "v_mfma_f32_16x16x32_f16 a[116:119], v[138:141], v[186:189], a[116:119]\n\t"
      "v_mfma_f32_16x16x32_f16 a[112:115], v[138:141], v[178:181], a[112:115]\n\t"
      "v_accvgpr_read_b32 v146, a17\n\t"
      "v_accvgpr_read_b32 v147, a21\n\t"
      "v_mfma_f32_16x16x32_f16 a[112:115], v[134:137], v[174:177], a[112:115]\n\t"
      "ds_read_b128 v[174:177], %[vlr] offset:40960\n\t"
      "ds_read_b128 v[178:181], %[vlr] offset:41984\n\t"
      "s_waitcnt lgkmcnt(10)\n\t"
      "v_mfma_f32_16x16x32_f16 a[88:91], v[126:129], v[190:193], a[88:91]\n\t"
      "ds_read_b128 v[182:185], %[vlr] offset:43008\n\t"
      "ds_read_b128 v[186:189], %[vlr] offset:44032\n\t"
      "v_mfma_f32_16x16x32_f16 a[88:91], v[130:133], v[194:197], a[88:91]\n\t"
      "v_accvgpr_read_b32 v148, a25\n\t"
      "v_accvgpr_read_b32 v149, a29\n\t"
      "v_mfma_f32_16x16x32_f16 a[92:95], v[130:133], v[202:205], a[92:95]\n\t"
      "s_mov_b32 s63, 131072\n\t"
      "buffer_store_dwordx4 v[146:149], v244, %[srdo], s63 offen offset:256 sc1\n\t"
      "v_mfma_f32_16x16x32_f16 a[92:95], v[126:129], v[198:201], a[92:95]\n\t"
      "buffer_load_dwordx4 v[126:129], %[vx0], s[68:71], s60 offen offset:0 sc0 sc1\n\t"
      "v_mfma_f32_16x16x32_f16 a[124:127], v[134:137], v[198:201], a[124:127]\n\t"
      "v_mfma_f32_16x16x32_f16 a[124:127], v[138:141], v[202:205], a[124:127]\n\t"
      "v_accvgpr_read_b32 v150, a18\n\t"
      "v_accvgpr_read_b32 v151, a22\n\t"
      "v_mfma_f32_16x16x32_f16 a[120:123], v[138:141], v[194:197], a[120:123]\n\t"
      "buffer_load_dwordx4 v[130:133], %[vx0], s[68:71], s60 offen offset:1024 sc0 sc1\n\t"
      "v_accvgpr_read_b32 v152, a26\n\t"
      "v_mfma_f32_16x16x32_f16 a[120:123], v[134:137], v[190:193], a[120:123]\n\t"
      "ds_read_b128 v[190:193], %[vlr] offset:45056\n\t"
      "ds_read_b128 v[194:197], %[vlr] offset:46080\n\t"
      "s_waitcnt vmcnt(30) lgkmcnt(10)\n\t"
      "v_mfma_f32_16x16x32_f16 a[64:67], v[14:17], v[222:225], a[64:67]\n\t"
      "ds_read_b128 v[198:201], %[vlr] offset:47104\n\t"
      "ds_read_b128 v[202:205], %[vlr] offset:48128\n\t"
      "s_sleep 3\n\t"
      "v_mfma_f32_16x16x32_f16 a[64:67], v[18:21], v[226:229], a[64:67]\n\t"
      "buffer_load_dwordx4 v[134:137], %[vx0], s[68:71], s60 offen offset:2048 sc0 sc1\n\t"
      "v_accvgpr_read_b32 v153, a30\n\t"
      "v_mfma_f32_16x16x32_f16 a[68:71], v[18:21], v[234:237], a[68:71]\n\t"
      "s_mov_b32 s63, 262144\n\t"
      "buffer_store_dwordx4 v[150:153], v244, %[srdo], s63 offen offset:256 sc1\n\t"
      "v_mfma_f32_16x16x32_f16 a[68:71], v[14:17], v[230:233], a[68:71]\n\t"
      "v_mfma_f32_16x16x32_f16 a[100:103], v[22:25], v[230:233], a[100:103]\n\t"
      "buffer_load_dwordx4 v[138:141], %[vx0], s[68:71], s60 offen offset:3072 sc0 sc1\n\t"
      "s_add_u32 s60, s64, 139264\n\t"
      "v_mfma_f32_16x16x32_f16 a[100:103], v[26:29], v[234:237], a[100:103]\n\t"
      "v_accvgpr_read_b32 v154, a19\n\t"
      "v_accvgpr_read_b32 v155, a23\n\t"
      "v_mfma_f32_16x16x32_f16 a[96:99], v[26:29], v[226:229], a[96:99]\n\t"
      "v_accvgpr_read_b32 v156, a27\n\t"
      "v_accvgpr_read_b32 v157, a31\n\t"
      "v_mfma_f32_16x16x32_f16 a[96:99], v[22:25], v[222:225], a[96:99]\n\t"
      "ds_read_b128 v[222:225], %[vlr] offset:49152\n\t"
      "ds_read_b128 v[226:229], %[vlr] offset:50176\n\t"
      "s_waitcnt lgkmcnt(10)\n\t"
      "v_mfma_f32_16x16x32_f16 a[72:75], v[14:17], v[206:209], a[72:75]\n\t"
      "ds_read_b128 v[230:233], %[vlr] offset:51200\n\t"
      "ds_read_b128 v[234:237], %[vlr] offset:52224\n\t"
      "v_mfma_f32_16x16x32_f16 a[72:75], v[18:21], v[210:213], a[72:75]\n\t"
      "s_mov_b32 s63, 393216\n\t"
      "buffer_store_dwordx4 v[154:157], v244, %[srdo], s63 offen offset:256 sc1\n\t"
      "v_mfma_f32_16x16x32_f16 a[76:79], v[18:21], v[218:221], a[76:79]\n\t"
      "v_mfma_f32_16x16x32_f16 a[76:79], v[14:17], v[214:217], a[76:79]\n\t"
      "v_mfma_f32_16x16x32_f16 a[108:111], v[22:25], v[214:217], a[108:111]\n\t"
      "v_accvgpr_read_b32 v158, a48\n\t"
      "v_accvgpr_read_b32 v159, a52\n\t"
      "v_mfma_f32_16x16x32_f16 a[108:111], v[26:29], v[218:221], a[108:111]\n\t"
      "v_accvgpr_read_b32 v160, a56\n\t"
      "v_accvgpr_read_b32 v161, a60\n\t"
      "v_mfma_f32_16x16x32_f16 a[104:107], v[26:29], v[210:213], a[104:107]\n\t"
      "s_mov_b32 s63, 2097152\n\t"
      "buffer_store_dwordx4 v[158:161], v244, %[srdo], s63 offen offset:256 sc1\n\t"
      "v_mfma_f32_16x16x32_f16 a[104:107], v[22:25], v[206:209], a[104:107]\n\t"
      "ds_read_b128 v[206:209], %[vlr] offset:53248\n\t"
      "ds_read_b128 v[210:213], %[vlr] offset:54272\n\t"
      "s_waitcnt lgkmcnt(10)\n\t"
      "v_mfma_f32_16x16x32_f16 a[80:83], v[14:17], v[174:177], a[80:83]\n\t"
      "ds_read_b128 v[214:217], %[vlr] offset:55296\n\t"
      "ds_read_b128 v[218:221], %[vlr] offset:56320\n\t"
      "v_mfma_f32_16x16x32_f16 a[80:83], v[18:21], v[178:181], a[80:83]\n\t"
      "v_accvgpr_read_b32 v162, a49\n\t"
      "v_accvgpr_read_b32 v163, a53\n\t"
      "v_mfma_f32_16x16x32_f16 a[84:87], v[18:21], v[186:189], a[84:87]\n\t"
      "v_accvgpr_read_b32 v164, a57\n\t"
      "v_accvgpr_read_b32 v165, a61\n\t"
      "v_mfma_f32_16x16x32_f16 a[84:87], v[14:17], v[182:185], a[84:87]\n\t"
      "s_mov_b32 s63, 2228224\n\t"
      "buffer_store_dwordx4 v[162:165], v244, %[srdo], s63 offen offset:256 sc1\n\t"
      "v_mfma_f32_16x16x32_f16 a[116:119], v[22:25], v[182:185], a[116:119]\n\t"
      "v_mfma_f32_16x16x32_f16 a[116:119], v[26:29], v[186:189], a[116:119]\n\t"
      "v_mfma_f32_16x16x32_f16 a[112:115], v[26:29], v[178:181], a[112:115]\n\t"
      "v_accvgpr_read_b32 v166, a50\n\t"
      "v_accvgpr_read_b32 v167, a54\n\t"
      "v_mfma_f32_16x16x32_f16 a[112:115], v[22:25], v[174:177], a[112:115]\n\t"
      "ds_read_b128 v[174:177], %[vlr] offset:57344\n\t"
      "ds_read_b128 v[178:181], %[vlr] offset:58368\n\t"
      "s_waitcnt lgkmcnt(10)\n\t"
      "v_mfma_f32_16x16x32_f16 a[88:91], v[14:17], v[190:193], a[88:91]\n\t"
      "ds_read_b128 v[182:185], %[vlr] offset:59392\n\t"
      "ds_read_b128 v[186:189], %[vlr] offset:60416\n\t"
      "v_mfma_f32_16x16x32_f16 a[88:91], v[18:21], v[194:197], a[88:91]\n\t"
      "v_accvgpr_read_b32 v168, a58\n\t"
      "v_accvgpr_read_b32 v169, a62\n\t"
      "v_mfma_f32_16x16x32_f16 a[92:95], v[18:21], v[202:205], a[92:95]\n\t"
      "s_mov_b32 s63, 2359296\n\t"
      "buffer_store_dwordx4 v[166:169], v244, %[srdo], s63 offen offset:256 sc1\n\t"
      "v_mfma_f32_16x16x32_f16 a[92:95], v[14:17], v[198:201], a[92:95]\n\t"
      "buffer_load_dwordx4 v[14:17], %[vx0], s[68:71], s60 offen offset:0 sc0 sc1\n\t"
      "v_mfma_f32_16x16x32_f16 a[124:127], v[22:25], v[198:201], a[124:127]\n\t"
      "v_mfma_f32_16x16x32_f16 a[124:127], v[26:29], v[202:205], a[124:127]\n\t"
      "v_accvgpr_read_b32 v170, a51\n\t"
      "v_accvgpr_read_b32 v171, a55\n\t"
      "v_mfma_f32_16x16x32_f16 a[120:123], v[26:29], v[194:197], a[120:123]\n\t"
      "buffer_load_dwordx4 v[18:21], %[vx0], s[68:71], s60 offen offset:1024 sc0 sc1\n\t"
      "v_accvgpr_read_b32 v172, a59\n\t"
      "v_mfma_f32_16x16x32_f16 a[120:123], v[22:25], v[190:193], a[120:123]\n\t"
      "ds_read_b128 v[190:193], %[vlr] offset:61440\n\t"
      "ds_read_b128 v[194:197], %[vlr] offset:62464\n\t"
      "s_waitcnt vmcnt(33) lgkmcnt(10)\n\t"
      "v_mfma_f32_16x16x32_f16 a[64:67], v[78:81], v[222:225], a[64:67]\n\t"
      "ds_read_b128 v[198:201], %[vlr] offset:63488\n\t"
      "ds_read_b128 v[202:205], %[vlr] offset:64512\n\t"
      "s_sleep 3\n\t"
      "v_mfma_f32_16x16x32_f16 a[64:67], v[82:85], v[226:229], a[64:67]\n\t"
      "buffer_load_dwordx4 v[22:25], %[vx0], s[68:71], s60 offen offset:2048 sc0 sc1\n\t"
      "v_accvgpr_read_b32 v173, a63\n\t"
      "v_mfma_f32_16x16x32_f16 a[68:71], v[82:85], v[234:237], a[68:71]\n\t"
      "s_mov_b32 s63, 2490368\n\t"
      "buffer_store_dwordx4 v[170:173], v244, %[srdo], s63 offen offset:256 sc1\n\t"
      "v_mfma_f32_16x16x32_f16 a[68:71], v[78:81], v[230:233], a[68:71]\n\t"
      "v_mfma_f32_16x16x32_f16 a[100:103], v[86:89], v[230:233], a[100:103]\n\t"
      "buffer_load_dwordx4 v[26:29], %[vx0], s[68:71], s60 offen offset:3072 sc0 sc1\n\t"
      "s_add_u32 s60, s64, 143360\n\t"
      "v_mfma_f32_16x16x32_f16 a[100:103], v[90:93], v[234:237], a[100:103]\n\t"
      "v_mfma_f32_16x16x32_f16 a[96:99], v[90:93], v[226:229], a[96:99]\n\t"
      "v_mfma_f32_16x16x32_f16 a[96:99], v[86:89], v[222:225], a[96:99]\n\t"
      "ds_read_b128 v[222:225], %[vlr] offset:4096\n\t"
      "s_waitcnt lgkmcnt(9)\n\t"
      "v_mfma_f32_16x16x32_f16 a[72:75], v[78:81], v[206:209], a[72:75]\n\t"
      "ds_read_b128 v[226:229], %[vlr] offset:5120\n\t"
      "v_mfma_f32_16x16x32_f16 a[72:75], v[82:85], v[210:213], a[72:75]\n\t"
      "ds_read_b128 v[230:233], %[vlr] offset:6144\n\t"
      "v_mfma_f32_16x16x32_f16 a[76:79], v[82:85], v[218:221], a[76:79]\n\t"
      "ds_read_b128 v[234:237], %[vlr] offset:7168\n\t"
      "v_mfma_f32_16x16x32_f16 a[76:79], v[78:81], v[214:217], a[76:79]\n\t"
      "v_mfma_f32_16x16x32_f16 a[108:111], v[86:89], v[214:217], a[108:111]\n\t"
      "v_mfma_f32_16x16x32_f16 a[108:111], v[90:93], v[218:221], a[108:111]\n\t"
      "v_mfma_f32_16x16x32_f16 a[104:107], v[90:93], v[210:213], a[104:107]\n\t"
      "v_mfma_f32_16x16x32_f16 a[104:107], v[86:89], v[206:209], a[104:107]\n\t"
      "s_waitcnt lgkmcnt(8)\n\t"
      "v_mfma_f32_16x16x32_f16 a[80:83], v[78:81], v[174:177], a[80:83]\n\t"
      "v_mfma_f32_16x16x32_f16 a[80:83], v[82:85], v[178:181], a[80:83]\n\t"
      "v_mfma_f32_16x16x32_f16 a[84:87], v[82:85], v[186:189], a[84:87]\n\t"
      "v_mfma_f32_16x16x32_f16 a[84:87], v[78:81], v[182:185], a[84:87]\n\t"
      "v_mfma_f32_16x16x32_f16 a[116:119], v[86:89], v[182:185], a[116:119]\n\t"
      "ds_read_b128 v[206:209], %[vlr] offset:8192\n\t"
      "v_accvgpr_read_b32 v142, a64\n\t"
      "v_mfma_f32_16x16x32_f16 a[116:119], v[90:93], v[186:189], a[116:119]\n\t"
      "ds_read_b128 v[210:213], %[vlr] offset:9216\n\t"
      "v_accvgpr_read_b32 v143, a68\n\t"
      "v_mfma_f32_16x16x32_f16 a[112:115], v[90:93], v[178:181], a[112:115]\n\t"
      "ds_read_b128 v[214:217], %[vlr] offset:10240\n\t"
      "v_accvgpr_read_b32 v144, a72\n\t"
      "v_mfma_f32_16x16x32_f16 a[112:115], v[86:89], v[174:177], a[112:115]\n\t"
      "ds_read_b128 v[218:221], %[vlr] offset:11264\n\t"
      "v_accvgpr_read_b32 v145, a76\n\t"
      "s_waitcnt lgkmcnt(8)\n\t"
      "v_mfma_f32_16x16x32_f16 a[88:91], v[78:81], v[190:193], a[88:91]\n\t"
      "s_mov_b32 s63, 4194304\n\t"
      "buffer_store_dwordx4 v[142:145], v244, %[srdo], s63 offen offset:0 sc1\n\t"
      "v_mfma_f32_16x16x32_f16 a[88:91], v[82:85], v[194:197], a[88:91]\n\t"
      "v_mfma_f32_16x16x32_f16 a[92:95], v[82:85], v[202:205], a[92:95]\n\t"
      "v_mfma_f32_16x16x32_f16 a[92:95], v[78:81], v[198:201], a[92:95]\n\t"
      "buffer_load_dwordx4 v[78:81], %[vx0], s[68:71], s60 offen offset:0 sc0 sc1\n\t"
      "v_accvgpr_read_b32 v146, a65\n\t"
      "v_mfma_f32_16x16x32_f16 a[124:127], v[86:89], v[198:201], a[124:127]\n\t"
      "v_accvgpr_read_b32 v147, a69\n\t"
      "v_accvgpr_read_b32 v148, a73\n\t"
      "v_mfma_f32_16x16x32_f16 a[124:127], v[90:93], v[202:205], a[124:127]\n\t"
      "v_accvgpr_read_b32 v149, a77\n\t"
      "s_mov_b32 s63, 4325376\n\t"
      "v_mfma_f32_16x16x32_f16 a[120:123], v[90:93], v[194:197], a[120:123]\n\t"
      "buffer_load_dwordx4 v[82:85], %[vx0], s[68:71], s60 offen offset:1024 sc0 sc1\n\t"
      "buffer_store_dwordx4 v[146:149], v244, %[srdo], s63 offen offset:0 sc1\n\t"
      "v_mfma_f32_16x16x32_f16 a[120:123], v[86:89], v[190:193], a[120:123]\n\t"
      "s_waitcnt lgkmcnt(4)\n\t"
      "v_mfma_f32_16x16x32_f16 a[136:139], v[46:49], v[222:225], a[136:139]\n\t"
      "s_sleep 3\n\t"
      "v_mfma_f32_16x16x32_f16 a[136:139], v[50:53], v[226:229], a[136:139]\n\t"
      "buffer_load_dwordx4 v[86:89], %[vx0], s[68:71], s60 offen offset:2048 sc0 sc1\n\t"
      "v_accvgpr_read_b32 v150, a66\n\t"
      "v_mfma_f32_16x16x32_f16 a[140:143], v[50:53], v[234:237], a[140:143]\n\t"
      "v_accvgpr_read_b32 v151, a70\n\t"
      "v_accvgpr_read_b32 v152, a74\n\t"
      "v_mfma_f32_16x16x32_f16 a[140:143], v[46:49], v[230:233], a[140:143]\n\t"
      "v_accvgpr_read_b32 v153, a78\n\t"
      "s_mov_b32 s63, 4456448\n\t"
      "v_mfma_f32_16x16x32_f16 a[172:175], v[54:57], v[230:233], a[172:175]\n\t"
      "buffer_load_dwordx4 v[90:93], %[vx0], s[68:71], s60 offen offset:3072 sc0 sc1\n\t"
      "buffer_store_dwordx4 v[150:153], v244, %[srdo], s63 offen offset:0 sc1\n\t"
      "v_mfma_f32_16x16x32_f16 a[172:175], v[58:61], v[234:237], a[172:175]\n\t"
      "ds_read_b128 v[174:177], %[vlr] offset:12288\n\t"
      "v_mfma_f32_16x16x32_f16 a[168:171], v[58:61], v[226:229], a[168:171]\n\t"
      "ds_read_b128 v[178:181], %[vlr] offset:13312\n\t"
      "v_mfma_f32_16x16x32_f16 a[168:171], v[54:57], v[222:225], a[168:171]\n\t"
      "ds_read_b128 v[182:185], %[vlr] offset:14336\n\t"
      "v_accvgpr_read_b32 v154, a67\n\t"
      "v_mfma_f32_16x16x32_f16 a[200:203], v[62:65], v[222:225], a[200:203]\n\t"
      "ds_read_b128 v[186:189], %[vlr] offset:15360\n\t"
      "v_accvgpr_read_b32 v155, a71\n\t"
      "v_mfma_f32_16x16x32_f16 a[200:203], v[66:69], v[226:229], a[200:203]\n\t"
      "v_accvgpr_read_b32 v156, a75\n\t"
      "v_accvgpr_read_b32 v157, a79\n\t"
      "v_mfma_f32_16x16x32_f16 a[204:207], v[66:69], v[234:237], a[204:207]\n\t"
      "s_mov_b32 s63, 4587520\n\t"
      "buffer_store_dwordx4 v[154:157], v244, %[srdo], s63 offen offset:0 sc1\n\t"
      "v_mfma_f32_16x16x32_f16 a[204:207], v[62:65], v[230:233], a[204:207]\n\t"
      "v_mfma_f32_16x16x32_f16 a[236:239], v[70:73], v[230:233], a[236:239]\n\t"
      "v_mfma_f32_16x16x32_f16 a[236:239], v[74:77], v[234:237], a[236:239]\n\t"
      "v_accvgpr_read_b32 v158, a96\n\t"
      "v_accvgpr_read_b32 v159, a100\n\t"
      "v_mfma_f32_16x16x32_f16 a[232:235], v[74:77], v[226:229], a[232:235]\n\t"
      "v_accvgpr_read_b32 v160, a104\n\t"
      "v_accvgpr_read_b32 v161, a108\n\t"
      "v_mfma_f32_16x16x32_f16 a[232:235], v[70:73], v[222:225], a[232:235]\n\t"
      "s_mov_b32 s63, 6291456\n\t"
      "buffer_store_dwordx4 v[158:161], v244, %[srdo], s63 offen offset:0 sc1\n\t"
      "s_waitcnt lgkmcnt(4)\n\t"
      "v_mfma_f32_16x16x32_f16 a[144:147], v[46:49], v[206:209], a[144:147]\n\t"
      "v_mfma_f32_16x16x32_f16 a[144:147], v[50:53], v[210:213], a[144:147]\n\t"
      "v_mfma_f32_16x16x32_f16 a[148:151], v[50:53], v[218:221], a[148:151]\n\t"
      "v_accvgpr_read_b32 v162, a97\n\t"
      "v_accvgpr_read_b32 v163, a101\n\t"
      "v_mfma_f32_16x16x32_f16 a[148:151], v[46:49], v[214:217], a[148:151]\n\t"
      "v_accvgpr_read_b32 v164, a105\n\t"
      "v_accvgpr_read_b32 v165, a109\n\t"
      "v_mfma_f32_16x16x32_f16 a[180:183], v[54:57], v[214:217], a[180:183]\n\t"
      "ds_read_b128 v[190:193], %[vlr] offset:16384\n\t"
      "s_mov_b32 s63, 6422528\n\t"
      "v_mfma_f32_16x16x32_f16 a[180:183], v[58:61], v[218:221], a[180:183]\n\t"
      "ds_read_b128 v[194:197], %[vlr] offset:17408\n\t"
      "buffer_store_dwordx4 v[162:165], v244, %[srdo], s63 offen offset:0 sc1\n\t"
      "v_mfma_f32_16x16x32_f16 a[176:179], v[58:61], v[210:213], a[176:179]\n\t"
      "ds_read_b128 v[198:201], %[vlr] offset:18432\n\t"
      "v_mfma_f32_16x16x32_f16 a[176:179], v[54:57], v[206:209], a[176:179]\n\t"
      "ds_read_b128 v[202:205], %[vlr] offset:19456\n\t"
      "v_mfma_f32_16x16x32_f16 a[208:211], v[62:65], v[206:209], a[208:211]\n\t"
      "v_accvgpr_read_b32 v166, a98\n\t"
      "v_accvgpr_read_b32 v167, a102\n\t"
      "v_mfma_f32_16x16x32_f16 a[208:211], v[66:69], v[210:213], a[208:211]\n\t"
      "v_accvgpr_read_b32 v168, a106\n\t"
      "v_accvgpr_read_b32 v169, a110\n\t"
      "v_mfma_f32_16x16x32_f16 a[212:215], v[66:69], v[218:221], a[212:215]\n\t"
      "s_mov_b32 s63, 6553600\n\t"
      "buffer_store_dwordx4 v[166:169], v244, %[srdo], s63 offen offset:0 sc1\n\t"
      "v_mfma_f32_16x16x32_f16 a[212:215], v[62:65], v[214:217], a[212:215]\n\t"
      "v_mfma_f32_16x16x32_f16 a[244:247], v[70:73], v[214:217], a[244:247]\n\t"
      "v_mfma_f32_16x16x32_f16 a[244:247], v[74:77], v[218:221], a[244:247]\n\t"
      "v_accvgpr_read_b32 v170, a99\n\t"
      "v_accvgpr_read_b32 v171, a103\n\t"
      "v_mfma_f32_16x16x32_f16 a[240:243], v[74:77], v[210:213], a[240:243]\n\t"
      "v_accvgpr_read_b32 v172, a107\n\t"
      "v_accvgpr_read_b32 v173, a111\n\t"
      "v_mfma_f32_16x16x32_f16 a[240:243], v[70:73], v[206:209], a[240:243]\n\t"
      "s_mov_b32 s63, 6684672\n\t"
      "buffer_store_dwordx4 v[170:173], v244, %[srdo], s63 offen offset:0 sc1\n\t"
      "s_waitcnt lgkmcnt(4)\n\t"
      "v_mfma_f32_16x16x32_f16 a[152:155], v[46:49], v[174:177], a[152:155]\n\t"
      "s_sleep 3\n\t"
      "v_mfma_f32_16x16x32_f16 a[152:155], v[50:53], v[178:181], a[152:155]\n\t"
      "v_mfma_f32_16x16x32_f16 a[156:159], v[50:53], v[186:189], a[156:159]\n\t"
      "v_accvgpr_read_b32 v142, a80\n\t"
      "v_accvgpr_read_b32 v143, a84\n\t"
      "v_mfma_f32_16x16x32_f16 a[156:159], v[46:49], v[182:185], a[156:159]\n\t"
      "v_accvgpr_read_b32 v144, a88\n\t"
      "v_accvgpr_read_b32 v145, a92\n\t"
      "v_mfma_f32_16x16x32_f16 a[188:191], v[54:57], v[182:185], a[188:191]\n\t"
      "ds_read_b128 v[222:225], %[vlr] offset:20480\n\t"
      "s_mov_b32 s63, 4194304\n\t"
      "v_mfma_f32_16x16x32_f16 a[188:191], v[58:61], v[186:189], a[188:191]\n\t"
      "ds_read_b128 v[226:229], %[vlr] offset:21504\n\t"
      "buffer_store_dwordx4 v[142:145], v244, %[srdo], s63 offen offset:256 sc1\n\t"
      "v_mfma_f32_16x16x32_f16 a[184:187], v[58:61], v[178:181], a[184:187]\n\t"
      "ds_read_b128 v[230:233], %[vlr] offset:22528\n\t"
      "v_mfma_f32_16x16x32_f16 a[184:187], v[54:57], v[174:177], a[184:187]\n\t"
      "ds_read_b128 v[234:237], %[vlr] offset:23552\n\t"
      "v_mfma_f32_16x16x32_f16 a[216:219], v[62:65], v[174:177], a[216:219]\n\t"
      "v_accvgpr_read_b32 v146, a81\n\t"
      "v_accvgpr_read_b32 v147, a85\n\t"
      "v_mfma_f32_16x16x32_f16 a[216:219], v[66:69], v[178:181], a[216:219]\n\t"
      "v_accvgpr_read_b32 v148, a89\n\t"
      "v_accvgpr_read_b32 v149, a93\n\t"
      "v_mfma_f32_16x16x32_f16 a[220:223], v[66:69], v[186:189], a[220:223]\n\t"
      "s_mov_b32 s63, 4325376\n\t"
      "buffer_store_dwordx4 v[146:149], v244, %[srdo], s63 offen offset:256 sc1\n\t"
      "v_mfma_f32_16x16x32_f16 a[220:223], v[62:65], v[182:185], a[220:223]\n\t"
      "v_mfma_f32_16x16x32_f16 a[252:255], v[70:73], v[182:185], a[252:255]\n\t"
      "v_mfma_f32_16x16x32_f16 a[252:255], v[74:77], v[186:189], a[252:255]\n\t"
      "v_accvgpr_read_b32 v150, a82\n\t"
      "v_accvgpr_read_b32 v151, a86\n\t"
      "v_mfma_f32_16x16x32_f16 a[248:251], v[74:77], v[178:181], a[248:251]\n\t"
      "v_accvgpr_read_b32 v152, a90\n\t"
      "v_accvgpr_read_b32 v153, a94\n\t"
      "v_mfma_f32_16x16x32_f16 a[248:251], v[70:73], v[174:177], a[248:251]\n\t"
      "s_mov_b32 s63, 4456448\n\t"
      "buffer_store_dwordx4 v[150:153], v244, %[srdo], s63 offen offset:256 sc1\n\t"
      "s_waitcnt vmcnt(41) lgkmcnt(4)\n\t"
      "v_mfma_f32_16x16x32_f16 a[128:131], v[94:97], v[190:193], a[128:131]\n\t"
      "v_mfma_f32_16x16x32_f16 a[128:131], v[98:101], v[194:197], a[128:131]\n\t"
      "v_mfma_f32_16x16x32_f16 a[132:135], v[98:101], v[202:205], a[132:135]\n\t"
      "v_accvgpr_read_b32 v154, a83\n\t"
      "v_accvgpr_read_b32 v155, a87\n\t"
      "v_mfma_f32_16x16x32_f16 a[132:135], v[94:97], v[198:201], a[132:135]\n\t"
      "v_accvgpr_read_b32 v156, a91\n\t"
      "v_accvgpr_read_b32 v157, a95\n\t"
      "v_mfma_f32_16x16x32_f16 a[164:167], v[102:105], v[198:201], a[164:167]\n\t"
      "ds_read_b128 v[206:209], %[vlr] offset:24576\n\t"
      "s_mov_b32 s63, 4587520\n\t"
      "v_mfma_f32_16x16x32_f16 a[164:167], v[106:109], v[202:205], a[164:167]\n\t"
      "ds_read_b128 v[210:213], %[vlr] offset:25600\n\t"
      "buffer_store_dwordx4 v[154:157], v244, %[srdo], s63 offen offset:256 sc1\n\t"
      "v_mfma_f32_16x16x32_f16 a[160:163], v[106:109], v[194:197], a[160:163]\n\t"
      "ds_read_b128 v[214:217], %[vlr] offset:26624\n\t"
      "v_mfma_f32_16x16x32_f16 a[160:163], v[102:105], v[190:193], a[160:163]\n\t"
      "ds_read_b128 v[218:221], %[vlr] offset:27648\n\t"
      "v_mfma_f32_16x16x32_f16 a[192:195], v[110:113], v[190:193], a[192:195]\n\t"
      "v_accvgpr_read_b32 v158, a112\n\t"
      "v_accvgpr_read_b32 v159, a116\n\t"
      "v_mfma_f32_16x16x32_f16 a[192:195], v[114:117], v[194:197], a[192:195]\n\t"
      "v_accvgpr_read_b32 v160, a120\n\t"
      "v_accvgpr_read_b32 v161, a124\n\t"
      "v_mfma_f32_16x16x32_f16 a[196:199], v[114:117], v[202:205], a[196:199]\n\t"
      "s_mov_b32 s63, 6291456\n\t"
      "buffer_store_dwordx4 v[158:161], v244, %[srdo], s63 offen offset:256 sc1\n\t"
      "v_mfma_f32_16x16x32_f16 a[196:199], v[110:113], v[198:201], a[196:199]\n\t"
      "v_mfma_f32_16x16x32_f16 a[228:231], v[118:121], v[198:201], a[228:231]\n\t"
      "v_mfma_f32_16x16x32_f16 a[228:231], v[122:125], v[202:205], a[228:231]\n\t"
      "v_accvgpr_read_b32 v162, a113\n\t"
      "v_accvgpr_read_b32 v163, a117\n\t"
      "v_mfma_f32_16x16x32_f16 a[224:227], v[122:125], v[194:197], a[224:227]\n\t"
      "v_accvgpr_read_b32 v164, a121\n\t"
      "v_accvgpr_read_b32 v165, a125\n\t"
      "v_mfma_f32_16x16x32_f16 a[224:227], v[118:121], v[190:193], a[224:227]\n\t"
      "s_mov_b32 s63, 6422528\n\t"
      "buffer_store_dwordx4 v[162:165], v244, %[srdo], s63 offen offset:256 sc1\n\t"
      "s_waitcnt lgkmcnt(4)\n\t"
      "v_mfma_f32_16x16x32_f16 a[136:139], v[94:97], v[222:225], a[136:139]\n\t"
      "s_sleep 3\n\t"
      "v_mfma_f32_16x16x32_f16 a[136:139], v[98:101], v[226:229], a[136:139]\n\t"
      "v_mfma_f32_16x16x32_f16 a[140:143], v[98:101], v[234:237], a[140:143]\n\t"
      "v_accvgpr_read_b32 v166, a114\n\t"
      "v_accvgpr_read_b32 v167, a118\n\t"
      "v_mfma_f32_16x16x32_f16 a[140:143], v[94:97], v[230:233], a[140:143]\n\t"
      "v_accvgpr_read_b32 v168, a122\n\t"
      "v_accvgpr_read_b32 v169, a126\n\t"
      "v_mfma_f32_16x16x32_f16 a[172:175], v[102:105], v[230:233], a[172:175]\n\t"
      "ds_read_b128 v[174:177], %[vlr] offset:28672\n\t"
      "s_mov_b32 s63, 6553600\n\t"
      "v_mfma_f32_16x16x32_f16 a[172:175], v[106:109], v[234:237], a[172:175]\n\t"
      "ds_read_b128 v[178:181], %[vlr] offset:29696\n\t"
      "buffer_store_dwordx4 v[166:169], v244, %[srdo], s63 offen offset:256 sc1\n\t"
      "v_mfma_f32_16x16x32_f16 a[168:171], v[106:109], v[226:229], a[168:171]\n\t"
      "ds_read_b128 v[182:185], %[vlr] offset:30720\n\t"
      "v_mfma_f32_16x16x32_f16 a[168:171], v[102:105], v[222:225], a[168:171]\n\t"
      "ds_read_b128 v[186:189], %[vlr] offset:31744\n\t"
      "v_mfma_f32_16x16x32_f16 a[200:203], v[110:113], v[222:225], a[200:203]\n\t"
      "v_accvgpr_read_b32 v170, a115\n\t"
      "v_accvgpr_read_b32 v171, a119\n\t"
      "v_mfma_f32_16x16x32_f16 a[200:203], v[114:117], v[226:229], a[200:203]\n\t"
      "v_accvgpr_read_b32 v172, a123\n\t"
      "v_accvgpr_read_b32 v173, a127\n\t"
      "v_mfma_f32_16x16x32_f16 a[204:207], v[114:117], v[234:237], a[204:207]\n\t"
      "s_mov_b32 s63, 6684672\n\t"
      "buffer_store_dwordx4 v[170:173], v244, %[srdo], s63 offen offset:256 sc1\n\t"
      "v_mfma_f32_16x16x32_f16 a[204:207], v[110:113], v[230:233], a[204:207]\n\t"
      "v_mfma_f32_16x16x32_f16 a[236:239], v[118:121], v[230:233], a[236:239]\n\t"
      "v_mfma_f32_16x16x32_f16 a[236:239], v[122:125], v[234:237], a[236:239]\n\t"
      "v_mfma_f32_16x16x32_f16 a[232:235], v[122:125], v[226:229], a[232:235]\n\t"
      "v_mfma_f32_16x16x32_f16 a[232:235], v[118:121], v[222:225], a[232:235]\n\t"
      "s_waitcnt lgkmcnt(4)\n\t"
      "v_mfma_f32_16x16x32_f16 a[144:147], v[94:97], v[206:209], a[144:147]\n\t"
      "v_mfma_f32_16x16x32_f16 a[144:147], v[98:101], v[210:213], a[144:147]\n\t"
      "v_mfma_f32_16x16x32_f16 a[148:151], v[98:101], v[218:221], a[148:151]\n\t"
      "v_mfma_f32_16x16x32_f16 a[148:151], v[94:97], v[214:217], a[148:151]\n\t"
      "v_mfma_f32_16x16x32_f16 a[180:183], v[102:105], v[214:217], a[180:183]\n\t"
      "ds_read_b128 v[190:193], %[vlr] offset:32768\n\t"
      "v_mfma_f32_16x16x32_f16 a[180:183], v[106:109], v[218:221], a[180:183]\n\t"
      "ds_read_b128 v[194:197], %[vlr] offset:33792\n\t"
      "v_mfma_f32_16x16x32_f16 a[176:179], v[106:109], v[210:213], a[176:179]\n\t"
      "ds_read_b128 v[198:201], %[vlr] offset:34816\n\t"
      "v_mfma_f32_16x16x32_f16 a[176:179], v[102:105], v[206:209], a[176:179]\n\t"
      "ds_read_b128 v[202:205], %[vlr] offset:35840\n\t"
      "v_mfma_f32_16x16x32_f16 a[208:211], v[110:113], v[206:209], a[208:211]\n\t"
      "v_mfma_f32_16x16x32_f16 a[208:211], v[114:117], v[210:213], a[208:211]\n\t"
      "v_mfma_f32_16x16x32_f16 a[212:215], v[114:117], v[218:221], a[212:215]\n\t"
      "v_mfma_f32_16x16x32_f16 a[212:215], v[110:113], v[214:217], a[212:215]\n\t"
      "v_mfma_f32_16x16x32_f16 a[244:247], v[118:121], v[214:217], a[244:247]\n\t"
      "v_mfma_f32_16x16x32_f16 a[244:247], v[122:125], v[218:221], a[244:247]\n\t"
      "v_mfma_f32_16x16x32_f16 a[240:243], v[122:125], v[210:213], a[240:243]\n\t"
      "v_mfma_f32_16x16x32_f16 a[240:243], v[118:121], v[206:209], a[240:243]\n\t"
      "s_waitcnt lgkmcnt(4)\n\t"
      "v_mfma_f32_16x16x32_f16 a[152:155], v[94:97], v[174:177], a[152:155]\n\t"
      "s_sleep 3\n\t"
      "v_mfma_f32_16x16x32_f16 a[152:155], v[98:101], v[178:181], a[152:155]\n\t"
      "v_mfma_f32_16x16x32_f16 a[156:159], v[98:101], v[186:189], a[156:159]\n\t"
      "v_mfma_f32_16x16x32_f16 a[156:159], v[94:97], v[182:185], a[156:159]\n\t"
      "v_mfma_f32_16x16x32_f16 a[188:191], v[102:105], v[182:185], a[188:191]\n\t"
      "ds_read_b128 v[222:225], %[vlr] offset:36864\n\t"
      "v_mfma_f32_16x16x32_f16 a[188:191], v[106:109], v[186:189], a[188:191]\n\t"
      "ds_read_b128 v[226:229], %[vlr] offset:37888\n\t"
      "v_mfma_f32_16x16x32_f16 a[184:187], v[106:109], v[178:181], a[184:187]\n\t"
      "ds_read_b128 v[230:233], %[vlr] offset:38912\n\t"
      "v_mfma_f32_16x16x32_f16 a[184:187], v[102:105], v[174:177], a[184:187]\n\t"
      "ds_read_b128 v[234:237], %[vlr] offset:39936\n\t"
      "v_mfma_f32_16x16x32_f16 a[216:219], v[110:113], v[174:177], a[216:219]\n\t"
      "v_mfma_f32_16x16x32_f16 a[216:219], v[114:117], v[178:181], a[216:219]\n\t"
      "v_mfma_f32_16x16x32_f16 a[220:223], v[114:117], v[186:189], a[220:223]\n\t"
      "v_mfma_f32_16x16x32_f16 a[220:223], v[110:113], v[182:185], a[220:223]\n\t"
      "v_mfma_f32_16x16x32_f16 a[252:255], v[118:121], v[182:185], a[252:255]\n\t"
      "v_mfma_f32_16x16x32_f16 a[252:255], v[122:125], v[186:189], a[252:255]\n\t"
      "v_mfma_f32_16x16x32_f16 a[248:251], v[122:125], v[178:181], a[248:251]\n\t"
      "v_mfma_f32_16x16x32_f16 a[248:251], v[118:121], v[174:177], a[248:251]\n\t"
      "s_waitcnt vmcnt(29) lgkmcnt(4)\n\t"
      "v_mfma_f32_16x16x32_f16 a[128:131], v[30:33], v[190:193], a[128:131]\n\t"
      "v_mfma_f32_16x16x32_f16 a[128:131], v[34:37], v[194:197], a[128:131]\n\t"
      "v_mfma_f32_16x16x32_f16 a[132:135], v[34:37], v[202:205], a[132:135]\n\t"
      "v_mfma_f32_16x16x32_f16 a[132:135], v[30:33], v[198:201], a[132:135]\n\t"
      "v_mfma_f32_16x16x32_f16 a[164:167], v[38:41], v[198:201], a[164:167]\n\t"
      "ds_read_b128 v[206:209], %[vlr] offset:40960\n\t"
      "v_mfma_f32_16x16x32_f16 a[164:167], v[42:45], v[202:205], a[164:167]\n\t"
      "ds_read_b128 v[210:213], %[vlr] offset:41984\n\t"
      "v_mfma_f32_16x16x32_f16 a[160:163], v[42:45], v[194:197], a[160:163]\n\t"
      "ds_read_b128 v[214:217], %[vlr] offset:43008\n\t"
      "v_mfma_f32_16x16x32_f16 a[160:163], v[38:41], v[190:193], a[160:163]\n\t"
      "ds_read_b128 v[218:221], %[vlr] offset:44032\n\t"
      "v_mfma_f32_16x16x32_f16 a[192:195], v[126:129], v[190:193], a[192:195]\n\t"
      "v_mfma_f32_16x16x32_f16 a[192:195], v[130:133], v[194:197], a[192:195]\n\t"
      "v_mfma_f32_16x16x32_f16 a[196:199], v[130:133], v[202:205], a[196:199]\n\t"
      "v_mfma_f32_16x16x32_f16 a[196:199], v[126:129], v[198:201], a[196:199]\n\t"
      "v_mfma_f32_16x16x32_f16 a[228:231], v[134:137], v[198:201], a[228:231]\n\t"
      "v_mfma_f32_16x16x32_f16 a[228:231], v[138:141], v[202:205], a[228:231]\n\t"
      "v_mfma_f32_16x16x32_f16 a[224:227], v[138:141], v[194:197], a[224:227]\n\t"
      "v_mfma_f32_16x16x32_f16 a[224:227], v[134:137], v[190:193], a[224:227]\n\t"
      "s_waitcnt lgkmcnt(4)\n\t"
      "v_mfma_f32_16x16x32_f16 a[136:139], v[30:33], v[222:225], a[136:139]\n\t"
      "s_sleep 3\n\t"
      "v_mfma_f32_16x16x32_f16 a[136:139], v[34:37], v[226:229], a[136:139]\n\t"
      "v_mfma_f32_16x16x32_f16 a[140:143], v[34:37], v[234:237], a[140:143]\n\t"
      "v_mfma_f32_16x16x32_f16 a[140:143], v[30:33], v[230:233], a[140:143]\n\t"
      "v_mfma_f32_16x16x32_f16 a[172:175], v[38:41], v[230:233], a[172:175]\n\t"
      "ds_read_b128 v[174:177], %[vlr] offset:45056\n\t"
      "v_mfma_f32_16x16x32_f16 a[172:175], v[42:45], v[234:237], a[172:175]\n\t"
      "ds_read_b128 v[178:181], %[vlr] offset:46080\n\t"
      "v_mfma_f32_16x16x32_f16 a[168:171], v[42:45], v[226:229], a[168:171]\n\t"
      "ds_read_b128 v[182:185], %[vlr] offset:47104\n\t"
      "v_mfma_f32_16x16x32_f16 a[168:171], v[38:41], v[222:225], a[168:171]\n\t"
      "ds_read_b128 v[186:189], %[vlr] offset:48128\n\t"
      "v_mfma_f32_16x16x32_f16 a[200:203], v[126:129], v[222:225], a[200:203]\n\t"
      "v_mfma_f32_16x16x32_f16 a[200:203], v[130:133], v[226:229], a[200:203]\n\t"
      "v_mfma_f32_16x16x32_f16 a[204:207], v[130:133], v[234:237], a[204:207]\n\t"
      "v_mfma_f32_16x16x32_f16 a[204:207], v[126:129], v[230:233], a[204:207]\n\t"
      "v_mfma_f32_16x16x32_f16 a[236:239], v[134:137], v[230:233], a[236:239]\n\t"
      "v_mfma_f32_16x16x32_f16 a[236:239], v[138:141], v[234:237], a[236:239]\n\t"
      "v_mfma_f32_16x16x32_f16 a[232:235], v[138:141], v[226:229], a[232:235]\n\t"
      "v_mfma_f32_16x16x32_f16 a[232:235], v[134:137], v[222:225], a[232:235]\n\t"
      "s_waitcnt lgkmcnt(4)\n\t"
      "v_mfma_f32_16x16x32_f16 a[144:147], v[30:33], v[206:209], a[144:147]\n\t"
      "v_mfma_f32_16x16x32_f16 a[144:147], v[34:37], v[210:213], a[144:147]\n\t"
      "v_mfma_f32_16x16x32_f16 a[148:151], v[34:37], v[218:221], a[148:151]\n\t"
      "v_mfma_f32_16x16x32_f16 a[148:151], v[30:33], v[214:217], a[148:151]\n\t"
      "v_mfma_f32_16x16x32_f16 a[180:183], v[38:41], v[214:217], a[180:183]\n\t"
      "ds_read_b128 v[190:193], %[vlr] offset:49152\n\t"
      "v_mfma_f32_16x16x32_f16 a[180:183], v[42:45], v[218:221], a[180:183]\n\t"
      "ds_read_b128 v[194:197], %[vlr] offset:50176\n\t"
      "v_mfma_f32_16x16x32_f16 a[176:179], v[42:45], v[210:213], a[176:179]\n\t"
      "ds_read_b128 v[198:201], %[vlr] offset:51200\n\t"
      "v_mfma_f32_16x16x32_f16 a[176:179], v[38:41], v[206:209], a[176:179]\n\t"
      "ds_read_b128 v[202:205], %[vlr] offset:52224\n\t"
      "v_mfma_f32_16x16x32_f16 a[208:211], v[126:129], v[206:209], a[208:211]\n\t"
      "v_mfma_f32_16x16x32_f16 a[208:211], v[130:133], v[210:213], a[208:211]\n\t"
      "v_mfma_f32_16x16x32_f16 a[212:215], v[130:133], v[218:221], a[212:215]\n\t"
      "v_mfma_f32_16x16x32_f16 a[212:215], v[126:129], v[214:217], a[212:215]\n\t"
      "v_mfma_f32_16x16x32_f16 a[244:247], v[134:137], v[214:217], a[244:247]\n\t"
      "v_mfma_f32_16x16x32_f16 a[244:247], v[138:141], v[218:221], a[244:247]\n\t"
      "v_mfma_f32_16x16x32_f16 a[240:243], v[138:141], v[210:213], a[240:243]\n\t"
      "v_mfma_f32_16x16x32_f16 a[240:243], v[134:137], v[206:209], a[240:243]\n\t"
      "s_waitcnt lgkmcnt(4)\n\t"
      "v_mfma_f32_16x16x32_f16 a[152:155], v[30:33], v[174:177], a[152:155]\n\t"
      "s_sleep 3\n\t"
      "v_mfma_f32_16x16x32_f16 a[152:155], v[34:37], v[178:181], a[152:155]\n\t"
      "v_mfma_f32_16x16x32_f16 a[156:159], v[34:37], v[186:189], a[156:159]\n\t"
      "v_mfma_f32_16x16x32_f16 a[156:159], v[30:33], v[182:185], a[156:159]\n\t"
      "v_mfma_f32_16x16x32_f16 a[188:191], v[38:41], v[182:185], a[188:191]\n\t"
      "ds_read_b128 v[222:225], %[vlr] offset:53248\n\t"
      "v_mfma_f32_16x16x32_f16 a[188:191], v[42:45], v[186:189], a[188:191]\n\t"
      "ds_read_b128 v[226:229], %[vlr] offset:54272\n\t"
      "v_mfma_f32_16x16x32_f16 a[184:187], v[42:45], v[178:181], a[184:187]\n\t"
      "ds_read_b128 v[230:233], %[vlr] offset:55296\n\t"
      "v_mfma_f32_16x16x32_f16 a[184:187], v[38:41], v[174:177], a[184:187]\n\t"
      "ds_read_b128 v[234:237], %[vlr] offset:56320\n\t"
      "v_mfma_f32_16x16x32_f16 a[216:219], v[126:129], v[174:177], a[216:219]\n\t"
      "v_mfma_f32_16x16x32_f16 a[216:219], v[130:133], v[178:181], a[216:219]\n\t"
      "v_mfma_f32_16x16x32_f16 a[220:223], v[130:133], v[186:189], a[220:223]\n\t"
      "v_mfma_f32_16x16x32_f16 a[220:223], v[126:129], v[182:185], a[220:223]\n\t"
      "v_mfma_f32_16x16x32_f16 a[252:255], v[134:137], v[182:185], a[252:255]\n\t"
      "v_mfma_f32_16x16x32_f16 a[252:255], v[138:141], v[186:189], a[252:255]\n\t"
      "v_mfma_f32_16x16x32_f16 a[248:251], v[138:141], v[178:181], a[248:251]\n\t"
      "v_mfma_f32_16x16x32_f16 a[248:251], v[134:137], v[174:177], a[248:251]\n\t"
      "s_waitcnt vmcnt(14) lgkmcnt(4)\n\t"
      "v_mfma_f32_16x16x32_f16 a[128:131], v[14:17], v[190:193], a[128:131]\n\t"
      "v_mfma_f32_16x16x32_f16 a[128:131], v[18:21], v[194:197], a[128:131]\n\t"
      "v_mfma_f32_16x16x32_f16 a[132:135], v[18:21], v[202:205], a[132:135]\n\t"
      "v_mfma_f32_16x16x32_f16 a[132:135], v[14:17], v[198:201], a[132:135]\n\t"
      "v_mfma_f32_16x16x32_f16 a[164:167], v[22:25], v[198:201], a[164:167]\n\t"
      "ds_read_b128 v[206:209], %[vlr] offset:57344\n\t"
      "v_mfma_f32_16x16x32_f16 a[164:167], v[26:29], v[202:205], a[164:167]\n\t"
      "ds_read_b128 v[210:213], %[vlr] offset:58368\n\t"
      "v_mfma_f32_16x16x32_f16 a[160:163], v[26:29], v[194:197], a[160:163]\n\t"
      "ds_read_b128 v[214:217], %[vlr] offset:59392\n\t"
      "v_mfma_f32_16x16x32_f16 a[160:163], v[22:25], v[190:193], a[160:163]\n\t"
      "ds_read_b128 v[218:221], %[vlr] offset:60416\n\t"
      "v_mfma_f32_16x16x32_f16 a[192:195], v[78:81], v[190:193], a[192:195]\n\t"
      "v_mfma_f32_16x16x32_f16 a[192:195], v[82:85], v[194:197], a[192:195]\n\t"
      "v_mfma_f32_16x16x32_f16 a[196:199], v[82:85], v[202:205], a[196:199]\n\t"
      "v_mfma_f32_16x16x32_f16 a[196:199], v[78:81], v[198:201], a[196:199]\n\t"
      "v_mfma_f32_16x16x32_f16 a[228:231], v[86:89], v[198:201], a[228:231]\n\t"
      "v_mfma_f32_16x16x32_f16 a[228:231], v[90:93], v[202:205], a[228:231]\n\t"
      "v_mfma_f32_16x16x32_f16 a[224:227], v[90:93], v[194:197], a[224:227]\n\t"
      "v_mfma_f32_16x16x32_f16 a[224:227], v[86:89], v[190:193], a[224:227]\n\t"
      "s_waitcnt lgkmcnt(4)\n\t"
      "v_mfma_f32_16x16x32_f16 a[136:139], v[14:17], v[222:225], a[136:139]\n\t"
      "s_sleep 3\n\t"
      "v_mfma_f32_16x16x32_f16 a[136:139], v[18:21], v[226:229], a[136:139]\n\t"
      "v_mfma_f32_16x16x32_f16 a[140:143], v[18:21], v[234:237], a[140:143]\n\t"
      "v_mfma_f32_16x16x32_f16 a[140:143], v[14:17], v[230:233], a[140:143]\n\t"
      "v_mfma_f32_16x16x32_f16 a[172:175], v[22:25], v[230:233], a[172:175]\n\t"
      "ds_read_b128 v[174:177], %[vlr] offset:61440\n\t"
      "v_mfma_f32_16x16x32_f16 a[172:175], v[26:29], v[234:237], a[172:175]\n\t"
      "ds_read_b128 v[178:181], %[vlr] offset:62464\n\t"
      "v_mfma_f32_16x16x32_f16 a[168:171], v[26:29], v[226:229], a[168:171]\n\t"
      "ds_read_b128 v[182:185], %[vlr] offset:63488\n\t"
      "v_mfma_f32_16x16x32_f16 a[168:171], v[22:25], v[222:225], a[168:171]\n\t"
      "ds_read_b128 v[186:189], %[vlr] offset:64512\n\t"
      "v_mfma_f32_16x16x32_f16 a[200:203], v[78:81], v[222:225], a[200:203]\n\t"
      "v_mfma_f32_16x16x32_f16 a[200:203], v[82:85], v[226:229], a[200:203]\n\t"
      "v_mfma_f32_16x16x32_f16 a[204:207], v[82:85], v[234:237], a[204:207]\n\t"
      "v_mfma_f32_16x16x32_f16 a[204:207], v[78:81], v[230:233], a[204:207]\n\t"
      "v_mfma_f32_16x16x32_f16 a[236:239], v[86:89], v[230:233], a[236:239]\n\t"
      "v_accvgpr_read_b32 v142, a128\n\t"
      "v_accvgpr_read_b32 v143, a132\n\t"
      "v_mfma_f32_16x16x32_f16 a[236:239], v[90:93], v[234:237], a[236:239]\n\t"
      "v_accvgpr_read_b32 v144, a136\n\t"
      "v_accvgpr_read_b32 v145, a140\n\t"
      "v_mfma_f32_16x16x32_f16 a[232:235], v[90:93], v[226:229], a[232:235]\n\t"
      "s_mov_b32 s63, 8388608\n\t"
      "buffer_store_dwordx4 v[142:145], v244, %[srdo], s63 offen offset:0 sc1\n\t"
      "v_mfma_f32_16x16x32_f16 a[232:235], v[86:89], v[222:225], a[232:235]\n\t"
      "s_waitcnt lgkmcnt(4)\n\t"
      "v_mfma_f32_16x16x32_f16 a[144:147], v[14:17], v[206:209], a[144:147]\n\t"
      "v_mfma_f32_16x16x32_f16 a[144:147], v[18:21], v[210:213], a[144:147]\n\t"
      "v_accvgpr_read_b32 v146, a129\n\t"
      "v_accvgpr_read_b32 v147, a133\n\t"
      "v_mfma_f32_16x16x32_f16 a[148:151], v[18:21], v[218:221], a[148:151]\n\t"
      "v_accvgpr_read_b32 v148, a137\n\t"
      "v_accvgpr_read_b32 v149, a141\n\t"
      "v_mfma_f32_16x16x32_f16 a[148:151], v[14:17], v[214:217], a[148:151]\n\t"
      "s_mov_b32 s63, 8519680\n\t"
      "buffer_store_dwordx4 v[146:149], v244, %[srdo], s63 offen offset:0 sc1\n\t"
      "v_mfma_f32_16x16x32_f16 a[180:183], v[22:25], v[214:217], a[180:183]\n\t"
      "v_mfma_f32_16x16x32_f16 a[180:183], v[26:29], v[218:221], a[180:183]\n\t"
      "v_mfma_f32_16x16x32_f16 a[176:179], v[26:29], v[210:213], a[176:179]\n\t"
      "v_accvgpr_read_b32 v150, a130\n\t"
      "v_accvgpr_read_b32 v151, a134\n\t"
      "v_mfma_f32_16x16x32_f16 a[176:179], v[22:25], v[206:209], a[176:179]\n\t"
      "v_accvgpr_read_b32 v152, a138\n\t"
      "v_accvgpr_read_b32 v153, a142\n\t"
      "v_mfma_f32_16x16x32_f16 a[208:211], v[78:81], v[206:209], a[208:211]\n\t"
      "s_mov_b32 s63, 8650752\n\t"
      "buffer_store_dwordx4 v[150:153], v244, %[srdo], s63 offen offset:0 sc1\n\t"
      "v_mfma_f32_16x16x32_f16 a[208:211], v[82:85], v[210:213], a[208:211]\n\t"
      "v_mfma_f32_16x16x32_f16 a[212:215], v[82:85], v[218:221], a[212:215]\n\t"
      "v_mfma_f32_16x16x32_f16 a[212:215], v[78:81], v[214:217], a[212:215]\n\t"
      "v_accvgpr_read_b32 v154, a131\n\t"
      "v_accvgpr_read_b32 v155, a135\n\t"
      "v_mfma_f32_16x16x32_f16 a[244:247], v[86:89], v[214:217], a[244:247]\n\t"
      "v_accvgpr_read_b32 v156, a139\n\t"
      "v_accvgpr_read_b32 v157, a143\n\t"
      "v_mfma_f32_16x16x32_f16 a[244:247], v[90:93], v[218:221], a[244:247]\n\t"
      "s_mov_b32 s63, 8781824\n\t"
      "buffer_store_dwordx4 v[154:157], v244, %[srdo], s63 offen offset:0 sc1\n\t"
      "v_mfma_f32_16x16x32_f16 a[240:243], v[90:93], v[210:213], a[240:243]\n\t"
      "v_mfma_f32_16x16x32_f16 a[240:243], v[86:89], v[206:209], a[240:243]\n\t"
      "s_waitcnt lgkmcnt(0)\n\t"
      "v_mfma_f32_16x16x32_f16 a[152:155], v[14:17], v[174:177], a[152:155]\n\t"
      "v_accvgpr_read_b32 v158, a160\n\t"
      "v_accvgpr_read_b32 v159, a164\n\t"
      "s_sleep 3\n\t"
      "v_mfma_f32_16x16x32_f16 a[152:155], v[18:21], v[178:181], a[152:155]\n\t"
      "v_accvgpr_read_b32 v160, a168\n\t"
      "v_accvgpr_read_b32 v161, a172\n\t"
      "v_mfma_f32_16x16x32_f16 a[156:159], v[18:21], v[186:189], a[156:159]\n\t"
      "s_mov_b32 s63, 10485760\n\t"
      "buffer_store_dwordx4 v[158:161], v244, %[srdo], s63 offen offset:0 sc1\n\t"
      "v_mfma_f32_16x16x32_f16 a[156:159], v[14:17], v[182:185], a[156:159]\n\t"
      "v_mfma_f32_16x16x32_f16 a[188:191], v[22:25], v[182:185], a[188:191]\n\t"
      "v_mfma_f32_16x16x32_f16 a[188:191], v[26:29], v[186:189], a[188:191]\n\t"
      "v_accvgpr_read_b32 v162, a161\n\t"
      "v_accvgpr_read_b32 v163, a165\n\t"
      "v_mfma_f32_16x16x32_f16 a[184:187], v[26:29], v[178:181], a[184:187]\n\t"
      "v_accvgpr_read_b32 v164, a169\n\t"
      "v_accvgpr_read_b32 v165, a173\n\t"
      "v_mfma_f32_16x16x32_f16 a[184:187], v[22:25], v[174:177], a[184:187]\n\t"
      "s_mov_b32 s63, 10616832\n\t"
      "buffer_store_dwordx4 v[162:165], v244, %[srdo], s63 offen offset:0 sc1\n\t"
      "v_mfma_f32_16x16x32_f16 a[216:219], v[78:81], v[174:177], a[216:219]\n\t"
      "v_mfma_f32_16x16x32_f16 a[216:219], v[82:85], v[178:181], a[216:219]\n\t"
      "v_mfma_f32_16x16x32_f16 a[220:223], v[82:85], v[186:189], a[220:223]\n\t"
      "v_mfma_f32_16x16x32_f16 a[220:223], v[78:81], v[182:185], a[220:223]\n\t"
      "v_mfma_f32_16x16x32_f16 a[252:255], v[86:89], v[182:185], a[252:255]\n\t"
      "v_mfma_f32_16x16x32_f16 a[252:255], v[90:93], v[186:189], a[252:255]\n\t"
      "v_mfma_f32_16x16x32_f16 a[248:251], v[90:93], v[178:181], a[248:251]\n\t"
      "v_mfma_f32_16x16x32_f16 a[248:251], v[86:89], v[174:177], a[248:251]\n\t"
      "s_memrealtime s[80:81]\n\t"
      "s_nop 7\n\t"
      "v_accvgpr_read_b32 v166, a162\n\t"
      "v_accvgpr_read_b32 v167, a166\n\t"
      "v_accvgpr_read_b32 v168, a170\n\t"
      "v_accvgpr_read_b32 v169, a174\n\t"
      "s_mov_b32 s63, 10747904\n\t"
      "buffer_store_dwordx4 v[166:169], v244, %[srdo], s63 offen offset:0 sc1\n\t"
      "v_accvgpr_read_b32 v170, a163\n\t"
      "v_accvgpr_read_b32 v171, a167\n\t"
      "v_accvgpr_read_b32 v172, a171\n\t"
      "v_accvgpr_read_b32 v173, a175\n\t"
      "s_mov_b32 s63, 10878976\n\t"
      "buffer_store_dwordx4 v[170:173], v244, %[srdo], s63 offen offset:0 sc1\n\t"
      "v_accvgpr_read_b32 v142, a192\n\t"
      "v_accvgpr_read_b32 v143, a196\n\t"
      "v_accvgpr_read_b32 v144, a200\n\t"
      "v_accvgpr_read_b32 v145, a204\n\t"
      "s_mov_b32 s63, 12582912\n\t"
      "buffer_store_dwordx4 v[142:145], v244, %[srdo], s63 offen offset:0 sc1\n\t"
      "v_accvgpr_read_b32 v146, a193\n\t"
      "v_accvgpr_read_b32 v147, a197\n\t"
      "v_accvgpr_read_b32 v148, a201\n\t"
      "v_accvgpr_read_b32 v149, a205\n\t"
      "s_mov_b32 s63, 12713984\n\t"
      "buffer_store_dwordx4 v[146:149], v244, %[srdo], s63 offen offset:0 sc1\n\t"
      "v_accvgpr_read_b32 v150, a194\n\t"
      "v_accvgpr_read_b32 v151, a198\n\t"
      "v_accvgpr_read_b32 v152, a202\n\t"
      "v_accvgpr_read_b32 v153, a206\n\t"
      "s_mov_b32 s63, 12845056\n\t"
      "buffer_store_dwordx4 v[150:153], v244, %[srdo], s63 offen offset:0 sc1\n\t"
      "v_accvgpr_read_b32 v154, a195\n\t"
      "v_accvgpr_read_b32 v155, a199\n\t"
      "v_accvgpr_read_b32 v156, a203\n\t"
      "v_accvgpr_read_b32 v157, a207\n\t"
      "s_mov_b32 s63, 12976128\n\t"
      "buffer_store_dwordx4 v[154:157], v244, %[srdo], s63 offen offset:0 sc1\n\t"
      "v_accvgpr_read_b32 v158, a224\n\t"
      "v_accvgpr_read_b32 v159, a228\n\t"
      "v_accvgpr_read_b32 v160, a232\n\t"
      "v_accvgpr_read_b32 v161, a236\n\t"
      "s_mov_b32 s63, 14680064\n\t"
      "buffer_store_dwordx4 v[158:161], v244, %[srdo], s63 offen offset:0 sc1\n\t"
      "v_accvgpr_read_b32 v162, a225\n\t"
      "v_accvgpr_read_b32 v163, a229\n\t"
      "v_accvgpr_read_b32 v164, a233\n\t"
      "v_accvgpr_read_b32 v165, a237\n\t"
      "s_mov_b32 s63, 14811136\n\t"
      "buffer_store_dwordx4 v[162:165], v244, %[srdo], s63 offen offset:0 sc1\n\t"
      "v_accvgpr_read_b32 v166, a226\n\t"
      "v_accvgpr_read_b32 v167, a230\n\t"
      "v_accvgpr_read_b32 v168, a234\n\t"
      "v_accvgpr_read_b32 v169, a238\n\t"
      "s_mov_b32 s63, 14942208\n\t"
      "buffer_store_dwordx4 v[166:169], v244, %[srdo], s63 offen offset:0 sc1\n\t"
      "v_accvgpr_read_b32 v170, a227\n\t"
      "v_accvgpr_read_b32 v171, a231\n\t"
      "v_accvgpr_read_b32 v172, a235\n\t"
      "v_accvgpr_read_b32 v173, a239\n\t"
      "s_mov_b32 s63, 15073280\n\t"
      "buffer_store_dwordx4 v[170:173], v244, %[srdo], s63 offen offset:0 sc1\n\t"
      "v_accvgpr_read_b32 v142, a144\n\t"
      "v_accvgpr_read_b32 v143, a148\n\t"
      "v_accvgpr_read_b32 v144, a152\n\t"
      "v_accvgpr_read_b32 v145, a156\n\t"
      "s_mov_b32 s63, 8388608\n\t"
      "buffer_store_dwordx4 v[142:145], v244, %[srdo], s63 offen offset:256 sc1\n\t"
      "v_accvgpr_read_b32 v146, a145\n\t"
      "v_accvgpr_read_b32 v147, a149\n\t"
      "v_accvgpr_read_b32 v148, a153\n\t"
      "v_accvgpr_read_b32 v149, a157\n\t"
      "s_mov_b32 s63, 8519680\n\t"
      "buffer_store_dwordx4 v[146:149], v244, %[srdo], s63 offen offset:256 sc1\n\t"
      "v_accvgpr_read_b32 v150, a146\n\t"
      "v_accvgpr_read_b32 v151, a150\n\t"
      "v_accvgpr_read_b32 v152, a154\n\t"
      "v_accvgpr_read_b32 v153, a158\n\t"
      "s_mov_b32 s63, 8650752\n\t"
      "buffer_store_dwordx4 v[150:153], v244, %[srdo], s63 offen offset:256 sc1\n\t"
      "v_accvgpr_read_b32 v154, a147\n\t"
      "v_accvgpr_read_b32 v155, a151\n\t"
      "v_accvgpr_read_b32 v156, a155\n\t"
      "v_accvgpr_read_b32 v157, a159\n\t"
      "s_mov_b32 s63, 8781824\n\t"
      "buffer_store_dwordx4 v[154:157], v244, %[srdo], s63 offen offset:256 sc1\n\t"
      "v_accvgpr_read_b32 v158, a176\n\t"
      "v_accvgpr_read_b32 v159, a180\n\t"
      "v_accvgpr_read_b32 v160, a184\n\t"
      "v_accvgpr_read_b32 v161, a188\n\t"
      "s_mov_b32 s63, 10485760\n\t"
      "buffer_store_dwordx4 v[158:161], v244, %[srdo], s63 offen offset:256 sc1\n\t"
      "v_accvgpr_read_b32 v162, a177\n\t"
      "v_accvgpr_read_b32 v163, a181\n\t"
      "v_accvgpr_read_b32 v164, a185\n\t"
      "v_accvgpr_read_b32 v165, a189\n\t"
      "s_mov_b32 s63, 10616832\n\t"
      "buffer_store_dwordx4 v[162:165], v244, %[srdo], s63 offen offset:256 sc1\n\t"
      "v_accvgpr_read_b32 v166, a178\n\t"
      "v_accvgpr_read_b32 v167, a182\n\t"
      "v_accvgpr_read_b32 v168, a186\n\t"
      "v_accvgpr_read_b32 v169, a190\n\t"
      "s_mov_b32 s63, 10747904\n\t"
      "buffer_store_dwordx4 v[166:169], v244, %[srdo], s63 offen offset:256 sc1\n\t"
      "v_accvgpr_read_b32 v170, a179\n\t"
      "v_accvgpr_read_b32 v171, a183\n\t"
      "v_accvgpr_read_b32 v172, a187\n\t"
      "v_accvgpr_read_b32 v173, a191\n\t"
      "s_mov_b32 s63, 10878976\n\t"
      "buffer_store_dwordx4 v[170:173], v244, %[srdo], s63 offen offset:256 sc1\n\t"
      "v_accvgpr_read_b32 v142, a208\n\t"
      "v_accvgpr_read_b32 v143, a212\n\t"
      "v_accvgpr_read_b32 v144, a216\n\t"
      "v_accvgpr_read_b32 v145, a220\n\t"
      "s_mov_b32 s63, 12582912\n\t"
      "buffer_store_dwordx4 v[142:145], v244, %[srdo], s63 offen offset:256 sc1\n\t"
      "v_accvgpr_read_b32 v146, a209\n\t"
      "v_accvgpr_read_b32 v147, a213\n\t"
      "v_accvgpr_read_b32 v148, a217\n\t"
      "v_accvgpr_read_b32 v149, a221\n\t"
      "s_mov_b32 s63, 12713984\n\t"
      "buffer_store_dwordx4 v[146:149], v244, %[srdo], s63 offen offset:256 sc1\n\t"
      "v_accvgpr_read_b32 v150, a210\n\t"
      "v_accvgpr_read_b32 v151, a214\n\t"
      "v_accvgpr_read_b32 v152, a218\n\t"
      "v_accvgpr_read_b32 v153, a222\n\t"
      "s_mov_b32 s63, 12845056\n\t"
      "buffer_store_dwordx4 v[150:153], v244, %[srdo], s63 offen offset:256 sc1\n\t"
      "v_accvgpr_read_b32 v154, a211\n\t"
      "v_accvgpr_read_b32 v155, a215\n\t"
      "v_accvgpr_read_b32 v156, a219\n\t"
      "v_accvgpr_read_b32 v157, a223\n\t"
      "s_mov_b32 s63, 12976128\n\t"
      "buffer_store_dwordx4 v[154:157], v244, %[srdo], s63 offen offset:256 sc1\n\t"
      "v_accvgpr_read_b32 v158, a240\n\t"
      "v_accvgpr_read_b32 v159, a244\n\t"
      "v_accvgpr_read_b32 v160, a248\n\t"
      "v_accvgpr_read_b32 v161, a252\n\t"
      "s_mov_b32 s63, 14680064\n\t"
      "buffer_store_dwordx4 v[158:161], v244, %[srdo], s63 offen offset:256 sc1\n\t"
      "v_accvgpr_read_b32 v162, a241\n\t"
      "v_accvgpr_read_b32 v163, a245\n\t"
      "v_accvgpr_read_b32 v164, a249\n\t"
      "v_accvgpr_read_b32 v165, a253\n\t"
      "s_mov_b32 s63, 14811136\n\t"
      "buffer_store_dwordx4 v[162:165], v244, %[srdo], s63 offen offset:256 sc1\n\t"
      "v_accvgpr_read_b32 v166, a242\n\t"
      "v_accvgpr_read_b32 v167, a246\n\t"
      "v_accvgpr_read_b32 v168, a250\n\t"
      "v_accvgpr_read_b32 v169, a254\n\t"
      "s_mov_b32 s63, 14942208\n\t"
      "buffer_store_dwordx4 v[166:169], v244, %[srdo], s63 offen offset:256 sc1\n\t"
      "v_accvgpr_read_b32 v170, a243\n\t"
      "v_accvgpr_read_b32 v171, a247\n\t"
      "v_accvgpr_read_b32 v172, a251\n\t"
      "v_accvgpr_read_b32 v173, a255\n\t"
      "s_mov_b32 s63, 15073280\n\t"
      "buffer_store_dwordx4 v[170:173], v244, %[srdo], s63 offen offset:256 sc1\n\t"
      "s_waitcnt lgkmcnt(0)\n\t"
      "v_mov_b32 v255, s80\n\t"
      "s_cmp_lg_u32 %[swave], 0\n\t"
      "s_cbranch_scc1 Lfin2_%=\n\t"
      "s_mov_b64 exec, 1\n\t"
      "buffer_store_dword v255, off, %[srdws], %[sseq] offset:4 sc1\n\t"
      "s_mov_b64 exec, -1\n\t"
      "s_branch Lfin2_%=\n\t"
      "LghC_%=:\n\t"
      "s_mov_b32 s62, 60\n\t"
      "Lgh1C_%=:\n\t"
      "buffer_load_dword v255, off, %[srdws], s84 sc1\n\t"
      "s_waitcnt vmcnt(0)\n\t"
      "v_readfirstlane_b32 s63, v255\n\t"
      "s_sub_u32 s62, s62, 1\n\t"
      "s_cmp_eq_u32 s62, 0\n\t"
      "s_cbranch_scc1 Lgh2C_%=\n\t"
      "s_cmp_ge_u32 s63, s74\n\t"
      "s_cbranch_scc1 Lgh2C_%=\n\t"
      "s_sleep 8\n\t"
      "s_branch Lgh1C_%=\n\t"
      ".p2align 6\n\t"
      "LepiH_%=:\n\t"
      "v_mfma_f32_16x16x32_f16 a[24:27], v[78:81], v[222:225], a[24:27]\n\t"
      "s_mov_b32 s67, 36864\n\t"
      "s_add_u32 s84, %[sfin], 128\n\t"
      "v_mfma_f32_16x16x32_f16 a[24:27], v[82:85], v[226:229], a[24:27]\n\t"
      "buffer_load_dword v254, off, %[srdws], s67 sc0 sc1\n\t"
      "v_cvt_pk_f16_f32 v238, v142, v146\n\t"
      "v_mfma_f32_16x16x32_f16 a[28:31], v[82:85], v[234:237], a[28:31]\n\t"
      "v_cvt_pk_f16_f32 v239, v150, v154\n\t"
      "ds_read_b128 v[206:209], %[vlr] offset:0\n\t"
      "v_mfma_f32_16x16x32_f16 a[28:31], v[78:81], v[230:233], a[28:31]\n\t"
      "v_cvt_pk_f16_f32 v240, v158, v162\n\t"
      "ds_read_b128 v[210:213], %[vlr] offset:1024\n\t"
      "v_mfma_f32_16x16x32_f16 a[60:63], v[86:89], v[230:233], a[60:63]\n\t"
      "v_cvt_pk_f16_f32 v241, v166, v170\n\t"
      "ds_read_b128 v[214:217], %[vlr] offset:2048\n\t"
      "v_mfma_f32_16x16x32_f16 a[60:63], v[90:93], v[234:237], a[60:63]\n\t"
      "buffer_load_dwordx4 v[78:81], %[vx0], s[68:71], s60 offen offset:0 sc0 sc1\n\t"
      "ds_write_b128 %[vlw], v[238:241] offset:16384\n\t"
      "v_mfma_f32_16x16x32_f16 a[56:59], v[90:93], v[226:229], a[56:59]\n\t"
      "v_cvt_pk_f16_f32 v238, v143, v147\n\t"
      "ds_read_b128 v[218:221], %[vlr] offset:3072\n\t"
      "v_mfma_f32_16x16x32_f16 a[56:59], v[86:89], v[222:225], a[56:59]\n\t"
      "buffer_load_dwordx4 v[82:85], %[vx0], s[68:71], s60 offen offset:1024 sc0 sc1\n\t"
      "v_cvt_pk_f16_f32 v239, v151, v155\n\t"
      "v_mfma_f32_16x16x32_f16 a[88:91], v[94:97], v[222:225], a[88:91]\n\t"
      "v_cvt_pk_f16_f32 v240, v159, v163\n\t"
      "v_mfma_f32_16x16x32_f16 a[88:91], v[98:101], v[226:229], a[88:91]\n\t"
      "buffer_load_dwordx4 v[86:89], %[vx0], s[68:71], s60 offen offset:2048 sc0 sc1\n\t"
      "v_cvt_pk_f16_f32 v241, v167, v171\n\t"
      "v_mfma_f32_16x16x32_f16 a[92:95], v[98:101], v[234:237], a[92:95]\n\t"
      "ds_write_b128 %[vlw], v[238:241] offset:18432\n\t"
      "v_mfma_f32_16x16x32_f16 a[92:95], v[94:97], v[230:233], a[92:95]\n\t"
      "buffer_load_dwordx4 v[90:93], %[vx0], s[68:71], s60 offen offset:3072 sc0 sc1\n\t"
      "v_cvt_pk_f16_f32 v238, v144, v148\n\t"
      "s_waitcnt vmcnt(5)\n\t"
      "v_mfma_f32_16x16x32_f16 a[124:127], v[102:105], v[230:233], a[124:127]\n\t"
      "v_cvt_pk_f16_f32 v239, v152, v156\n\t"
      "ds_read_b128 v[174:177], %[vlr] offset:4096\n\t"
      "v_mfma_f32_16x16x32_f16 a[124:127], v[106:109], v[234:237], a[124:127]\n\t"
      "buffer_load_dwordx4 v[94:97], %[vx1], s[68:71], s60 offen offset:0 sc0 sc1\n\t"
      "v_cvt_pk_f16_f32 v240, v160, v164\n\t"
      "v_mfma_f32_16x16x32_f16 a[120:123], v[106:109], v[226:229], a[120:123]\n\t"
      "v_cvt_pk_f16_f32 v241, v168, v172\n\t"
      "ds_read_b128 v[178:181], %[vlr] offset:5120\n\t"
      "v_mfma_f32_16x16x32_f16 a[120:123], v[102:105], v[222:225], a[120:123]\n\t"
      "buffer_load_dwordx4 v[98:101], %[vx1], s[68:71], s60 offen offset:1024 sc0 sc1\n\t"
      "ds_write_b128 %[vlw], v[238:241] offset:20480\n\t"
      "v_mfma_f32_16x16x32_f16 a[152:155], v[110:113], v[222:225], a[152:155]\n\t"
      "v_cvt_pk_f16_f32 v238, v145, v149\n\t"
      "ds_read_b128 v[182:185], %[vlr] offset:6144\n\t"
      "v_mfma_f32_16x16x32_f16 a[152:155], v[114:117], v[226:229], a[152:155]\n\t"
      "buffer_load_dwordx4 v[102:105], %[vx1], s[68:71], s60 offen offset:2048 sc0 sc1\n\t"
      "v_cvt_pk_f16_f32 v239, v153, v157\n\t"
      "v_mfma_f32_16x16x32_f16 a[156:159], v[114:117], v[234:237], a[156:159]\n\t"
      "v_cvt_pk_f16_f32 v240, v161, v165\n\t"
      "ds_read_b128 v[186:189], %[vlr] offset:7168\n\t"
      "v_mfma_f32_16x16x32_f16 a[156:159], v[110:113], v[230:233], a[156:159]\n\t"
      "buffer_load_dwordx4 v[106:109], %[vx1], s[68:71], s60 offen offset:3072 sc0 sc1\n\t"
      "s_add_u32 s60, s60, 0x2000\n\t"
      "v_mfma_f32_16x16x32_f16 a[188:191], v[118:121], v[230:233], a[188:191]\n\t"
      "buffer_load_dword v255, off, %[srdws], s84 sc1\n\t"
      "v_cvt_pk_f16_f32 v241, v169, v173\n\t"
      "v_mfma_f32_16x16x32_f16 a[188:191], v[122:125], v[234:237], a[188:191]\n\t"
      "buffer_load_dwordx4 v[110:113], %[vx0], s[68:71], s60 offen offset:0 sc0 sc1\n\t"
      "ds_write_b128 %[vlw], v[238:241] offset:22528\n\t"
      "v_mfma_f32_16x16x32_f16 a[184:187], v[122:125], v[226:229], a[184:187]\n\t"
      "v_bfe_u32 v244, %[vlr], 8, 2\n\t"
      "v_mfma_f32_16x16x32_f16 a[184:187], v[118:121], v[222:225], a[184:187]\n\t"
      "buffer_load_dwordx4 v[114:117], %[vx0], s[68:71], s60 offen offset:1024 sc0 sc1\n\t"
      "v_lshlrev_b32 v244, 19, v244\n\t"
      "v_mfma_f32_16x16x32_f16 a[216:219], v[126:129], v[222:225], a[216:219]\n\t"
      "v_and_b32 v245, 0xff, %[vlr]\n\t"
      "v_mfma_f32_16x16x32_f16 a[216:219], v[130:133], v[226:229], a[216:219]\n\t"
      "buffer_load_dwordx4 v[118:121], %[vx0], s[68:71], s60 offen offset:2048 sc0 sc1\n\t"
      "v_add3_u32 v244, v244, v245, %[sob]\n\t"
      "v_mfma_f32_16x16x32_f16 a[220:223], v[130:133], v[234:237], a[220:223]\n\t"
      "v_mfma_f32_16x16x32_f16 a[220:223], v[126:129], v[230:233], a[220:223]\n\t"
      "buffer_load_dwordx4 v[122:125], %[vx0], s[68:71], s60 offen offset:3072 sc0 sc1\n\t"
      "v_mfma_f32_16x16x32_f16 a[252:255], v[134:137], v[230:233], a[252:255]\n\t"
      "v_mfma_f32_16x16x32_f16 a[252:255], v[138:141], v[234:237], a[252:255]\n\t"
      "buffer_load_dwordx4 v[126:129], %[vx1], s[68:71], s60 offen offset:0 sc0 sc1\n\t"
      "v_mfma_f32_16x16x32_f16 a[248:251], v[138:141], v[226:229], a[248:251]\n\t"
      "s_waitcnt lgkmcnt(0)\n\t"
      "s_barrier\n\t"
      "v_mfma_f32_16x16x32_f16 a[248:251], v[134:137], v[222:225], a[248:251]\n\t"
      "buffer_load_dwordx4 v[130:133], %[vx1], s[68:71], s60 offen offset:1024 sc0 sc1\n\t"
      "ds_read_b128 v[190:193], %[vlr] offset:16384\n\t"
      "s_waitcnt lgkmcnt(8)\n\t"
      "v_mfma_f32_16x16x32_f16 a[0:3], v[14:17], v[206:209], a[0:3]\n\t"
      "ds_read_b128 v[194:197], %[vlr] offset:17408\n\t"
      "ds_read_b128 v[198:201], %[vlr] offset:18432\n\t"
      "v_mfma_f32_16x16x32_f16 a[0:3], v[18:21], v[210:213], a[0:3]\n\t"
      "buffer_load_dwordx4 v[134:137], %[vx1], s[68:71], s60 offen offset:2048 sc0 sc1\n\t"
      "ds_read_b128 v[202:205], %[vlr] offset:19456\n\t"
      "v_mfma_f32_16x16x32_f16 a[4:7], v[18:21], v[218:221], a[4:7]\n\t"
      "ds_read_b128 v[222:225], %[vlr] offset:20480\n\t"
      "ds_read_b128 v[226:229], %[vlr] offset:21504\n\t"
      "v_mfma_f32_16x16x32_f16 a[4:7], v[14:17], v[214:217], a[4:7]\n\t"
      "buffer_load_dwordx4 v[138:141], %[vx1], s[68:71], s60 offen offset:3072 sc0 sc1\n\t"
      "ds_read_b128 v[230:233], %[vlr] offset:22528\n\t"
      "v_mfma_f32_16x16x32_f16 a[36:39], v[22:25], v[214:217], a[36:39]\n\t"
      "ds_read_b128 v[234:237], %[vlr] offset:23552\n\t"
      "v_mfma_f32_16x16x32_f16 a[36:39], v[26:29], v[218:221], a[36:39]\n\t"
      "v_mfma_f32_16x16x32_f16 a[32:35], v[26:29], v[210:213], a[32:35]\n\t"
      "v_mfma_f32_16x16x32_f16 a[32:35], v[22:25], v[206:209], a[32:35]\n\t"
      "ds_read_b128 v[206:209], %[vlr] offset:8192\n\t"
      "ds_read_b128 v[210:213], %[vlr] offset:9216\n\t"
      "s_waitcnt lgkmcnt(11)\n\t"
      "v_mfma_f32_16x16x32_f16 a[8:11], v[14:17], v[174:177], a[8:11]\n\t"
      "ds_read_b128 v[214:217], %[vlr] offset:10240\n\t"
      "ds_read_b128 v[218:221], %[vlr] offset:11264\n\t"
      "v_mfma_f32_16x16x32_f16 a[8:11], v[18:21], v[178:181], a[8:11]\n\t"
      "v_mfma_f32_16x16x32_f16 a[12:15], v[18:21], v[186:189], a[12:15]\n\t"
      "v_mfma_f32_16x16x32_f16 a[12:15], v[14:17], v[182:185], a[12:15]\n\t"
      "v_mfma_f32_16x16x32_f16 a[44:47], v[22:25], v[182:185], a[44:47]\n\t"
      "v_mfma_f32_16x16x32_f16 a[44:47], v[26:29], v[186:189], a[44:47]\n\t"
      "v_mfma_f32_16x16x32_f16 a[40:43], v[26:29], v[178:181], a[40:43]\n\t"
      "v_mfma_f32_16x16x32_f16 a[40:43], v[22:25], v[174:177], a[40:43]\n\t"
      "ds_read_b128 v[174:177], %[vlr] offset:12288\n\t"
      "ds_read_b128 v[178:181], %[vlr] offset:13312\n\t"
      "s_waitcnt lgkmcnt(10)\n\t"
      "s_waitcnt vmcnt(13)\n\t"
      "v_mfma_f32_16x16x32_f16 a[0:3], v[78:81], v[190:193], a[0:3]\n\t"
      "ds_read_b128 v[182:185], %[vlr] offset:14336\n\t"
      "ds_read_b128 v[186:189], %[vlr] offset:15360\n\t"
      "v_mfma_f32_16x16x32_f16 a[0:3], v[82:85], v[194:197], a[0:3]\n\t"
      "v_mfma_f32_16x16x32_f16 a[4:7], v[82:85], v[202:205], a[4:7]\n\t"
      "v_mfma_f32_16x16x32_f16 a[4:7], v[78:81], v[198:201], a[4:7]\n\t"
      "v_mfma_f32_16x16x32_f16 a[36:39], v[86:89], v[198:201], a[36:39]\n\t"
      "v_mfma_f32_16x16x32_f16 a[36:39], v[90:93], v[202:205], a[36:39]\n\t"
      "v_mfma_f32_16x16x32_f16 a[32:35], v[90:93], v[194:197], a[32:35]\n\t"
      "v_mfma_f32_16x16x32_f16 a[32:35], v[86:89], v[190:193], a[32:35]\n\t"
      "ds_read_b128 v[190:193], %[vlr] offset:24576\n\t"
      "ds_read_b128 v[194:197], %[vlr] offset:25600\n\t"
      "s_waitcnt lgkmcnt(10)\n\t"
      "v_mfma_f32_16x16x32_f16 a[8:11], v[78:81], v[222:225], a[8:11]\n\t"
      "ds_read_b128 v[198:201], %[vlr] offset:26624\n\t"
      "ds_read_b128 v[202:205], %[vlr] offset:27648\n\t"
      "v_mfma_f32_16x16x32_f16 a[8:11], v[82:85], v[226:229], a[8:11]\n\t"
      "v_mfma_f32_16x16x32_f16 a[12:15], v[82:85], v[234:237], a[12:15]\n\t"
      "v_mfma_f32_16x16x32_f16 a[12:15], v[78:81], v[230:233], a[12:15]\n\t"
      "v_mfma_f32_16x16x32_f16 a[44:47], v[86:89], v[230:233], a[44:47]\n\t"
      "v_mfma_f32_16x16x32_f16 a[44:47], v[90:93], v[234:237], a[44:47]\n\t"
      "v_mfma_f32_16x16x32_f16 a[40:43], v[90:93], v[226:229], a[40:43]\n\t"
      "v_mfma_f32_16x16x32_f16 a[40:43], v[86:89], v[222:225], a[40:43]\n\t"
      "ds_read_b128 v[222:225], %[vlr] offset:28672\n\t"
      "ds_read_b128 v[226:229], %[vlr] offset:29696\n\t"
      "s_waitcnt lgkmcnt(10)\n\t"
      "v_mfma_f32_16x16x32_f16 a[16:19], v[14:17], v[206:209], a[16:19]\n\t"
      "ds_read_b128 v[230:233], %[vlr] offset:30720\n\t"
      "ds_read_b128 v[234:237], %[vlr] offset:31744\n\t"
      "s_waitcnt vmcnt(8)\n\t"
      "v_mfma_f32_16x16x32_f16 a[16:19], v[18:21], v[210:213], a[16:19]\n\t"
      "v_readfirstlane_b32 s63, v254\n\t"
      "v_readfirstlane_b32 s91, v255\n\t"
      "v_mfma_f32_16x16x32_f16 a[20:23], v[18:21], v[218:221], a[20:23]\n\t"
      "s_cmp_eq_u32 s63, s74\n\t"
      "s_cselect_b32 s75, 1, s75\n\t"
      "v_mfma_f32_16x16x32_f16 a[20:23], v[14:17], v[214:217], a[20:23]\n\t"
      "s_cmp_lg_u32 s75, 0\n\t"
      "s_cbranch_scc1 Lfbad_%=\n\t"
      "v_mfma_f32_16x16x32_f16 a[52:55], v[22:25], v[214:217], a[52:55]\n\t"
      "s_cmp_lt_u32 s91, s74\n\t"
      "s_cbranch_scc1 LghH_%=\n\t"
      "Lgh2H_%=:\n\t"
      "v_mfma_f32_16x16x32_f16 a[52:55], v[26:29], v[218:221], a[52:55]\n\t"
      "v_mfma_f32_16x16x32_f16 a[48:51], v[26:29], v[210:213], a[48:51]\n\t"
      "v_mfma_f32_16x16x32_f16 a[48:51], v[22:25], v[206:209], a[48:51]\n\t"
      "ds_read_b128 v[206:209], %[vlr] offset:0\n\t"
      "s_waitcnt lgkmcnt(9)\n\t"
      "v_mfma_f32_16x16x32_f16 a[24:27], v[14:17], v[174:177], a[24:27]\n\t"
      "ds_read_b128 v[210:213], %[vlr] offset:1024\n\t"
      "v_mfma_f32_16x16x32_f16 a[24:27], v[18:21], v[178:181], a[24:27]\n\t"
      "ds_read_b128 v[214:217], %[vlr] offset:2048\n\t"
      "v_accvgpr_read_b32 v142, a0\n\t"
      "v_mfma_f32_16x16x32_f16 a[28:31], v[18:21], v[186:189], a[28:31]\n\t"
      "ds_read_b128 v[218:221], %[vlr] offset:3072\n\t"
      "v_accvgpr_read_b32 v143, a4\n\t"
      "v_mfma_f32_16x16x32_f16 a[28:31], v[14:17], v[182:185], a[28:31]\n\t"
      "v_accvgpr_read_b32 v144, a8\n\t"
      "v_accvgpr_read_b32 v145, a12\n\t"
      "v_mfma_f32_16x16x32_f16 a[60:63], v[22:25], v[182:185], a[60:63]\n\t"
      "s_mov_b32 s63, 0\n\t"
      "buffer_store_dwordx4 v[142:145], v244, %[srdo], s63 offen offset:0 nt\n\t"
      "v_mfma_f32_16x16x32_f16 a[60:63], v[26:29], v[186:189], a[60:63]\n\t"
      "v_accvgpr_read_b32 v146, a1\n\t"
      "v_accvgpr_read_b32 v147, a5\n\t"
      "v_mfma_f32_16x16x32_f16 a[56:59], v[26:29], v[178:181], a[56:59]\n\t"
      "v_accvgpr_read_b32 v148, a9\n\t"
      "v_accvgpr_read_b32 v149, a13\n\t"
      "v_mfma_f32_16x16x32_f16 a[56:59], v[22:25], v[174:177], a[56:59]\n\t"
      "s_mov_b32 s63, 131072\n\t"
      "buffer_store_dwordx4 v[146:149], v244, %[srdo], s63 offen offset:0 nt\n\t"
      "s_waitcnt lgkmcnt(8)\n\t"
      "v_mfma_f32_16x16x32_f16 a[16:19], v[78:81], v[190:193], a[16:19]\n\t"
      "ds_read_b128 v[174:177], %[vlr] offset:4096\n\t"
      "v_accvgpr_read_b32 v150, a2\n\t"
      "v_mfma_f32_16x16x32_f16 a[16:19], v[82:85], v[194:197], a[16:19]\n\t"
      "ds_read_b128 v[178:181], %[vlr] offset:5120\n\t"
      "v_accvgpr_read_b32 v151, a6\n\t"
      "v_mfma_f32_16x16x32_f16 a[20:23], v[82:85], v[202:205], a[20:23]\n\t"
      "ds_read_b128 v[182:185], %[vlr] offset:6144\n\t"
      "v_accvgpr_read_b32 v152, a10\n\t"
      "v_mfma_f32_16x16x32_f16 a[20:23], v[78:81], v[198:201], a[20:23]\n\t"
      "ds_read_b128 v[186:189], %[vlr] offset:7168\n\t"
      "v_accvgpr_read_b32 v153, a14\n\t"
      "v_mfma_f32_16x16x32_f16 a[52:55], v[86:89], v[198:201], a[52:55]\n\t"
      "s_mov_b32 s63, 262144\n\t"
      "buffer_store_dwordx4 v[150:153], v244, %[srdo], s63 offen offset:0 nt\n\t"
      "v_mfma_f32_16x16x32_f16 a[52:55], v[90:93], v[202:205], a[52:55]\n\t"
      "v_accvgpr_read_b32 v154, a3\n\t"
      "v_accvgpr_read_b32 v155, a7\n\t"
      "v_mfma_f32_16x16x32_f16 a[48:51], v[90:93], v[194:197], a[48:51]\n\t"
      "v_accvgpr_read_b32 v156, a11\n\t"
      "v_accvgpr_read_b32 v157, a15\n\t"
      "v_mfma_f32_16x16x32_f16 a[48:51], v[86:89], v[190:193], a[48:51]\n\t"
      "s_mov_b32 s63, 393216\n\t"
      "buffer_store_dwordx4 v[154:157], v244, %[srdo], s63 offen offset:0 nt\n\t"
      "s_waitcnt lgkmcnt(8)\n\t"
      "v_mfma_f32_16x16x32_f16 a[24:27], v[78:81], v[222:225], a[24:27]\n\t"
      "v_accvgpr_read_b32 v158, a32\n\t"
      "v_accvgpr_read_b32 v159, a36\n\t"
      "v_mfma_f32_16x16x32_f16 a[24:27], v[82:85], v[226:229], a[24:27]\n\t"
      "v_accvgpr_read_b32 v160, a40\n\t"
      "v_accvgpr_read_b32 v161, a44\n\t"
      "v_mfma_f32_16x16x32_f16 a[28:31], v[82:85], v[234:237], a[28:31]\n\t"
      "s_mov_b32 s63, 2097152\n\t"
      "buffer_store_dwordx4 v[158:161], v244, %[srdo], s63 offen offset:0 nt\n\t"
      "v_mfma_f32_16x16x32_f16 a[28:31], v[78:81], v[230:233], a[28:31]\n\t"
      "v_accvgpr_read_b32 v162, a33\n\t"
      "v_accvgpr_read_b32 v163, a37\n\t"
      "v_mfma_f32_16x16x32_f16 a[60:63], v[86:89], v[230:233], a[60:63]\n\t"
      "v_accvgpr_read_b32 v164, a41\n\t"
      "v_accvgpr_read_b32 v165, a45\n\t"
      "v_mfma_f32_16x16x32_f16 a[60:63], v[90:93], v[234:237], a[60:63]\n\t"
      "s_mov_b32 s63, 2228224\n\t"
      "buffer_store_dwordx4 v[162:165], v244, %[srdo], s63 offen offset:0 nt\n\t"
      "v_mfma_f32_16x16x32_f16 a[56:59], v[90:93], v[226:229], a[56:59]\n\t"
      "v_accvgpr_read_b32 v166, a34\n\t"
      "v_accvgpr_read_b32 v167, a38\n\t"
      "v_mfma_f32_16x16x32_f16 a[56:59], v[86:89], v[222:225], a[56:59]\n\t"
      "v_accvgpr_read_b32 v168, a42\n\t"
      "v_accvgpr_read_b32 v169, a46\n\t"
      "s_waitcnt lgkmcnt(4)\n\t"
      "v_mfma_f32_16x16x32_f16 a[64:67], v[30:33], v[206:209], a[64:67]\n\t"
      "s_mov_b32 s63, 2359296\n\t"
      "buffer_store_dwordx4 v[166:169], v244, %[srdo], s63 offen offset:0 nt\n\t"
      "v_mfma_f32_16x16x32_f16 a[64:67], v[34:37], v[210:213], a[64:67]\n\t"
      "v_accvgpr_read_b32 v170, a35\n\t"
      "v_accvgpr_read_b32 v171, a39\n\t"
      "v_mfma_f32_16x16x32_f16 a[68:71], v[34:37], v[218:221], a[68:71]\n\t"
      "v_accvgpr_read_b32 v172, a43\n\t"
      "v_accvgpr_read_b32 v173, a47\n\t"
      "v_mfma_f32_16x16x32_f16 a[68:71], v[30:33], v[214:217], a[68:71]\n\t"
      "s_mov_b32 s63, 2490368\n\t"
      "buffer_store_dwordx4 v[170:173], v244, %[srdo], s63 offen offset:0 nt\n\t"
      "v_mfma_f32_16x16x32_f16 a[100:103], v[38:41], v[214:217], a[100:103]\n\t"
      "v_mfma_f32_16x16x32_f16 a[100:103], v[42:45], v[218:221], a[100:103]\n\t"
      "v_mfma_f32_16x16x32_f16 a[96:99], v[42:45], v[210:213], a[96:99]\n\t"
      "v_mfma_f32_16x16x32_f16 a[96:99], v[38:41], v[206:209], a[96:99]\n\t"
      "v_mfma_f32_16x16x32_f16 a[128:131], v[46:49], v[206:209], a[128:131]\n\t"
      "ds_read_b128 v[190:193], %[vlr] offset:16384\n\t"
      "v_mfma_f32_16x16x32_f16 a[128:131], v[50:53], v[210:213], a[128:131]\n\t"
      "ds_read_b128 v[194:197], %[vlr] offset:17408\n\t"
      "v_accvgpr_read_b32 v142, a16\n\t"
      "v_mfma_f32_16x16x32_f16 a[132:135], v[50:53], v[218:221], a[132:135]\n\t"
      "ds_read_b128 v[198:201], %[vlr] offset:18432\n\t"
      "v_accvgpr_read_b32 v143, a20\n\t"
      "v_mfma_f32_16x16x32_f16 a[132:135], v[46:49], v[214:217], a[132:135]\n\t"
      "ds_read_b128 v[202:205], %[vlr] offset:19456\n\t"
      "v_accvgpr_read_b32 v144, a24\n\t"
      "v_mfma_f32_16x16x32_f16 a[164:167], v[54:57], v[214:217], a[164:167]\n\t"
      "v_accvgpr_read_b32 v145, a28\n\t"
      "s_mov_b32 s63, 0\n\t"
      "v_mfma_f32_16x16x32_f16 a[164:167], v[58:61], v[218:221], a[164:167]\n\t"
      "buffer_store_dwordx4 v[142:145], v244, %[srdo], s63 offen offset:256 nt\n\t"
      "v_accvgpr_read_b32 v146, a17\n\t"
      "v_mfma_f32_16x16x32_f16 a[160:163], v[58:61], v[210:213], a[160:163]\n\t"
      "v_accvgpr_read_b32 v147, a21\n\t"
      "v_accvgpr_read_b32 v148, a25\n\t"
      "v_mfma_f32_16x16x32_f16 a[160:163], v[54:57], v[206:209], a[160:163]\n\t"
      "v_accvgpr_read_b32 v149, a29\n\t"
      "s_mov_b32 s63, 131072\n\t"
      "v_mfma_f32_16x16x32_f16 a[192:195], v[62:65], v[206:209], a[192:195]\n\t"
      "buffer_store_dwordx4 v[146:149], v244, %[srdo], s63 offen offset:256 nt\n\t"
      "v_accvgpr_read_b32 v150, a18\n\t"
      "v_mfma_f32_16x16x32_f16 a[192:195], v[66:69], v[210:213], a[192:195]\n\t"
      "v_accvgpr_read_b32 v151, a22\n\t"
      "v_accvgpr_read_b32 v152, a26\n\t"
      "v_mfma_f32_16x16x32_f16 a[196:199], v[66:69], v[218:221], a[196:199]\n\t"
      "v_accvgpr_read_b32 v153, a30\n\t"
      "s_mov_b32 s63, 262144\n\t"
      "v_mfma_f32_16x16x32_f16 a[196:199], v[62:65], v[214:217], a[196:199]\n\t"
      "buffer_store_dwordx4 v[150:153], v244, %[srdo], s63 offen offset:256 nt\n\t"
      "v_accvgpr_read_b32 v154, a19\n\t"
      "v_mfma_f32_16x16x32_f16 a[228:231], v[70:73], v[214:217], a[228:231]\n\t"
      "v_accvgpr_read_b32 v155, a23\n\t"
      "v_accvgpr_read_b32 v156, a27\n\t"
      "v_mfma_f32_16x16x32_f16 a[228:231], v[74:77], v[218:221], a[228:231]\n\t"
      "v_accvgpr_read_b32 v157, a31\n\t"
      "s_mov_b32 s63, 393216\n\t"
      "v_mfma_f32_16x16x32_f16 a[224:227], v[74:77], v[210:213], a[224:227]\n\t"
      "buffer_store_dwordx4 v[154:157], v244, %[srdo], s63 offen offset:256 nt\n\t"
      "v_accvgpr_read_b32 v158, a48\n\t"
      "v_mfma_f32_16x16x32_f16 a[224:227], v[70:73], v[206:209], a[224:227]\n\t"
      "v_accvgpr_read_b32 v159, a52\n\t"
      "v_accvgpr_read_b32 v160, a56\n\t"
      "s_waitcnt lgkmcnt(4)\n\t"
      "v_mfma_f32_16x16x32_f16 a[72:75], v[30:33], v[174:177], a[72:75]\n\t"
      "v_accvgpr_read_b32 v161, a60\n\t"
      "s_mov_b32 s63, 2097152\n\t"
      "v_mfma_f32_16x16x32_f16 a[72:75], v[34:37], v[178:181], a[72:75]\n\t"
      "buffer_store_dwordx4 v[158:161], v244, %[srdo], s63 offen offset:256 nt\n\t"
      "v_accvgpr_read_b32 v162, a49\n\t"
      "v_mfma_f32_16x16x32_f16 a[76:79], v[34:37], v[186:189], a[76:79]\n\t"
      "v_accvgpr_read_b32 v163, a53\n\t"
      "v_accvgpr_read_b32 v164, a57\n\t"
      "v_mfma_f32_16x16x32_f16 a[76:79], v[30:33], v[182:185], a[76:79]\n\t"
      "v_accvgpr_read_b32 v165, a61\n\t"
      "s_mov_b32 s63, 2228224\n\t"
      "v_mfma_f32_16x16x32_f16 a[108:111], v[38:41], v[182:185], a[108:111]\n\t"
      "buffer_store_dwordx4 v[162:165], v244, %[srdo], s63 offen offset:256 nt\n\t"
      "v_accvgpr_read_b32 v166, a50\n\t"
      "v_mfma_f32_16x16x32_f16 a[108:111], v[42:45], v[186:189], a[108:111]\n\t"
      "v_accvgpr_read_b32 v167, a54\n\t"
      "v_accvgpr_read_b32 v168, a58\n\t"
      "v_mfma_f32_16x16x32_f16 a[104:107], v[42:45], v[178:181], a[104:107]\n\t"
      "v_accvgpr_read_b32 v169, a62\n\t"
      "s_mov_b32 s63, 2359296\n\t"
      "v_mfma_f32_16x16x32_f16 a[104:107], v[38:41], v[174:177], a[104:107]\n\t"
      "buffer_store_dwordx4 v[166:169], v244, %[srdo], s63 offen offset:256 nt\n\t"
      "v_accvgpr_read_b32 v170, a51\n\t"
      "v_mfma_f32_16x16x32_f16 a[136:139], v[46:49], v[174:177], a[136:139]\n\t"
      "ds_read_b128 v[222:225], %[vlr] offset:20480\n\t"
      "v_accvgpr_read_b32 v171, a55\n\t"
      "v_mfma_f32_16x16x32_f16 a[136:139], v[50:53], v[178:181], a[136:139]\n\t"
      "ds_read_b128 v[226:229], %[vlr] offset:21504\n\t"
      "v_accvgpr_read_b32 v172, a59\n\t"
      "v_mfma_f32_16x16x32_f16 a[140:143], v[50:53], v[186:189], a[140:143]\n\t"
      "ds_read_b128 v[230:233], %[vlr] offset:22528\n\t"
      "v_accvgpr_read_b32 v173, a63\n\t"
      "v_mfma_f32_16x16x32_f16 a[140:143], v[46:49], v[182:185], a[140:143]\n\t"
      "ds_read_b128 v[234:237], %[vlr] offset:23552\n\t"
      "s_mov_b32 s63, 2490368\n\t"
      "v_mfma_f32_16x16x32_f16 a[172:175], v[54:57], v[182:185], a[172:175]\n\t"
      "buffer_store_dwordx4 v[170:173], v244, %[srdo], s63 offen offset:256 nt\n\t"
      "v_mfma_f32_16x16x32_f16 a[172:175], v[58:61], v[186:189], a[172:175]\n\t"
      "v_mfma_f32_16x16x32_f16 a[168:171], v[58:61], v[178:181], a[168:171]\n\t"
      "v_mfma_f32_16x16x32_f16 a[168:171], v[54:57], v[174:177], a[168:171]\n\t"
      "v_mfma_f32_16x16x32_f16 a[200:203], v[62:65], v[174:177], a[200:203]\n\t"
      "v_mfma_f32_16x16x32_f16 a[200:203], v[66:69], v[178:181], a[200:203]\n\t"
      "v_mfma_f32_16x16x32_f16 a[204:207], v[66:69], v[186:189], a[204:207]\n\t"
      "v_mfma_f32_16x16x32_f16 a[204:207], v[62:65], v[182:185], a[204:207]\n\t"
      "v_mfma_f32_16x16x32_f16 a[236:239], v[70:73], v[182:185], a[236:239]\n\t"
      "v_mfma_f32_16x16x32_f16 a[236:239], v[74:77], v[186:189], a[236:239]\n\t"
      "v_mfma_f32_16x16x32_f16 a[232:235], v[74:77], v[178:181], a[232:235]\n\t"
      "v_mfma_f32_16x16x32_f16 a[232:235], v[70:73], v[174:177], a[232:235]\n\t"
      "s_waitcnt lgkmcnt(4)\n\t"
      "s_waitcnt vmcnt(16)\n\t"
      "v_mfma_f32_16x16x32_f16 a[64:67], v[94:97], v[190:193], a[64:67]\n\t"
      "v_mfma_f32_16x16x32_f16 a[64:67], v[98:101], v[194:197], a[64:67]\n\t"
      "v_mfma_f32_16x16x32_f16 a[68:71], v[98:101], v[202:205], a[68:71]\n\t"
      "v_mfma_f32_16x16x32_f16 a[68:71], v[94:97], v[198:201], a[68:71]\n\t"
      "v_mfma_f32_16x16x32_f16 a[100:103], v[102:105], v[198:201], a[100:103]\n\t"
      "v_mfma_f32_16x16x32_f16 a[100:103], v[106:109], v[202:205], a[100:103]\n\t"
      "v_mfma_f32_16x16x32_f16 a[96:99], v[106:109], v[194:197], a[96:99]\n\t"
      "v_mfma_f32_16x16x32_f16 a[96:99], v[102:105], v[190:193], a[96:99]\n\t"
      "v_mfma_f32_16x16x32_f16 a[128:131], v[110:113], v[190:193], a[128:131]\n\t"
      "ds_read_b128 v[206:209], %[vlr] offset:8192\n\t"
      "v_mfma_f32_16x16x32_f16 a[128:131], v[114:117], v[194:197], a[128:131]\n\t"
      "ds_read_b128 v[210:213], %[vlr] offset:9216\n\t"
      "v_mfma_f32_16x16x32_f16 a[132:135], v[114:117], v[202:205], a[132:135]\n\t"
      "ds_read_b128 v[214:217], %[vlr] offset:10240\n\t"
      "v_mfma_f32_16x16x32_f16 a[132:135], v[110:113], v[198:201], a[132:135]\n\t"
      "ds_read_b128 v[218:221], %[vlr] offset:11264\n\t"
      "v_mfma_f32_16x16x32_f16 a[164:167], v[118:121], v[198:201], a[164:167]\n\t"
      "v_mfma_f32_16x16x32_f16 a[164:167], v[122:125], v[202:205], a[164:167]\n\t"
      "v_mfma_f32_16x16x32_f16 a[160:163], v[122:125], v[194:197], a[160:163]\n\t"
      "v_mfma_f32_16x16x32_f16 a[160:163], v[118:121], v[190:193], a[160:163]\n\t"
      "v_mfma_f32_16x16x32_f16 a[192:195], v[126:129], v[190:193], a[192:195]\n\t"
      "v_mfma_f32_16x16x32_f16 a[192:195], v[130:133], v[194:197], a[192:195]\n\t"
      "v_mfma_f32_16x16x32_f16 a[196:199], v[130:133], v[202:205], a[196:199]\n\t"
      "v_mfma_f32_16x16x32_f16 a[196:199], v[126:129], v[198:201], a[196:199]\n\t"
      "v_mfma_f32_16x16x32_f16 a[228:231], v[134:137], v[198:201], a[228:231]\n\t"
      "v_mfma_f32_16x16x32_f16 a[228:231], v[138:141], v[202:205], a[228:231]\n\t"
      "v_mfma_f32_16x16x32_f16 a[224:227], v[138:141], v[194:197], a[224:227]\n\t"
      "v_mfma_f32_16x16x32_f16 a[224:227], v[134:137], v[190:193], a[224:227]\n\t"
      "s_waitcnt lgkmcnt(4)\n\t"
      "v_mfma_f32_16x16x32_f16 a[72:75], v[94:97], v[222:225], a[72:75]\n\t"
      "v_mfma_f32_16x16x32_f16 a[72:75], v[98:101], v[226:229], a[72:75]\n\t"
      "v_mfma_f32_16x16x32_f16 a[76:79], v[98:101], v[234:237], a[76:79]\n\t"
      "v_mfma_f32_16x16x32_f16 a[76:79], v[94:97], v[230:233], a[76:79]\n\t"
      "v_mfma_f32_16x16x32_f16 a[108:111], v[102:105], v[230:233], a[108:111]\n\t"
      "v_mfma_f32_16x16x32_f16 a[108:111], v[106:109], v[234:237], a[108:111]\n\t"
      "v_mfma_f32_16x16x32_f16 a[104:107], v[106:109], v[226:229], a[104:107]\n\t"
      "v_mfma_f32_16x16x32_f16 a[104:107], v[102:105], v[222:225], a[104:107]\n\t"
      "v_mfma_f32_16x16x32_f16 a[136:139], v[110:113], v[222:225], a[136:139]\n\t"
      "ds_read_b128 v[174:177], %[vlr] offset:12288\n\t"
      "v_mfma_f32_16x16x32_f16 a[136:139], v[114:117], v[226:229], a[136:139]\n\t"
      "ds_read_b128 v[178:181], %[vlr] offset:13312\n\t"
      "v_mfma_f32_16x16x32_f16 a[140:143], v[114:117], v[234:237], a[140:143]\n\t"
      "ds_read_b128 v[182:185], %[vlr] offset:14336\n\t"
      "v_mfma_f32_16x16x32_f16 a[140:143], v[110:113], v[230:233], a[140:143]\n\t"
      "ds_read_b128 v[186:189], %[vlr] offset:15360\n\t"
      "v_mfma_f32_16x16x32_f16 a[172:175], v[118:121], v[230:233], a[172:175]\n\t"
      "v_mfma_f32_16x16x32_f16 a[172:175], v[122:125], v[234:237], a[172:175]\n\t"
      "v_mfma_f32_16x16x32_f16 a[168:171], v[122:125], v[226:229], a[168:171]\n\t"
      "v_mfma_f32_16x16x32_f16 a[168:171], v[118:121], v[222:225], a[168:171]\n\t"
      "v_mfma_f32_16x16x32_f16 a[200:203], v[126:129], v[222:225], a[200:203]\n\t"
      "v_mfma_f32_16x16x32_f16 a[200:203], v[130:133], v[226:229], a[200:203]\n\t"
      "v_mfma_f32_16x16x32_f16 a[204:207], v[130:133], v[234:237], a[204:207]\n\t"
      "v_mfma_f32_16x16x32_f16 a[204:207], v[126:129], v[230:233], a[204:207]\n\t"
      "v_mfma_f32_16x16x32_f16 a[236:239], v[134:137], v[230:233], a[236:239]\n\t"
      "v_mfma_f32_16x16x32_f16 a[236:239], v[138:141], v[234:237], a[236:239]\n\t"
      "v_mfma_f32_16x16x32_f16 a[232:235], v[138:141], v[226:229], a[232:235]\n\t"
      "v_mfma_f32_16x16x32_f16 a[232:235], v[134:137], v[222:225], a[232:235]\n\t"
      "s_waitcnt lgkmcnt(4)\n\t"
      "v_mfma_f32_16x16x32_f16 a[80:83], v[30:33], v[206:209], a[80:83]\n\t"
      "v_mfma_f32_16x16x32_f16 a[80:83], v[34:37], v[210:213], a[80:83]\n\t"
      "v_mfma_f32_16x16x32_f16 a[84:87], v[34:37], v[218:221], a[84:87]\n\t"
      "v_mfma_f32_16x16x32_f16 a[84:87], v[30:33], v[214:217], a[84:87]\n\t"
      "v_mfma_f32_16x16x32_f16 a[116:119], v[38:41], v[214:217], a[116:119]\n\t"
      "v_mfma_f32_16x16x32_f16 a[116:119], v[42:45], v[218:221], a[116:119]\n\t"
      "v_mfma_f32_16x16x32_f16 a[112:115], v[42:45], v[210:213], a[112:115]\n\t"
      "v_mfma_f32_16x16x32_f16 a[112:115], v[38:41], v[206:209], a[112:115]\n\t"
      "v_mfma_f32_16x16x32_f16 a[144:147], v[46:49], v[206:209], a[144:147]\n\t"
      "ds_read_b128 v[190:193], %[vlr] offset:24576\n\t"
      "v_mfma_f32_16x16x32_f16 a[144:147], v[50:53], v[210:213], a[144:147]\n\t"
      "ds_read_b128 v[194:197], %[vlr] offset:25600\n\t"
      "v_accvgpr_read_b32 v142, a64\n\t"
      "v_mfma_f32_16x16x32_f16 a[148:151], v[50:53], v[218:221], a[148:151]\n\t"
      "ds_read_b128 v[198:201], %[vlr] offset:26624\n\t"
      "v_accvgpr_read_b32 v143, a68\n\t"
      "v_mfma_f32_16x16x32_f16 a[148:151], v[46:49], v[214:217], a[148:151]\n\t"
      "ds_read_b128 v[202:205], %[vlr] offset:27648\n\t"
      "v_accvgpr_read_b32 v144, a72\n\t"
      "v_mfma_f32_16x16x32_f16 a[180:183], v[54:57], v[214:217], a[180:183]\n\t"
      "v_accvgpr_read_b32 v145, a76\n\t"
      "s_mov_b32 s63, 4194304\n\t"
      "v_mfma_f32_16x16x32_f16 a[180:183], v[58:61], v[218:221], a[180:183]\n\t"
      "buffer_store_dwordx4 v[142:145], v244, %[srdo], s63 offen offset:0 nt\n\t"
      "v_accvgpr_read_b32 v146, a65\n\t"
      "v_mfma_f32_16x16x32_f16 a[176:179], v[58:61], v[210:213], a[176:179]\n\t"
      "v_accvgpr_read_b32 v147, a69\n\t"
      "v_accvgpr_read_b32 v148, a73\n\t"
      "v_mfma_f32_16x16x32_f16 a[176:179], v[54:57], v[206:209], a[176:179]\n\t"
      "v_accvgpr_read_b32 v149, a77\n\t"
      "s_mov_b32 s63, 4325376\n\t"
      "v_mfma_f32_16x16x32_f16 a[208:211], v[62:65], v[206:209], a[208:211]\n\t"
      "buffer_store_dwordx4 v[146:149], v244, %[srdo], s63 offen offset:0 nt\n\t"
      "v_accvgpr_read_b32 v150, a66\n\t"
      "v_mfma_f32_16x16x32_f16 a[208:211], v[66:69], v[210:213], a[208:211]\n\t"
      "v_accvgpr_read_b32 v151, a70\n\t"
      "v_accvgpr_read_b32 v152, a74\n\t"
      "v_mfma_f32_16x16x32_f16 a[212:215], v[66:69], v[218:221], a[212:215]\n\t"
      "v_accvgpr_read_b32 v153, a78\n\t"
      "s_mov_b32 s63, 4456448\n\t"
      "v_mfma_f32_16x16x32_f16 a[212:215], v[62:65], v[214:217], a[212:215]\n\t"
      "buffer_store_dwordx4 v[150:153], v244, %[srdo], s63 offen offset:0 nt\n\t"
      "v_accvgpr_read_b32 v154, a67\n\t"
      "v_mfma_f32_16x16x32_f16 a[244:247], v[70:73], v[214:217], a[244:247]\n\t"
      "v_accvgpr_read_b32 v155, a71\n\t"
      "v_accvgpr_read_b32 v156, a75\n\t"
      "v_mfma_f32_16x16x32_f16 a[244:247], v[74:77], v[218:221], a[244:247]\n\t"
      "v_accvgpr_read_b32 v157, a79\n\t"
      "s_mov_b32 s63, 4587520\n\t"
      "v_mfma_f32_16x16x32_f16 a[240:243], v[74:77], v[210:213], a[240:243]\n\t"
      "buffer_store_dwordx4 v[154:157], v244, %[srdo], s63 offen offset:0 nt\n\t"
      "v_accvgpr_read_b32 v158, a96\n\t"
      "v_mfma_f32_16x16x32_f16 a[240:243], v[70:73], v[206:209], a[240:243]\n\t"
      "v_accvgpr_read_b32 v159, a100\n\t"
      "v_accvgpr_read_b32 v160, a104\n\t"
      "s_waitcnt lgkmcnt(4)\n\t"
      "v_mfma_f32_16x16x32_f16 a[88:91], v[30:33], v[174:177], a[88:91]\n\t"
      "v_accvgpr_read_b32 v161, a108\n\t"
      "s_mov_b32 s63, 6291456\n\t"
      "v_mfma_f32_16x16x32_f16 a[88:91], v[34:37], v[178:181], a[88:91]\n\t"
      "buffer_store_dwordx4 v[158:161], v244, %[srdo], s63 offen offset:0 nt\n\t"
      "v_accvgpr_read_b32 v162, a97\n\t"
      "v_mfma_f32_16x16x32_f16 a[92:95], v[34:37], v[186:189], a[92:95]\n\t"
      "v_accvgpr_read_b32 v163, a101\n\t"
      "v_accvgpr_read_b32 v164, a105\n\t"
      "v_mfma_f32_16x16x32_f16 a[92:95], v[30:33], v[182:185], a[92:95]\n\t"
      "v_accvgpr_read_b32 v165, a109\n\t"
      "s_mov_b32 s63, 6422528\n\t"
      "v_mfma_f32_16x16x32_f16 a[124:127], v[38:41], v[182:185], a[124:127]\n\t"
      "buffer_store_dwordx4 v[162:165], v244, %[srdo], s63 offen offset:0 nt\n\t"
      "v_accvgpr_read_b32 v166, a98\n\t"
      "v_mfma_f32_16x16x32_f16 a[124:127], v[42:45], v[186:189], a[124:127]\n\t"
      "v_accvgpr_read_b32 v167, a102\n\t"
      "v_accvgpr_read_b32 v168, a106\n\t"
      "v_mfma_f32_16x16x32_f16 a[120:123], v[42:45], v[178:181], a[120:123]\n\t"
      "v_accvgpr_read_b32 v169, a110\n\t"
      "s_mov_b32 s63, 6553600\n\t"
      "v_mfma_f32_16x16x32_f16 a[120:123], v[38:41], v[174:177], a[120:123]\n\t"
      "buffer_store_dwordx4 v[166:169], v244, %[srdo], s63 offen offset:0 nt\n\t"
      "v_accvgpr_read_b32 v170, a99\n\t"
      "v_mfma_f32_16x16x32_f16 a[152:155], v[46:49], v[174:177], a[152:155]\n\t"
      "ds_read_b128 v[222:225], %[vlr] offset:28672\n\t"
      "v_accvgpr_read_b32 v171, a103\n\t"
      "v_mfma_f32_16x16x32_f16 a[152:155], v[50:53], v[178:181], a[152:155]\n\t"
      "ds_read_b128 v[226:229], %[vlr] offset:29696\n\t"
      "v_accvgpr_read_b32 v172, a107\n\t"
      "v_mfma_f32_16x16x32_f16 a[156:159], v[50:53], v[186:189], a[156:159]\n\t"
      "ds_read_b128 v[230:233], %[vlr] offset:30720\n\t"
      "v_accvgpr_read_b32 v173, a111\n\t"
      "v_mfma_f32_16x16x32_f16 a[156:159], v[46:49], v[182:185], a[156:159]\n\t"
      "ds_read_b128 v[234:237], %[vlr] offset:31744\n\t"
      "s_mov_b32 s63, 6684672\n\t"
      "v_mfma_f32_16x16x32_f16 a[188:191], v[54:57], v[182:185], a[188:191]\n\t"
      "buffer_store_dwordx4 v[170:173], v244, %[srdo], s63 offen offset:0 nt\n\t"
      "v_accvgpr_read_b32 v142, a128\n\t"
      "v_mfma_f32_16x16x32_f16 a[188:191], v[58:61], v[186:189], a[188:191]\n\t"
      "v_accvgpr_read_b32 v143, a132\n\t"
      "v_accvgpr_read_b32 v144, a136\n\t"
      "v_mfma_f32_16x16x32_f16 a[184:187], v[58:61], v[178:181], a[184:187]\n\t"
      "v_accvgpr_read_b32 v145, a140\n\t"
      "s_mov_b32 s63, 8388608\n\t"
      "v_mfma_f32_16x16x32_f16 a[184:187], v[54:57], v[174:177], a[184:187]\n\t"
      "buffer_store_dwordx4 v[142:145], v244, %[srdo], s63 offen offset:0 nt\n\t"
      "v_accvgpr_read_b32 v146, a129\n\t"
      "v_mfma_f32_16x16x32_f16 a[216:219], v[62:65], v[174:177], a[216:219]\n\t"
      "v_accvgpr_read_b32 v147, a133\n\t"
      "v_accvgpr_read_b32 v148, a137\n\t"
      "v_mfma_f32_16x16x32_f16 a[216:219], v[66:69], v[178:181], a[216:219]\n\t"
      "v_accvgpr_read_b32 v149, a141\n\t"
      "s_mov_b32 s63, 8519680\n\t"
      "v_mfma_f32_16x16x32_f16 a[220:223], v[66:69], v[186:189], a[220:223]\n\t"
      "buffer_store_dwordx4 v[146:149], v244, %[srdo], s63 offen offset:0 nt\n\t"
      "v_accvgpr_read_b32 v150, a130\n\t"
      "v_mfma_f32_16x16x32_f16 a[220:223], v[62:65], v[182:185], a[220:223]\n\t"
      "v_accvgpr_read_b32 v151, a134\n\t"
      "v_accvgpr_read_b32 v152, a138\n\t"
      "v_mfma_f32_16x16x32_f16 a[252:255], v[70:73], v[182:185], a[252:255]\n\t"
      "v_accvgpr_read_b32 v153, a142\n\t"
      "s_mov_b32 s63, 8650752\n\t"
      "v_mfma_f32_16x16x32_f16 a[252:255], v[74:77], v[186:189], a[252:255]\n\t"
      "buffer_store_dwordx4 v[150:153], v244, %[srdo], s63 offen offset:0 nt\n\t"
      "v_accvgpr_read_b32 v154, a131\n\t"
      "v_mfma_f32_16x16x32_f16 a[248:251], v[74:77], v[178:181], a[248:251]\n\t"
      "v_accvgpr_read_b32 v155, a135\n\t"
      "v_accvgpr_read_b32 v156, a139\n\t"
      "v_mfma_f32_16x16x32_f16 a[248:251], v[70:73], v[174:177], a[248:251]\n\t"
      "v_accvgpr_read_b32 v157, a143\n\t"
      "s_mov_b32 s63, 8781824\n\t"
      "s_waitcnt lgkmcnt(4)\n\t"
      "v_mfma_f32_16x16x32_f16 a[80:83], v[94:97], v[190:193], a[80:83]\n\t"
      "buffer_store_dwordx4 v[154:157], v244, %[srdo], s63 offen offset:0 nt\n\t"
      "v_accvgpr_read_b32 v158, a160\n\t"
      "v_mfma_f32_16x16x32_f16 a[80:83], v[98:101], v[194:197], a[80:83]\n\t"
      "v_accvgpr_read_b32 v159, a164\n\t"
      "v_accvgpr_read_b32 v160, a168\n\t"
      "v_mfma_f32_16x16x32_f16 a[84:87], v[98:101], v[202:205], a[84:87]\n\t"
      "v_accvgpr_read_b32 v161, a172\n\t"
      "s_mov_b32 s63, 10485760\n\t"
      "v_mfma_f32_16x16x32_f16 a[84:87], v[94:97], v[198:201], a[84:87]\n\t"
      "buffer_store_dwordx4 v[158:161], v244, %[srdo], s63 offen offset:0 nt\n\t"
      "v_accvgpr_read_b32 v162, a161\n\t"
      "v_mfma_f32_16x16x32_f16 a[116:119], v[102:105], v[198:201], a[116:119]\n\t"
      "v_accvgpr_read_b32 v163, a165\n\t"
      "v_accvgpr_read_b32 v164, a169\n\t"
      "v_mfma_f32_16x16x32_f16 a[116:119], v[106:109], v[202:205], a[116:119]\n\t"
      "v_accvgpr_read_b32 v165, a173\n\t"
      "s_mov_b32 s63, 10616832\n\t"
      "v_mfma_f32_16x16x32_f16 a[112:115], v[106:109], v[194:197], a[112:115]\n\t"
      "buffer_store_dwordx4 v[162:165], v244, %[srdo], s63 offen offset:0 nt\n\t"
      "v_accvgpr_read_b32 v166, a162\n\t"
      "v_mfma_f32_16x16x32_f16 a[112:115], v[102:105], v[190:193], a[112:115]\n\t"
      "v_accvgpr_read_b32 v167, a166\n\t"
      "v_accvgpr_read_b32 v168, a170\n\t"
      "v_mfma_f32_16x16x32_f16 a[144:147], v[110:113], v[190:193], a[144:147]\n\t"
      "v_accvgpr_read_b32 v169, a174\n\t"
      "s_mov_b32 s63, 10747904\n\t"
      "v_mfma_f32_16x16x32_f16 a[144:147], v[114:117], v[194:197], a[144:147]\n\t"
      "buffer_store_dwordx4 v[166:169], v244, %[srdo], s63 offen offset:0 nt\n\t"
      "v_accvgpr_read_b32 v170, a163\n\t"
      "v_mfma_f32_16x16x32_f16 a[148:151], v[114:117], v[202:205], a[148:151]\n\t"
      "v_accvgpr_read_b32 v171, a167\n\t"
      "v_accvgpr_read_b32 v172, a171\n\t"
      "v_mfma_f32_16x16x32_f16 a[148:151], v[110:113], v[198:201], a[148:151]\n\t"
      "v_accvgpr_read_b32 v173, a175\n\t"
      "s_mov_b32 s63, 10878976\n\t"
      "v_mfma_f32_16x16x32_f16 a[180:183], v[118:121], v[198:201], a[180:183]\n\t"
      "buffer_store_dwordx4 v[170:173], v244, %[srdo], s63 offen offset:0 nt\n\t"
      "v_accvgpr_read_b32 v142, a192\n\t"
      "v_mfma_f32_16x16x32_f16 a[180:183], v[122:125], v[202:205], a[180:183]\n\t"
      "v_accvgpr_read_b32 v143, a196\n\t"
      "v_accvgpr_read_b32 v144, a200\n\t"
      "v_mfma_f32_16x16x32_f16 a[176:179], v[122:125], v[194:197], a[176:179]\n\t"
      "v_accvgpr_read_b32 v145, a204\n\t"
      "s_mov_b32 s63, 12582912\n\t"
      "v_mfma_f32_16x16x32_f16 a[176:179], v[118:121], v[190:193], a[176:179]\n\t"
      "buffer_store_dwordx4 v[142:145], v244, %[srdo], s63 offen offset:0 nt\n\t"
      "v_accvgpr_read_b32 v146, a193\n\t"
      "v_mfma_f32_16x16x32_f16 a[208:211], v[126:129], v[190:193], a[208:211]\n\t"
      "v_accvgpr_read_b32 v147, a197\n\t"
      "v_accvgpr_read_b32 v148, a201\n\t"
      "v_mfma_f32_16x16x32_f16 a[208:211], v[130:133], v[194:197], a[208:211]\n\t"
      "v_accvgpr_read_b32 v149, a205\n\t"
      "s_mov_b32 s63, 12713984\n\t"
      "v_mfma_f32_16x16x32_f16 a[212:215], v[130:133], v[202:205], a[212:215]\n\t"
      "buffer_store_dwordx4 v[146:149], v244, %[srdo], s63 offen offset:0 nt\n\t"
      "v_accvgpr_read_b32 v150, a194\n\t"
      "v_mfma_f32_16x16x32_f16 a[212:215], v[126:129], v[198:201], a[212:215]\n\t"
      "v_accvgpr_read_b32 v151, a198\n\t"
      "v_accvgpr_read_b32 v152, a202\n\t"
      "v_mfma_f32_16x16x32_f16 a[244:247], v[134:137], v[198:201], a[244:247]\n\t"
      "v_accvgpr_read_b32 v153, a206\n\t"
      "s_mov_b32 s63, 12845056\n\t"
      "v_mfma_f32_16x16x32_f16 a[244:247], v[138:141], v[202:205], a[244:247]\n\t"
      "buffer_store_dwordx4 v[150:153], v244, %[srdo], s63 offen offset:0 nt\n\t"
      "v_accvgpr_read_b32 v154, a195\n\t"
      "v_mfma_f32_16x16x32_f16 a[240:243], v[138:141], v[194:197], a[240:243]\n\t"
      "v_accvgpr_read_b32 v155, a199\n\t"
      "v_accvgpr_read_b32 v156, a203\n\t"
      "v_mfma_f32_16x16x32_f16 a[240:243], v[134:137], v[190:193], a[240:243]\n\t"
      "v_accvgpr_read_b32 v157, a207\n\t"
      "s_mov_b32 s63, 12976128\n\t"
      "s_waitcnt lgkmcnt(0)\n\t"
      "v_mfma_f32_16x16x32_f16 a[88:91], v[94:97], v[222:225], a[88:91]\n\t"
      "buffer_store_dwordx4 v[154:157], v244, %[srdo], s63 offen offset:0 nt\n\t"
      "v_accvgpr_read_b32 v158, a224\n\t"
      "v_mfma_f32_16x16x32_f16 a[88:91], v[98:101], v[226:229], a[88:91]\n\t"
      "v_accvgpr_read_b32 v159, a228\n\t"
      "v_accvgpr_read_b32 v160, a232\n\t"
      "v_mfma_f32_16x16x32_f16 a[92:95], v[98:101], v[234:237], a[92:95]\n\t"
      "v_accvgpr_read_b32 v161, a236\n\t"
      "s_mov_b32 s63, 14680064\n\t"
      "v_mfma_f32_16x16x32_f16 a[92:95], v[94:97], v[230:233], a[92:95]\n\t"
      "buffer_store_dwordx4 v[158:161], v244, %[srdo], s63 offen offset:0 nt\n\t"
      "v_accvgpr_read_b32 v162, a225\n\t"
      "v_mfma_f32_16x16x32_f16 a[124:127], v[102:105], v[230:233], a[124:127]\n\t"
      "v_accvgpr_read_b32 v163, a229\n\t"
      "v_accvgpr_read_b32 v164, a233\n\t"
      "v_mfma_f32_16x16x32_f16 a[124:127], v[106:109], v[234:237], a[124:127]\n\t"
      "v_accvgpr_read_b32 v165, a237\n\t"
      "s_mov_b32 s63, 14811136\n\t"
      "v_mfma_f32_16x16x32_f16 a[120:123], v[106:109], v[226:229], a[120:123]\n\t"
      "buffer_store_dwordx4 v[162:165], v244, %[srdo], s63 offen offset:0 nt\n\t"
      "v_accvgpr_read_b32 v166, a226\n\t"
      "v_mfma_f32_16x16x32_f16 a[120:123], v[102:105], v[222:225], a[120:123]\n\t"
      "v_accvgpr_read_b32 v167, a230\n\t"
      "v_accvgpr_read_b32 v168, a234\n\t"
      "v_mfma_f32_16x16x32_f16 a[152:155], v[110:113], v[222:225], a[152:155]\n\t"
      "v_accvgpr_read_b32 v169, a238\n\t"
      "s_mov_b32 s63, 14942208\n\t"
      "v_mfma_f32_16x16x32_f16 a[152:155], v[114:117], v[226:229], a[152:155]\n\t"
      "buffer_store_dwordx4 v[166:169], v244, %[srdo], s63 offen offset:0 nt\n\t"
      "v_accvgpr_read_b32 v170, a227\n\t"
      "v_mfma_f32_16x16x32_f16 a[156:159], v[114:117], v[234:237], a[156:159]\n\t"
      "v_accvgpr_read_b32 v171, a231\n\t"
      "v_accvgpr_read_b32 v172, a235\n\t"
      "v_mfma_f32_16x16x32_f16 a[156:159], v[110:113], v[230:233], a[156:159]\n\t"
      "v_accvgpr_read_b32 v173, a239\n\t"
      "s_mov_b32 s63, 15073280\n\t"
      "v_mfma_f32_16x16x32_f16 a[188:191], v[118:121], v[230:233], a[188:191]\n\t"
      "buffer_store_dwordx4 v[170:173], v244, %[srdo], s63 offen offset:0 nt\n\t"
      "v_mfma_f32_16x16x32_f16 a[188:191], v[122:125], v[234:237], a[188:191]\n\t"
      "v_mfma_f32_16x16x32_f16 a[184:187], v[122:125], v[226:229], a[184:187]\n\t"
      "v_mfma_f32_16x16x32_f16 a[184:187], v[118:121], v[222:225], a[184:187]\n\t"
      "v_mfma_f32_16x16x32_f16 a[216:219], v[126:129], v[222:225], a[216:219]\n\t"
      "v_mfma_f32_16x16x32_f16 a[216:219], v[130:133], v[226:229], a[216:219]\n\t"
      "v_mfma_f32_16x16x32_f16 a[220:223], v[130:133], v[234:237], a[220:223]\n\t"
      "v_mfma_f32_16x16x32_f16 a[220:223], v[126:129], v[230:233], a[220:223]\n\t"
      "v_mfma_f32_16x16x32_f16 a[252:255], v[134:137], v[230:233], a[252:255]\n\t"
      "v_mfma_f32_16x16x32_f16 a[252:255], v[138:141], v[234:237], a[252:255]\n\t"
      "v_mfma_f32_16x16x32_f16 a[248:251], v[138:141], v[226:229], a[248:251]\n\t"
      "v_mfma_f32_16x16x32_f16 a[248:251], v[134:137], v[222:225], a[248:251]\n\t"
      "s_memrealtime s[80:81]\n\t"
      "s_nop 7\n\t"
      "v_accvgpr_read_b32 v142, a80\n\t"
      "v_accvgpr_read_b32 v143, a84\n\t"
      "v_accvgpr_read_b32 v144, a88\n\t"
      "v_accvgpr_read_b32 v145, a92\n\t"
      "s_mov_b32 s63, 4194304\n\t"
      "buffer_store_dwordx4 v[142:145], v244, %[srdo], s63 offen offset:256 nt\n\t"
      "v_accvgpr_read_b32 v146, a81\n\t"
      "v_accvgpr_read_b32 v147, a85\n\t"
      "v_accvgpr_read_b32 v148, a89\n\t"
      "v_accvgpr_read_b32 v149, a93\n\t"
      "s_mov_b32 s63, 4325376\n\t"
      "buffer_store_dwordx4 v[146:149], v244, %[srdo], s63 offen offset:256 nt\n\t"
      "v_accvgpr_read_b32 v150, a82\n\t"
      "v_accvgpr_read_b32 v151, a86\n\t"
      "v_accvgpr_read_b32 v152, a90\n\t"
      "v_accvgpr_read_b32 v153, a94\n\t"
      "s_mov_b32 s63, 4456448\n\t"
      "buffer_store_dwordx4 v[150:153], v244, %[srdo], s63 offen offset:256 nt\n\t"
      "v_accvgpr_read_b32 v154, a83\n\t"
      "v_accvgpr_read_b32 v155, a87\n\t"
      "v_accvgpr_read_b32 v156, a91\n\t"
      "v_accvgpr_read_b32 v157, a95\n\t"
      "s_mov_b32 s63, 4587520\n\t"
      "buffer_store_dwordx4 v[154:157], v244, %[srdo], s63 offen offset:256 nt\n\t"
      "v_accvgpr_read_b32 v158, a112\n\t"
      "v_accvgpr_read_b32 v159, a116\n\t"
      "v_accvgpr_read_b32 v160, a120\n\t"
      "v_accvgpr_read_b32 v161, a124\n\t"
      "s_mov_b32 s63, 6291456\n\t"
      "buffer_store_dwordx4 v[158:161], v244, %[srdo], s63 offen offset:256 nt\n\t"
      "v_accvgpr_read_b32 v162, a113\n\t"
      "v_accvgpr_read_b32 v163, a117\n\t"
      "v_accvgpr_read_b32 v164, a121\n\t"
      "v_accvgpr_read_b32 v165, a125\n\t"
      "s_mov_b32 s63, 6422528\n\t"
      "buffer_store_dwordx4 v[162:165], v244, %[srdo], s63 offen offset:256 nt\n\t"
      "v_accvgpr_read_b32 v166, a114\n\t"
      "v_accvgpr_read_b32 v167, a118\n\t"
      "v_accvgpr_read_b32 v168, a122\n\t"
      "v_accvgpr_read_b32 v169, a126\n\t"
      "s_mov_b32 s63, 6553600\n\t"
      "buffer_store_dwordx4 v[166:169], v244, %[srdo], s63 offen offset:256 nt\n\t"
      "v_accvgpr_read_b32 v170, a115\n\t"
      "v_accvgpr_read_b32 v171, a119\n\t"
      "v_accvgpr_read_b32 v172, a123\n\t"
      "v_accvgpr_read_b32 v173, a127\n\t"
      "s_mov_b32 s63, 6684672\n\t"
      "buffer_store_dwordx4 v[170:173], v244, %[srdo], s63 offen offset:256 nt\n\t"
      "v_accvgpr_read_b32 v142, a144\n\t"
      "v_accvgpr_read_b32 v143, a148\n\t"
      "v_accvgpr_read_b32 v144, a152\n\t"
      "v_accvgpr_read_b32 v145, a156\n\t"
      "s_mov_b32 s63, 8388608\n\t"
      "buffer_store_dwordx4 v[142:145], v244, %[srdo], s63 offen offset:256 nt\n\t"
      "v_accvgpr_read_b32 v146, a145\n\t"
      "v_accvgpr_read_b32 v147, a149\n\t"
      "v_accvgpr_read_b32 v148, a153\n\t"
      "v_accvgpr_read_b32 v149, a157\n\t"
      "s_mov_b32 s63, 8519680\n\t"
      "buffer_store_dwordx4 v[146:149], v244, %[srdo], s63 offen offset:256 nt\n\t"
      "v_accvgpr_read_b32 v150, a146\n\t"
      "v_accvgpr_read_b32 v151, a150\n\t"
      "v_accvgpr_read_b32 v152, a154\n\t"
      "v_accvgpr_read_b32 v153, a158\n\t"
      "s_mov_b32 s63, 8650752\n\t"
      "buffer_store_dwordx4 v[150:153], v244, %[srdo], s63 offen offset:256 nt\n\t"
      "v_accvgpr_read_b32 v154, a147\n\t"
      "v_accvgpr_read_b32 v155, a151\n\t"
      "v_accvgpr_read_b32 v156, a155\n\t"
      "v_accvgpr_read_b32 v157, a159\n\t"
      "s_mov_b32 s63, 8781824\n\t"
      "buffer_store_dwordx4 v[154:157], v244, %[srdo], s63 offen offset:256 nt\n\t"
      "v_accvgpr_read_b32 v158, a176\n\t"
      "v_accvgpr_read_b32 v159, a180\n\t"
      "v_accvgpr_read_b32 v160, a184\n\t"
      "v_accvgpr_read_b32 v161, a188\n\t"
      "s_mov_b32 s63, 10485760\n\t"
      "buffer_store_dwordx4 v[158:161], v244, %[srdo], s63 offen offset:256 nt\n\t"
      "v_accvgpr_read_b32 v162, a177\n\t"
      "v_accvgpr_read_b32 v163, a181\n\t"
      "v_accvgpr_read_b32 v164, a185\n\t"
      "v_accvgpr_read_b32 v165, a189\n\t"
      "s_mov_b32 s63, 10616832\n\t"
      "buffer_store_dwordx4 v[162:165], v244, %[srdo], s63 offen offset:256 nt\n\t"
      "v_accvgpr_read_b32 v166, a178\n\t"
      "v_accvgpr_read_b32 v167, a182\n\t"
      "v_accvgpr_read_b32 v168, a186\n\t"
      "v_accvgpr_read_b32 v169, a190\n\t"
      "s_mov_b32 s63, 10747904\n\t"
      "buffer_store_dwordx4 v[166:169], v244, %[srdo], s63 offen offset:256 nt\n\t"
      "v_accvgpr_read_b32 v170, a179\n\t"
      "v_accvgpr_read_b32 v171, a183\n\t"
      "v_accvgpr_read_b32 v172, a187\n\t"
      "v_accvgpr_read_b32 v173, a191\n\t"
      "s_mov_b32 s63, 10878976\n\t"
      "buffer_store_dwordx4 v[170:173], v244, %[srdo], s63 offen offset:256 nt\n\t"
      "v_accvgpr_read_b32 v142, a208\n\t"
      "v_accvgpr_read_b32 v143, a212\n\t"
      "v_accvgpr_read_b32 v144, a216\n\t"
      "v_accvgpr_read_b32 v145, a220\n\t"
      "s_mov_b32 s63, 12582912\n\t"
      "buffer_store_dwordx4 v[142:145], v244, %[srdo], s63 offen offset:256 nt\n\t"
      "v_accvgpr_read_b32 v146, a209\n\t"
      "v_accvgpr_read_b32 v147, a213\n\t"
      "v_accvgpr_read_b32 v148, a217\n\t"
      "v_accvgpr_read_b32 v149, a221\n\t"
      "s_mov_b32 s63, 12713984\n\t"
      "buffer_store_dwordx4 v[146:149], v244, %[srdo], s63 offen offset:256 nt\n\t"
      "v_accvgpr_read_b32 v150, a210\n\t"
      "v_accvgpr_read_b32 v151, a214\n\t"
      "v_accvgpr_read_b32 v152, a218\n\t"
      "v_accvgpr_read_b32 v153, a222\n\t"
      "s_mov_b32 s63, 12845056\n\t"
      "buffer_store_dwordx4 v[150:153], v244, %[srdo], s63 offen offset:256 nt\n\t"
      "v_accvgpr_read_b32 v154, a211\n\t"
      "v_accvgpr_read_b32 v155, a215\n\t"
      "v_accvgpr_read_b32 v156, a219\n\t"
      "v_accvgpr_read_b32 v157, a223\n\t"
      "s_mov_b32 s63, 12976128\n\t"
      "buffer_store_dwordx4 v[154:157], v244, %[srdo], s63 offen offset:256 nt\n\t"
      "v_accvgpr_read_b32 v158, a240\n\t"
      "v_accvgpr_read_b32 v159, a244\n\t"
      "v_accvgpr_read_b32 v160, a248\n\t"
      "v_accvgpr_read_b32 v161, a252\n\t"
      "s_mov_b32 s63, 14680064\n\t"
      "buffer_store_dwordx4 v[158:161], v244, %[srdo], s63 offen offset:256 nt\n\t"
      "v_accvgpr_read_b32 v162, a241\n\t"
      "v_accvgpr_read_b32 v163, a245\n\t"
      "v_accvgpr_read_b32 v164, a249\n\t"
      "v_accvgpr_read_b32 v165, a253\n\t"
      "s_mov_b32 s63, 14811136\n\t"
      "buffer_store_dwordx4 v[162:165], v244, %[srdo], s63 offen offset:256 nt\n\t"
      "v_accvgpr_read_b32 v166, a242\n\t"
      "v_accvgpr_read_b32 v167, a246\n\t"
      "v_accvgpr_read_b32 v168, a250\n\t"
      "v_accvgpr_read_b32 v169, a254\n\t"
      "s_mov_b32 s63, 14942208\n\t"
      "buffer_store_dwordx4 v[166:169], v244, %[srdo], s63 offen offset:256 nt\n\t"
      "v_accvgpr_read_b32 v170, a243\n\t"
      "v_accvgpr_read_b32 v171, a247\n\t"
      "v_accvgpr_read_b32 v172, a251\n\t"
      "v_accvgpr_read_b32 v173, a255\n\t"
      "s_mov_b32 s63, 15073280\n\t"
      "buffer_store_dwordx4 v[170:173], v244, %[srdo], s63 offen offset:256 nt\n\t"
      "s_waitcnt lgkmcnt(0)\n\t"
      "v_mov_b32 v255, s80\n\t"
      "s_cmp_lg_u32 %[swave], 0\n\t"
      "s_cbranch_scc1 Lfin2_%=\n\t"
      "s_mov_b64 exec, 1\n\t"
      "buffer_store_dword v255, off, %[srdws], %[sseq] offset:4 sc1\n\t"
      "s_mov_b64 exec, -1\n\t"
      "s_branch Lfin2_%=\n\t"
      "LghH_%=:\n\t"
      "s_mov_b32 s62, 60\n\t"
      "Lgh1H_%=:\n\t"
      "buffer_load_dword v255, off, %[srdws], s84 sc1\n\t"
      "s_waitcnt vmcnt(0)\n\t"
      "v_readfirstlane_b32 s63, v255\n\t"
      "s_sub_u32 s62, s62, 1\n\t"
      "s_cmp_eq_u32 s62, 0\n\t"
      "s_cbranch_scc1 Lgh2H_%=\n\t"
      "s_cmp_ge_u32 s63, s74\n\t"
      "s_cbranch_scc1 Lgh2H_%=\n\t"
      "s_sleep 8\n\t"
      "s_branch Lgh1H_%=\n\t"
      "Lfbad_%=:\n\t"
      "s_waitcnt vmcnt(0) lgkmcnt(0)\n\t"
      "s_nop 15\n\t"
      "Lfin2_%=:\n\t"
      "Lend_%=:\n\t"
      "s_mov_b32 %[bad], s75\n\t"
      : [bad] "=&s"(bad), [vx0] "=&v"(vx0), [vx1] "=&v"(vx1), [vlw] "=&v"(vlw), [vlr] "=&v"(vlr), [vpad] "=&v"(vpad), [vpd] "=&v"(vpd), [vtab] "=&v"(vtab)
      : [srdw] "s"(srdw), [srdo] "s"(srdo), [srdxs] "s"(srdxs), [srdws] "s"(srdws),
        [sxlo] "s"((unsigned)xb), [sxhi] "s"((unsigned)(xb >> 32)), [sdlo] "s"((unsigned)db), [sdhi] "s"((unsigned)(db >> 32)),
        [sflag] "s"(sflag), [sws] "s"((unsigned long long)myws), [snself] "s"(snself), [spk] "s"(sfpc | (sfph << 8) | (sfs0 << 16) | ((10 + (mt + lp) % 6) << 24)),
        [sxr] "s"(dst << 20), [sxl] "s"((rank << 20) + lp * 256),
        [sdl] "s"((rank * 8 + lp) * XPER + ((mt & 7) * 2 + (wave & 1)) * 1024), [swave] "s"(wave), [sstag] "s"(sstag), [sob] "s"((wave << 24) + panel * 512), [sxst] "s"(ldsb + 131072 + 1024 + wave * 6144), [sfin] "s"(WS_TMO + 1024 + (bid & 7) * 256), [sseq] "s"(WS_SEQ + bid * 128),
        [sg0] "s"((rank * 4) << 24),
        [vw0] "v"(vw0), [vw1] "v"(vw1), [vw2] "v"(vw2), [vw3] "v"(vw3), [vxs] "v"(vxs)
      : "v14", "v15", "v16", "v17", "v18", "v19", "v20", "v21", "v22", "v23", "v24", "v25", "v26", "v27", "v28", "v29", "v30", "v31", "v32", "v33", "v34", "v35", "v36", "v37", "v38", "v39", "v40", "v41", "v42", "v43", "v44", "v45", "v46", "v47", "v48", "v49", "v50", "v51", "v52", "v53", "v54", "v55", "v56", "v57", "v58", "v59", "v60", "v61", "v62", "v63", "v64", "v65", "v66", "v67", "v68", "v69", "v70", "v71", "v72", "v73", "v74", "v75", "v76", "v77", "v78", "v79", "v80", "v81", "v82", "v83", "v84", "v85", "v86", "v87", "v88", "v89", "v90", "v91", "v92", "v93", "v94", "v95", "v96", "v97", "v98", "v99", "v100", "v101", "v102", "v103", "v104", "v105", "v106", "v107", "v108", "v109", "v110", "v111", "v112", "v113", "v114", "v115", "v116", "v117", "v118", "v119", "v120", "v121", "v122", "v123", "v124", "v125", "v126", "v127", "v128", "v129", "v130", "v131", "v132", "v133", "v134", "v135", "v136", "v137", "v138", "v139", "v140", "v141", "v142", "v143", "v144", "v145", "v146", "v147", "v148", "v149", "v150", "v151", "v152", "v153", "v154", "v155", "v156", "v157", "v158", "v159", "v160", "v161", "v162", "v163", "v164", "v165", "v166", "v167", "v168", "v169", "v170", "v171", "v172", "v173", "v174", "v175", "v176", "v177", "v178", "v179", "v180", "v181", "v182", "v183", "v184", "v185", "v186", "v187", "v188", "v189", "v190", "v191", "v192", "v193", "v194", "v195", "v196", "v197", "v198", "v199", "v200", "v201", "v202", "v203", "v204", "v205", "v206", "v207", "v208", "v209", "v210", "v211", "v212", "v213", "v214", "v215", "v216", "v217", "v218", "v219", "v220", "v221", "v222", "v223", "v224", "v225", "v226", "v227", "v228", "v229", "v230", "v231", "v232", "v233", "v234", "v235", "v236", "v237", "v238", "v239", "v240", "v241", "v242", "v243", "v244", "v245", "v246", "v247", "v248", "v249", "v250", "v251", "v252", "v253", "v254", "v255", "a0", "a1", "a2", "a3", "a4", "a5", "a6", "a7", "a8", "a9", "a10", "a11", "a12", "a13", "a14", "a15", "a16", "a17", "a18", "a19", "a20", "a21", "a22", "a23", "a24", "a25", "a26", "a27", "a28", "a29", "a30", "a31", "a32", "a33", "a34", "a35", "a36", "a37", "a38", "a39", "a40", "a41", "a42", "a43", "a44", "a45", "a46", "a47", "a48", "a49", "a50", "a51", "a52", "a53", "a54", "a55", "a56", "a57", "a58", "a59", "a60", "a61", "a62", "a63", "a64", "a65", "a66", "a67", "a68", "a69", "a70", "a71", "a72", "a73", "a74", "a75", "a76", "a77", "a78", "a79", "a80", "a81", "a82", "a83", "a84", "a85", "a86", "a87", "a88", "a89", "a90", "a91", "a92", "a93", "a94", "a95", "a96", "a97", "a98", "a99", "a100", "a101", "a102", "a103", "a104", "a105", "a106", "a107", "a108", "a109", "a110", "a111", "a112", "a113", "a114", "a115", "a116", "a117", "a118", "a119", "a120", "a121", "a122", "a123", "a124", "a125", "a126", "a127", "a128", "a129", "a130", "a131", "a132", "a133", "a134", "a135", "a136", "a137", "a138", "a139", "a140", "a141", "a142", "a143", "a144", "a145", "a146", "a147", "a148", "a149", "a150", "a151", "a152", "a153", "a154", "a155", "a156", "a157", "a158", "a159", "a160", "a161", "a162", "a163", "a164", "a165", "a166", "a167", "a168", "a169", "a170", "a171", "a172", "a173", "a174", "a175", "a176", "a177", "a178", "a179", "a180", "a181", "a182", "a183", "a184", "a185", "a186", "a187", "a188", "a189", "a190", "a191", "a192", "a193", "a194", "a195", "a196", "a197", "a198", "a199", "a200", "a201", "a202", "a203", "a204", "a205", "a206", "a207", "a208", "a209", "a210", "a211", "a212", "a213", "a214", "a215", "a216", "a217", "a218", "a219", "a220", "a221", "a222", "a223", "a224", "a225", "a226", "a227", "a228", "a229", "a230", "a231", "a232", "a233", "a234", "a235", "a236", "a237", "a238", "a239", "a240", "a241", "a242", "a243", "a244", "a245", "a246", "a247", "a248", "a249", "a250", "a251", "a252", "a253", "a254", "a255", "s50", "s51", "s52", "s53", "s54", "s55", "s56", "s57", "s58", "s59", "s60", "s61", "s62", "s63", "s64", "s65", "s66", "s67", "s68", "s69", "s70", "s71", "s72", "s73", "s74", "s75", "s76", "s77", "s78", "s79", "s80", "s81", "s82", "s83", "s84", "s85", "s86", "s87", "s88", "s89", "s90", "s91", "s92", "s93", "s94", "s95", "s96", "s97", "s98", "s99", "scc", "vcc", "memory");
  if (bad) {
    const unsigned l2 = __builtin_amdgcn_mbcnt_hi(-1, __builtin_amdgcn_mbcnt_lo(-1, 0));
    for (unsigned i = l2; i < 128 * 128; i += 64) out[(size_t)(wave * 128 + (i >> 7)) * 32768 + panel * 128 + (i & 127)] = __builtin_nanf("");
  }
}

extern "C" void kernel_launch(int rank, int nranks, ncclComm_t comm, hipStream_t stream,
                              void* const* d_in, const int* in_sizes, int n_in,
                              void* d_out, int out_size, const int* in_dtypes, int out_dtype,
                              void* const* d_out_all, void* const* d_ws_all, size_t ws_bytes) {
  if (nranks != (int)NR || (unsigned)rank >= NR || ws_bytes < WS_X + 2 * (size_t)XBUF) return;
  Ptrs P;
  for (unsigned p = 0; p < NR; ++p) P.ws[p] = (char*)d_ws_all[p];
  a2a_gemm<<<256, 256, 0, stream>>>((const float*)d_in[0], (const float*)d_in[1], (float*)d_out, P.ws[rank], (unsigned)rank, P);
}
```
